```python
import numpy as np

kernel_name = "gpu_ag_gemm_m4096_k4096_n16384_e5m2e5m2_dequant"


ABSMAX_RTOL = 0.05

COMPUTE_DTYPE = "float32"
EPILOGUE = "dequant"
IN_QUANT = None
OUT_QUANT = None
IN_FP8 = "e5m2e5m2"
IN_DTYPES = {"x": "e5m2", "w_mat": "e5m2",
             "scale_x": "float32", "scale_w": "float32"}
WORLD_SIZE = 8

def _snap_fp8(x, mant_bits, max_abs, min_normal):
    a = np.abs(x.astype(np.float64))
    _m, _e = np.frexp(a)
    q = np.where(a >= min_normal, np.ldexp(1.0, _e - 1 - mant_bits),
                 min_normal * 2.0 ** -mant_bits)
    snapped = np.minimum(np.round(a / q) * q, max_abs)
    return (np.sign(x) * snapped).astype(np.float32)

def setup_inputs():

    x = _snap_fp8(np.random.randn(4096, 4096).astype(np.float32), 2, 57344.0, 2.0 ** -14)
    w_mat = _snap_fp8(np.random.randn(4096, 16384).astype(np.float32), 2, 57344.0, 2.0 ** -14)
    scale_x = (0.02 * (1.0 + np.random.rand(1))).astype(np.float32)
    scale_w = (0.02 * (1.0 + np.random.rand(1))).astype(np.float32)
    return {"x": x, "w_mat": w_mat, "scale_x": scale_x, "scale_w": scale_w}

def reference(x, w_mat, scale_x, scale_w):
    acc = (x @ w_mat).astype(np.float32)
    y = (acc * (scale_x * scale_w)).astype(np.float32)
    return y
```

```cpp
#include <hip/hip_runtime.h>
#include <rccl/rccl.h>
#include <cstdio>
#include <cstdint>

#define HIP_CHECK(expr) do { hipError_t _e = (expr); if (_e != hipSuccess) { \
    fprintf(stderr, "HIP error %s at %s:%d\n", hipGetErrorString(_e), __FILE__, __LINE__); \
    return; } } while (0)

namespace agk {

#define LAS __attribute__((address_space(3)))
#define GAS __attribute__((address_space(1)))
#define CAS __attribute__((address_space(4)))
typedef int i32x8 __attribute__((ext_vector_type(8)));
typedef int i32x32 __attribute__((ext_vector_type(32)));
typedef unsigned u32x4 __attribute__((ext_vector_type(4)));
typedef float f32x4 __attribute__((ext_vector_type(4)));
typedef float f32x16 __attribute__((ext_vector_type(16)));
typedef float f32x32 __attribute__((ext_vector_type(32)));

constexpr int NR = 8;
constexpr int KDIM = 4096, NL = 2048;
constexpr int MS = 512;
constexpr int NKT = 32;
constexpr int NSTEPS = 2 * NKT;
constexpr int NWG = 256, NTHREADS = 512;
constexpr int SLOT_A = 20480;
constexpr int SLOT_BYTES = SLOT_A + 16384, NSLOT = 4;
constexpr unsigned LDS_CTRL = NSLOT * SLOT_BYTES;
constexpr int LDS_BYTES = LDS_CTRL + 512 + 2048;
constexpr unsigned C_FULL = LDS_CTRL;
constexpr unsigned C_READY = LDS_CTRL + 16;
constexpr unsigned C_BAD = LDS_CTRL + 20;
constexpr unsigned C_FREE = LDS_CTRL + 512;
constexpr size_t WS_FLAGS = 0;
constexpr size_t WS_CALL = 16384;
constexpr size_t WS_TMO = 32768;
constexpr size_t WS_STAMP = 65536;
constexpr size_t WS_GATHER = 1u << 20;
constexpr size_t SHARD_BYTES = (size_t)MS * KDIM;
constexpr size_t WS_NEEDED = WS_GATHER + 2 * NR * SHARD_BYTES;
constexpr unsigned long long DEADLINE_TICKS = 100000000ull;
#define AG_PWIN 4
constexpr int PWIN = AG_PWIN;

struct Args {
    const unsigned char* x;
    const unsigned char* w;
    const float* sx;
    const float* sw;
    float* out;
    unsigned char* ws[NR];
    int rank;
};

template <int N> __device__ __forceinline__ void wait_vm() { asm volatile("s_waitcnt vmcnt(%0)" ::"n"(N) : "memory"); }
__device__ __forceinline__ unsigned uni(unsigned v) { return (unsigned)__builtin_amdgcn_readfirstlane((int)v); }
__device__ __forceinline__ unsigned lds_read_uni(unsigned addr) {
    unsigned t, sv;
    asm volatile("v_mov_b32 %0, %2\n\t"
                 "ds_read_b32 %0, %0\n\t"
                 "s_waitcnt lgkmcnt(0)\n\t"
                 "v_readfirstlane_b32 %1, %0"
                 : "=&v"(t), "=s"(sv) : "s"(addr) : "memory");
    return sv;
}
__device__ __forceinline__ void lds_wait_all_ge(unsigned vaddr, unsigned thr) {
    unsigned t;
    asm volatile("1:\n\t"
                 "ds_read_b32 %0, %1\n\t"
                 "s_waitcnt lgkmcnt(0)\n\t"
                 "v_cmp_gt_u32 vcc, %2, %0\n\t"
                 "s_cbranch_vccnz 1b"
                 : "=&v"(t) : "v"(vaddr), "s"(thr) : "memory", "vcc");
}
__device__ __forceinline__ void lds_write_uni(unsigned addr, unsigned val) {
    unsigned t0, t1;
    asm volatile("s_nop 1\n\t"
                 "v_mov_b32 %0, %2\n\t"
                 "v_mov_b32 %1, %3\n\t"
                 "s_mov_b64 exec, 1\n\t"
                 "ds_write_b32 %0, %1\n\t"
                 "s_mov_b64 exec, -1"
                 : "=&v"(t0), "=&v"(t1) : "s"(addr), "s"(val) : "memory");
}
__device__ __forceinline__ void store_wt(float* p, f32x4 v) {
    asm volatile("global_store_dwordx4 %0, %1, off sc1\n\ts_nop 1" ::"v"(p), "v"(v) : "memory");
}

#define STAMP(idx) do { } while (0)
#define STAMPL(idx) do { } while (0)

#define AG_ASM_PROLOGUE \
    "0:\n" \
    "ds_read_b32 %[vt], %[fu]\n" \
    "s_waitcnt lgkmcnt(0)\n" \
    "v_cmp_eq_u32 vcc, 0, %[vt]\n" \
    "s_cbranch_vccnz 0b\n" \
    "ds_read_b128 v[32:35], %[a00] offset:0\n" \
    "ds_read_b128 v[36:39], %[a01] offset:0\n" \
    "ds_read_b64_tr_b8 v[72:73], %[b00] offset:0\n" \
    "ds_read_b64_tr_b8 v[74:75], %[b00] offset:1024\n" \
    "ds_read_b64_tr_b8 v[76:77], %[b00] offset:8192\n" \
    "ds_read_b64_tr_b8 v[78:79], %[b00] offset:9216\n" \
    "ds_read_b128 v[40:43], %[a00] offset:2048\n" \
    "ds_read_b128 v[44:47], %[a01] offset:2048\n" \
    "ds_read_b64_tr_b8 v[80:81], %[b01] offset:0\n" \
    "ds_read_b64_tr_b8 v[82:83], %[b01] offset:1024\n" \
    "ds_read_b64_tr_b8 v[84:85], %[b01] offset:8192\n" \
    "ds_read_b64_tr_b8 v[86:87], %[b01] offset:9216\n" \
    "ds_read_b128 v[48:51], %[a00] offset:4096\n" \
    "ds_read_b128 v[52:55], %[a01] offset:4096\n" \
    "ds_read_b64_tr_b8 v[88:89], %[b02] offset:0\n" \
    "ds_read_b64_tr_b8 v[90:91], %[b02] offset:1024\n" \
    "ds_read_b64_tr_b8 v[92:93], %[b02] offset:8192\n" \
    "ds_read_b64_tr_b8 v[94:95], %[b02] offset:9216\n" \
    "ds_read_b128 v[56:59], %[a00] offset:6144\n" \
    "ds_read_b128 v[60:63], %[a01] offset:6144\n" \
    "ds_read_b64_tr_b8 v[96:97], %[b03] offset:0\n" \
    "ds_read_b64_tr_b8 v[98:99], %[b03] offset:1024\n" \
    "ds_read_b64_tr_b8 v[100:101], %[b03] offset:8192\n" \
    "ds_read_b64_tr_b8 v[102:103], %[b03] offset:9216\n" \
    "ds_read_b128 v[64:67], %[a00] offset:8192\n" \
    "ds_read_b128 v[68:71], %[a01] offset:8192\n" \
    "ds_write_b32 %[fr], %[vt]\n" \
    "ds_read_b32 %[tk], %[fu] offset:4\n" \
    "s_waitcnt lgkmcnt(0)\n" \
    ""
#define AG_ASM_LOOP5 \
    "s_nop 1\n" \
    "0:\n" \
    "s_waitcnt lgkmcnt(14)\n" \
    "v_mfma_f32_16x16x128_f8f6f4 v[176:179], v[72:79], v[32:39], v[176:179] cbsz:1 blgp:1\n" \
    "s_add_u32 %[thr], %[u], 1\n" \
    "ds_read_b128 v[104:107], %[a00] offset:36864\n" \
    "ds_read_b128 v[108:111], %[a01] offset:36864\n" \
    "s_waitcnt lgkmcnt(14)\n" \
    "v_mfma_f32_16x16x128_f8f6f4 v[180:183], v[80:87], v[32:39], v[180:183] cbsz:1 blgp:1\n" \
    "ds_read_b64_tr_b8 v[144:145], %[b00] offset:36864\n" \
    "ds_read_b64_tr_b8 v[146:147], %[b00] offset:37888\n" \
    "s_waitcnt lgkmcnt(14)\n" \
    "v_mfma_f32_16x16x128_f8f6f4 v[184:187], v[88:95], v[32:39], v[184:187] cbsz:1 blgp:1\n" \
    "ds_read_b64_tr_b8 v[148:149], %[b00] offset:45056\n" \
    "ds_read_b64_tr_b8 v[150:151], %[b00] offset:46080\n" \
    "s_waitcnt lgkmcnt(6)\n" \
    "v_mfma_f32_16x16x128_f8f6f4 v[188:191], v[96:103], v[32:39], v[188:191] cbsz:1 blgp:1\n" \
    "v_cmp_ge_u32 vcc, %[thr], %[tk]\n" \
    "ds_read_b128 v[112:115], %[a00] offset:38912\n" \
    "ds_read_b128 v[116:119], %[a01] offset:38912\n" \
    "v_mfma_f32_16x16x128_f8f6f4 v[192:195], v[72:79], v[40:47], v[192:195] cbsz:1 blgp:1\n" \
    "ds_read_b64_tr_b8 v[152:153], %[b01] offset:36864\n" \
    "ds_read_b64_tr_b8 v[154:155], %[b01] offset:37888\n" \
    "v_mfma_f32_16x16x128_f8f6f4 v[196:199], v[80:87], v[40:47], v[196:199] cbsz:1 blgp:1\n" \
    "ds_read_b64_tr_b8 v[156:157], %[b01] offset:45056\n" \
    "ds_read_b64_tr_b8 v[158:159], %[b01] offset:46080\n" \
    "v_mfma_f32_16x16x128_f8f6f4 v[200:203], v[88:95], v[40:47], v[200:203] cbsz:1 blgp:1\n" \
    "ds_read_b128 v[120:123], %[a00] offset:40960\n" \
    "ds_read_b128 v[124:127], %[a01] offset:40960\n" \
    "v_mfma_f32_16x16x128_f8f6f4 v[204:207], v[96:103], v[40:47], v[204:207] cbsz:1 blgp:1\n" \
    "ds_read_b64_tr_b8 v[160:161], %[b02] offset:36864\n" \
    "ds_read_b64_tr_b8 v[162:163], %[b02] offset:37888\n" \
    "v_mfma_f32_16x16x128_f8f6f4 v[208:211], v[72:79], v[48:55], v[208:211] cbsz:1 blgp:1\n" \
    "ds_read_b64_tr_b8 v[164:165], %[b02] offset:45056\n" \
    "ds_read_b64_tr_b8 v[166:167], %[b02] offset:46080\n" \
    "v_mfma_f32_16x16x128_f8f6f4 v[212:215], v[80:87], v[48:55], v[212:215] cbsz:1 blgp:1\n" \
    "ds_read_b128 v[128:131], %[a00] offset:43008\n" \
    "ds_read_b128 v[132:135], %[a01] offset:43008\n" \
    "v_mfma_f32_16x16x128_f8f6f4 v[216:219], v[88:95], v[48:55], v[216:219] cbsz:1 blgp:1\n" \
    "ds_read_b64_tr_b8 v[168:169], %[b03] offset:36864\n" \
    "ds_read_b64_tr_b8 v[170:171], %[b03] offset:37888\n" \
    "v_mfma_f32_16x16x128_f8f6f4 v[220:223], v[96:103], v[48:55], v[220:223] cbsz:1 blgp:1\n" \
    "ds_read_b64_tr_b8 v[172:173], %[b03] offset:45056\n" \
    "ds_read_b64_tr_b8 v[174:175], %[b03] offset:46080\n" \
    "v_mfma_f32_16x16x128_f8f6f4 v[224:227], v[72:79], v[56:63], v[224:227] cbsz:1 blgp:1\n" \
    "ds_read_b128 v[136:139], %[a00] offset:45056\n" \
    "ds_read_b128 v[140:143], %[a01] offset:45056\n" \
    "v_mfma_f32_16x16x128_f8f6f4 v[228:231], v[80:87], v[56:63], v[228:231] cbsz:1 blgp:1\n" \
    "ds_write_b32 %[fr], %[tk] offset:512\n" \
    "ds_read_b32 %[tk], %[fu] offset:8\n" \
    "v_mfma_f32_16x16x128_f8f6f4 v[232:235], v[88:95], v[56:63], v[232:235] cbsz:1 blgp:1\n" \
    "v_mfma_f32_16x16x128_f8f6f4 v[236:239], v[96:103], v[56:63], v[236:239] cbsz:1 blgp:1\n" \
    "v_mfma_f32_16x16x128_f8f6f4 v[240:243], v[72:79], v[64:71], v[240:243] cbsz:1 blgp:1\n" \
    "v_mfma_f32_16x16x128_f8f6f4 v[244:247], v[80:87], v[64:71], v[244:247] cbsz:1 blgp:1\n" \
    "v_mfma_f32_16x16x128_f8f6f4 v[248:251], v[88:95], v[64:71], v[248:251] cbsz:1 blgp:1\n" \
    "v_mfma_f32_16x16x128_f8f6f4 v[252:255], v[96:103], v[64:71], v[252:255] cbsz:1 blgp:1\n" \
    "s_cbranch_vccnz 20f\n" \
    "10:\n" \
    "s_waitcnt lgkmcnt(14)\n" \
    "v_mfma_f32_16x16x128_f8f6f4 v[176:179], v[144:151], v[104:111], v[176:179] cbsz:1 blgp:1\n" \
    "s_add_u32 %[thr], %[u], 2\n" \
    "ds_read_b128 v[32:35], %[a10] offset:0\n" \
    "ds_read_b128 v[36:39], %[a11] offset:0\n" \
    "s_waitcnt lgkmcnt(14)\n" \
    "v_mfma_f32_16x16x128_f8f6f4 v[180:183], v[152:159], v[104:111], v[180:183] cbsz:1 blgp:1\n" \
    "ds_read_b64_tr_b8 v[72:73], %[b10] offset:0\n" \
    "ds_read_b64_tr_b8 v[74:75], %[b10] offset:1024\n" \
    "s_waitcnt lgkmcnt(14)\n" \
    "v_mfma_f32_16x16x128_f8f6f4 v[184:187], v[160:167], v[104:111], v[184:187] cbsz:1 blgp:1\n" \
    "ds_read_b64_tr_b8 v[76:77], %[b10] offset:8192\n" \
    "ds_read_b64_tr_b8 v[78:79], %[b10] offset:9216\n" \
    "s_waitcnt lgkmcnt(6)\n" \
    "v_mfma_f32_16x16x128_f8f6f4 v[188:191], v[168:175], v[104:111], v[188:191] cbsz:1 blgp:1\n" \
    "v_cmp_ge_u32 vcc, %[thr], %[tk]\n" \
    "ds_read_b128 v[40:43], %[a10] offset:2048\n" \
    "ds_read_b128 v[44:47], %[a11] offset:2048\n" \
    "v_mfma_f32_16x16x128_f8f6f4 v[192:195], v[144:151], v[112:119], v[192:195] cbsz:1 blgp:1\n" \
    "ds_read_b64_tr_b8 v[80:81], %[b11] offset:0\n" \
    "ds_read_b64_tr_b8 v[82:83], %[b11] offset:1024\n" \
    "v_mfma_f32_16x16x128_f8f6f4 v[196:199], v[152:159], v[112:119], v[196:199] cbsz:1 blgp:1\n" \
    "ds_read_b64_tr_b8 v[84:85], %[b11] offset:8192\n" \
    "ds_read_b64_tr_b8 v[86:87], %[b11] offset:9216\n" \
    "v_mfma_f32_16x16x128_f8f6f4 v[200:203], v[160:167], v[112:119], v[200:203] cbsz:1 blgp:1\n" \
    "ds_read_b128 v[48:51], %[a10] offset:4096\n" \
    "ds_read_b128 v[52:55], %[a11] offset:4096\n" \
    "v_mfma_f32_16x16x128_f8f6f4 v[204:207], v[168:175], v[112:119], v[204:207] cbsz:1 blgp:1\n" \
    "ds_read_b64_tr_b8 v[88:89], %[b12] offset:0\n" \
    "ds_read_b64_tr_b8 v[90:91], %[b12] offset:1024\n" \
    "v_mfma_f32_16x16x128_f8f6f4 v[208:211], v[144:151], v[120:127], v[208:211] cbsz:1 blgp:1\n" \
    "ds_read_b64_tr_b8 v[92:93], %[b12] offset:8192\n" \
    "ds_read_b64_tr_b8 v[94:95], %[b12] offset:9216\n" \
    "v_mfma_f32_16x16x128_f8f6f4 v[212:215], v[152:159], v[120:127], v[212:215] cbsz:1 blgp:1\n" \
    "ds_read_b128 v[56:59], %[a10] offset:6144\n" \
    "ds_read_b128 v[60:63], %[a11] offset:6144\n" \
    "v_mfma_f32_16x16x128_f8f6f4 v[216:219], v[160:167], v[120:127], v[216:219] cbsz:1 blgp:1\n" \
    "ds_read_b64_tr_b8 v[96:97], %[b13] offset:0\n" \
    "ds_read_b64_tr_b8 v[98:99], %[b13] offset:1024\n" \
    "v_mfma_f32_16x16x128_f8f6f4 v[220:223], v[168:175], v[120:127], v[220:223] cbsz:1 blgp:1\n" \
    "ds_read_b64_tr_b8 v[100:101], %[b13] offset:8192\n" \
    "ds_read_b64_tr_b8 v[102:103], %[b13] offset:9216\n" \
    "v_mfma_f32_16x16x128_f8f6f4 v[224:227], v[144:151], v[128:135], v[224:227] cbsz:1 blgp:1\n" \
    "ds_read_b128 v[64:67], %[a10] offset:8192\n" \
    "ds_read_b128 v[68:71], %[a11] offset:8192\n" \
    "v_mfma_f32_16x16x128_f8f6f4 v[228:231], v[152:159], v[128:135], v[228:231] cbsz:1 blgp:1\n" \
    "ds_write_b32 %[fr], %[tk] offset:1024\n" \
    "ds_read_b32 %[tk], %[fu] offset:12\n" \
    "v_mfma_f32_16x16x128_f8f6f4 v[232:235], v[160:167], v[128:135], v[232:235] cbsz:1 blgp:1\n" \
    "v_mfma_f32_16x16x128_f8f6f4 v[236:239], v[168:175], v[128:135], v[236:239] cbsz:1 blgp:1\n" \
    "v_mfma_f32_16x16x128_f8f6f4 v[240:243], v[144:151], v[136:143], v[240:243] cbsz:1 blgp:1\n" \
    "v_mfma_f32_16x16x128_f8f6f4 v[244:247], v[152:159], v[136:143], v[244:247] cbsz:1 blgp:1\n" \
    "v_mfma_f32_16x16x128_f8f6f4 v[248:251], v[160:167], v[136:143], v[248:251] cbsz:1 blgp:1\n" \
    "v_mfma_f32_16x16x128_f8f6f4 v[252:255], v[168:175], v[136:143], v[252:255] cbsz:1 blgp:1\n" \
    "s_cbranch_vccnz 21f\n" \
    "11:\n" \
    "s_waitcnt lgkmcnt(14)\n" \
    "v_mfma_f32_16x16x128_f8f6f4 v[176:179], v[72:79], v[32:39], v[176:179] cbsz:1 blgp:1\n" \
    "s_add_u32 %[thr], %[u], 3\n" \
    "ds_read_b128 v[104:107], %[a10] offset:36864\n" \
    "ds_read_b128 v[108:111], %[a11] offset:36864\n" \
    "s_waitcnt lgkmcnt(14)\n" \
    "v_mfma_f32_16x16x128_f8f6f4 v[180:183], v[80:87], v[32:39], v[180:183] cbsz:1 blgp:1\n" \
    "ds_read_b64_tr_b8 v[144:145], %[b10] offset:36864\n" \
    "ds_read_b64_tr_b8 v[146:147], %[b10] offset:37888\n" \
    "s_waitcnt lgkmcnt(14)\n" \
    "v_mfma_f32_16x16x128_f8f6f4 v[184:187], v[88:95], v[32:39], v[184:187] cbsz:1 blgp:1\n" \
    "ds_read_b64_tr_b8 v[148:149], %[b10] offset:45056\n" \
    "ds_read_b64_tr_b8 v[150:151], %[b10] offset:46080\n" \
    "s_waitcnt lgkmcnt(6)\n" \
    "v_mfma_f32_16x16x128_f8f6f4 v[188:191], v[96:103], v[32:39], v[188:191] cbsz:1 blgp:1\n" \
    "v_cmp_ge_u32 vcc, %[thr], %[tk]\n" \
    "ds_read_b128 v[112:115], %[a10] offset:38912\n" \
    "ds_read_b128 v[116:119], %[a11] offset:38912\n" \
    "v_mfma_f32_16x16x128_f8f6f4 v[192:195], v[72:79], v[40:47], v[192:195] cbsz:1 blgp:1\n" \
    "ds_read_b64_tr_b8 v[152:153], %[b11] offset:36864\n" \
    "ds_read_b64_tr_b8 v[154:155], %[b11] offset:37888\n" \
    "v_mfma_f32_16x16x128_f8f6f4 v[196:199], v[80:87], v[40:47], v[196:199] cbsz:1 blgp:1\n" \
    "ds_read_b64_tr_b8 v[156:157], %[b11] offset:45056\n" \
    "ds_read_b64_tr_b8 v[158:159], %[b11] offset:46080\n" \
    "v_mfma_f32_16x16x128_f8f6f4 v[200:203], v[88:95], v[40:47], v[200:203] cbsz:1 blgp:1\n" \
    "ds_read_b128 v[120:123], %[a10] offset:40960\n" \
    "ds_read_b128 v[124:127], %[a11] offset:40960\n" \
    "v_mfma_f32_16x16x128_f8f6f4 v[204:207], v[96:103], v[40:47], v[204:207] cbsz:1 blgp:1\n" \
    "ds_read_b64_tr_b8 v[160:161], %[b12] offset:36864\n" \
    "ds_read_b64_tr_b8 v[162:163], %[b12] offset:37888\n" \
    "v_mfma_f32_16x16x128_f8f6f4 v[208:211], v[72:79], v[48:55], v[208:211] cbsz:1 blgp:1\n" \
    "ds_read_b64_tr_b8 v[164:165], %[b12] offset:45056\n" \
    "ds_read_b64_tr_b8 v[166:167], %[b12] offset:46080\n" \
    "v_mfma_f32_16x16x128_f8f6f4 v[212:215], v[80:87], v[48:55], v[212:215] cbsz:1 blgp:1\n" \
    "ds_read_b128 v[128:131], %[a10] offset:43008\n" \
    "ds_read_b128 v[132:135], %[a11] offset:43008\n" \
    "v_mfma_f32_16x16x128_f8f6f4 v[216:219], v[88:95], v[48:55], v[216:219] cbsz:1 blgp:1\n" \
    "ds_read_b64_tr_b8 v[168:169], %[b13] offset:36864\n" \
    "ds_read_b64_tr_b8 v[170:171], %[b13] offset:37888\n" \
    "v_mfma_f32_16x16x128_f8f6f4 v[220:223], v[96:103], v[48:55], v[220:223] cbsz:1 blgp:1\n" \
    "ds_read_b64_tr_b8 v[172:173], %[b13] offset:45056\n" \
    "ds_read_b64_tr_b8 v[174:175], %[b13] offset:46080\n" \
    "v_mfma_f32_16x16x128_f8f6f4 v[224:227], v[72:79], v[56:63], v[224:227] cbsz:1 blgp:1\n" \
    "ds_read_b128 v[136:139], %[a10] offset:45056\n" \
    "ds_read_b128 v[140:143], %[a11] offset:45056\n" \
    "v_mfma_f32_16x16x128_f8f6f4 v[228:231], v[80:87], v[56:63], v[228:231] cbsz:1 blgp:1\n" \
    "ds_write_b32 %[fr], %[tk] offset:1536\n" \
    "ds_read_b32 %[tk], %[fu] offset:0\n" \
    "v_mfma_f32_16x16x128_f8f6f4 v[232:235], v[88:95], v[56:63], v[232:235] cbsz:1 blgp:1\n" \
    "v_mfma_f32_16x16x128_f8f6f4 v[236:239], v[96:103], v[56:63], v[236:239] cbsz:1 blgp:1\n" \
    "v_mfma_f32_16x16x128_f8f6f4 v[240:243], v[72:79], v[64:71], v[240:243] cbsz:1 blgp:1\n" \
    "v_mfma_f32_16x16x128_f8f6f4 v[244:247], v[80:87], v[64:71], v[244:247] cbsz:1 blgp:1\n" \
    "v_mfma_f32_16x16x128_f8f6f4 v[248:251], v[88:95], v[64:71], v[248:251] cbsz:1 blgp:1\n" \
    "v_mfma_f32_16x16x128_f8f6f4 v[252:255], v[96:103], v[64:71], v[252:255] cbsz:1 blgp:1\n" \
    "s_cbranch_vccnz 22f\n" \
    "12:\n" \
    "s_waitcnt lgkmcnt(14)\n" \
    "v_mfma_f32_16x16x128_f8f6f4 v[176:179], v[144:151], v[104:111], v[176:179] cbsz:1 blgp:1\n" \
    "s_add_u32 %[thr], %[u], 4\n" \
    "ds_read_b128 v[32:35], %[a00] offset:0\n" \
    "ds_read_b128 v[36:39], %[a01] offset:0\n" \
    "s_waitcnt lgkmcnt(14)\n" \
    "v_mfma_f32_16x16x128_f8f6f4 v[180:183], v[152:159], v[104:111], v[180:183] cbsz:1 blgp:1\n" \
    "s_cmp_lt_u32 %[thr], %[ns]\n" \
    "s_cselect_b32 %[thr], %[thr], 0\n" \
    "ds_read_b64_tr_b8 v[72:73], %[b00] offset:0\n" \
    "ds_read_b64_tr_b8 v[74:75], %[b00] offset:1024\n" \
    "s_waitcnt lgkmcnt(14)\n" \
    "v_mfma_f32_16x16x128_f8f6f4 v[184:187], v[160:167], v[104:111], v[184:187] cbsz:1 blgp:1\n" \
    "ds_read_b64_tr_b8 v[76:77], %[b00] offset:8192\n" \
    "ds_read_b64_tr_b8 v[78:79], %[b00] offset:9216\n" \
    "s_waitcnt lgkmcnt(6)\n" \
    "v_mfma_f32_16x16x128_f8f6f4 v[188:191], v[168:175], v[104:111], v[188:191] cbsz:1 blgp:1\n" \
    "v_cmp_ge_u32 vcc, %[thr], %[tk]\n" \
    "ds_read_b128 v[40:43], %[a00] offset:2048\n" \
    "ds_read_b128 v[44:47], %[a01] offset:2048\n" \
    "v_mfma_f32_16x16x128_f8f6f4 v[192:195], v[144:151], v[112:119], v[192:195] cbsz:1 blgp:1\n" \
    "ds_read_b64_tr_b8 v[80:81], %[b01] offset:0\n" \
    "ds_read_b64_tr_b8 v[82:83], %[b01] offset:1024\n" \
    "v_mfma_f32_16x16x128_f8f6f4 v[196:199], v[152:159], v[112:119], v[196:199] cbsz:1 blgp:1\n" \
    "ds_read_b64_tr_b8 v[84:85], %[b01] offset:8192\n" \
    "ds_read_b64_tr_b8 v[86:87], %[b01] offset:9216\n" \
    "v_mfma_f32_16x16x128_f8f6f4 v[200:203], v[160:167], v[112:119], v[200:203] cbsz:1 blgp:1\n" \
    "ds_read_b128 v[48:51], %[a00] offset:4096\n" \
    "ds_read_b128 v[52:55], %[a01] offset:4096\n" \
    "v_mfma_f32_16x16x128_f8f6f4 v[204:207], v[168:175], v[112:119], v[204:207] cbsz:1 blgp:1\n" \
    "ds_read_b64_tr_b8 v[88:89], %[b02] offset:0\n" \
    "ds_read_b64_tr_b8 v[90:91], %[b02] offset:1024\n" \
    "v_mfma_f32_16x16x128_f8f6f4 v[208:211], v[144:151], v[120:127], v[208:211] cbsz:1 blgp:1\n" \
    "ds_read_b64_tr_b8 v[92:93], %[b02] offset:8192\n" \
    "ds_read_b64_tr_b8 v[94:95], %[b02] offset:9216\n" \
    "v_mfma_f32_16x16x128_f8f6f4 v[212:215], v[152:159], v[120:127], v[212:215] cbsz:1 blgp:1\n" \
    "ds_read_b128 v[56:59], %[a00] offset:6144\n" \
    "ds_read_b128 v[60:63], %[a01] offset:6144\n" \
    "v_mfma_f32_16x16x128_f8f6f4 v[216:219], v[160:167], v[120:127], v[216:219] cbsz:1 blgp:1\n" \
    "ds_read_b64_tr_b8 v[96:97], %[b03] offset:0\n" \
    "ds_read_b64_tr_b8 v[98:99], %[b03] offset:1024\n" \
    "v_mfma_f32_16x16x128_f8f6f4 v[220:223], v[168:175], v[120:127], v[220:223] cbsz:1 blgp:1\n" \
    "ds_read_b64_tr_b8 v[100:101], %[b03] offset:8192\n" \
    "ds_read_b64_tr_b8 v[102:103], %[b03] offset:9216\n" \
    "v_mfma_f32_16x16x128_f8f6f4 v[224:227], v[144:151], v[128:135], v[224:227] cbsz:1 blgp:1\n" \
    "ds_read_b128 v[64:67], %[a00] offset:8192\n" \
    "ds_read_b128 v[68:71], %[a01] offset:8192\n" \
    "v_mfma_f32_16x16x128_f8f6f4 v[228:231], v[152:159], v[128:135], v[228:231] cbsz:1 blgp:1\n" \
    "ds_write_b32 %[fr], %[tk] offset:0\n" \
    "ds_read_b32 %[tk], %[fu] offset:4\n" \
    "v_mfma_f32_16x16x128_f8f6f4 v[232:235], v[160:167], v[128:135], v[232:235] cbsz:1 blgp:1\n" \
    "v_mfma_f32_16x16x128_f8f6f4 v[236:239], v[168:175], v[128:135], v[236:239] cbsz:1 blgp:1\n" \
    "v_mfma_f32_16x16x128_f8f6f4 v[240:243], v[144:151], v[136:143], v[240:243] cbsz:1 blgp:1\n" \
    "v_mfma_f32_16x16x128_f8f6f4 v[244:247], v[152:159], v[136:143], v[244:247] cbsz:1 blgp:1\n" \
    "s_add_u32 %[u], %[u], 4\n" \
    "v_mfma_f32_16x16x128_f8f6f4 v[248:251], v[160:167], v[136:143], v[248:251] cbsz:1 blgp:1\n" \
    "s_cmp_lt_u32 %[u], %[uend]\n" \
    "v_mfma_f32_16x16x128_f8f6f4 v[252:255], v[168:175], v[136:143], v[252:255] cbsz:1 blgp:1\n" \
    "s_cbranch_vccnz 23f\n" \
    "13:\n" \
    "s_cbranch_scc1 0b\n" \
    "s_branch 3f\n" \
    "20:\n" \
    "ds_read_b32 %[vt], %[fu] offset:4\n" \
    "s_waitcnt lgkmcnt(0)\n" \
    "v_cmp_ge_u32 vcc, %[thr], %[vt]\n" \
    "s_cbranch_vccnz 20b\n" \
    "ds_read_b128 v[104:107], %[a00] offset:36864\n" \
    "ds_read_b128 v[108:111], %[a01] offset:36864\n" \
    "ds_read_b64_tr_b8 v[144:145], %[b00] offset:36864\n" \
    "ds_read_b64_tr_b8 v[146:147], %[b00] offset:37888\n" \
    "ds_read_b64_tr_b8 v[148:149], %[b00] offset:45056\n" \
    "ds_read_b64_tr_b8 v[150:151], %[b00] offset:46080\n" \
    "ds_read_b128 v[112:115], %[a00] offset:38912\n" \
    "ds_read_b128 v[116:119], %[a01] offset:38912\n" \
    "ds_read_b64_tr_b8 v[152:153], %[b01] offset:36864\n" \
    "ds_read_b64_tr_b8 v[154:155], %[b01] offset:37888\n" \
    "ds_read_b64_tr_b8 v[156:157], %[b01] offset:45056\n" \
    "ds_read_b64_tr_b8 v[158:159], %[b01] offset:46080\n" \
    "ds_read_b128 v[120:123], %[a00] offset:40960\n" \
    "ds_read_b128 v[124:127], %[a01] offset:40960\n" \
    "ds_read_b64_tr_b8 v[160:161], %[b02] offset:36864\n" \
    "ds_read_b64_tr_b8 v[162:163], %[b02] offset:37888\n" \
    "ds_read_b64_tr_b8 v[164:165], %[b02] offset:45056\n" \
    "ds_read_b64_tr_b8 v[166:167], %[b02] offset:46080\n" \
    "ds_read_b128 v[128:131], %[a00] offset:43008\n" \
    "ds_read_b128 v[132:135], %[a01] offset:43008\n" \
    "ds_read_b64_tr_b8 v[168:169], %[b03] offset:36864\n" \
    "ds_read_b64_tr_b8 v[170:171], %[b03] offset:37888\n" \
    "ds_read_b64_tr_b8 v[172:173], %[b03] offset:45056\n" \
    "ds_read_b64_tr_b8 v[174:175], %[b03] offset:46080\n" \
    "ds_read_b128 v[136:139], %[a00] offset:45056\n" \
    "ds_read_b128 v[140:143], %[a01] offset:45056\n" \
    "ds_write_b32 %[fr], %[vt] offset:512\n" \
    "ds_read_b32 %[tk], %[fu] offset:8\n" \
    "s_branch 10b\n" \
    "21:\n" \
    "ds_read_b32 %[vt], %[fu] offset:8\n" \
    "s_waitcnt lgkmcnt(0)\n" \
    "v_cmp_ge_u32 vcc, %[thr], %[vt]\n" \
    "s_cbranch_vccnz 21b\n" \
    "ds_read_b128 v[32:35], %[a10] offset:0\n" \
    "ds_read_b128 v[36:39], %[a11] offset:0\n" \
    "ds_read_b64_tr_b8 v[72:73], %[b10] offset:0\n" \
    "ds_read_b64_tr_b8 v[74:75], %[b10] offset:1024\n" \
    "ds_read_b64_tr_b8 v[76:77], %[b10] offset:8192\n" \
    "ds_read_b64_tr_b8 v[78:79], %[b10] offset:9216\n" \
    "ds_read_b128 v[40:43], %[a10] offset:2048\n" \
    "ds_read_b128 v[44:47], %[a11] offset:2048\n" \
    "ds_read_b64_tr_b8 v[80:81], %[b11] offset:0\n" \
    "ds_read_b64_tr_b8 v[82:83], %[b11] offset:1024\n" \
    "ds_read_b64_tr_b8 v[84:85], %[b11] offset:8192\n" \
    "ds_read_b64_tr_b8 v[86:87], %[b11] offset:9216\n" \
    "ds_read_b128 v[48:51], %[a10] offset:4096\n" \
    "ds_read_b128 v[52:55], %[a11] offset:4096\n" \
    "ds_read_b64_tr_b8 v[88:89], %[b12] offset:0\n" \
    "ds_read_b64_tr_b8 v[90:91], %[b12] offset:1024\n" \
    "ds_read_b64_tr_b8 v[92:93], %[b12] offset:8192\n" \
    "ds_read_b64_tr_b8 v[94:95], %[b12] offset:9216\n" \
    "ds_read_b128 v[56:59], %[a10] offset:6144\n" \
    "ds_read_b128 v[60:63], %[a11] offset:6144\n" \
    "ds_read_b64_tr_b8 v[96:97], %[b13] offset:0\n" \
    "ds_read_b64_tr_b8 v[98:99], %[b13] offset:1024\n" \
    "ds_read_b64_tr_b8 v[100:101], %[b13] offset:8192\n" \
    "ds_read_b64_tr_b8 v[102:103], %[b13] offset:9216\n" \
    "ds_read_b128 v[64:67], %[a10] offset:8192\n" \
    "ds_read_b128 v[68:71], %[a11] offset:8192\n" \
    "ds_write_b32 %[fr], %[vt] offset:1024\n" \
    "ds_read_b32 %[tk], %[fu] offset:12\n" \
    "s_branch 11b\n" \
    "22:\n" \
    "ds_read_b32 %[vt], %[fu] offset:12\n" \
    "s_waitcnt lgkmcnt(0)\n" \
    "v_cmp_ge_u32 vcc, %[thr], %[vt]\n" \
    "s_cbranch_vccnz 22b\n" \
    "ds_read_b128 v[104:107], %[a10] offset:36864\n" \
    "ds_read_b128 v[108:111], %[a11] offset:36864\n" \
    "ds_read_b64_tr_b8 v[144:145], %[b10] offset:36864\n" \
    "ds_read_b64_tr_b8 v[146:147], %[b10] offset:37888\n" \
    "ds_read_b64_tr_b8 v[148:149], %[b10] offset:45056\n" \
    "ds_read_b64_tr_b8 v[150:151], %[b10] offset:46080\n" \
    "ds_read_b128 v[112:115], %[a10] offset:38912\n" \
    "ds_read_b128 v[116:119], %[a11] offset:38912\n" \
    "ds_read_b64_tr_b8 v[152:153], %[b11] offset:36864\n" \
    "ds_read_b64_tr_b8 v[154:155], %[b11] offset:37888\n" \
    "ds_read_b64_tr_b8 v[156:157], %[b11] offset:45056\n" \
    "ds_read_b64_tr_b8 v[158:159], %[b11] offset:46080\n" \
    "ds_read_b128 v[120:123], %[a10] offset:40960\n" \
    "ds_read_b128 v[124:127], %[a11] offset:40960\n" \
    "ds_read_b64_tr_b8 v[160:161], %[b12] offset:36864\n" \
    "ds_read_b64_tr_b8 v[162:163], %[b12] offset:37888\n" \
    "ds_read_b64_tr_b8 v[164:165], %[b12] offset:45056\n" \
    "ds_read_b64_tr_b8 v[166:167], %[b12] offset:46080\n" \
    "ds_read_b128 v[128:131], %[a10] offset:43008\n" \
    "ds_read_b128 v[132:135], %[a11] offset:43008\n" \
    "ds_read_b64_tr_b8 v[168:169], %[b13] offset:36864\n" \
    "ds_read_b64_tr_b8 v[170:171], %[b13] offset:37888\n" \
    "ds_read_b64_tr_b8 v[172:173], %[b13] offset:45056\n" \
    "ds_read_b64_tr_b8 v[174:175], %[b13] offset:46080\n" \
    "ds_read_b128 v[136:139], %[a10] offset:45056\n" \
    "ds_read_b128 v[140:143], %[a11] offset:45056\n" \
    "ds_write_b32 %[fr], %[vt] offset:1536\n" \
    "ds_read_b32 %[tk], %[fu] offset:0\n" \
    "s_branch 12b\n" \
    "23:\n" \
    "ds_read_b32 %[vt], %[fu] offset:0\n" \
    "s_waitcnt lgkmcnt(0)\n" \
    "v_cmp_ge_u32 vcc, %[thr], %[vt]\n" \
    "s_cbranch_vccnz 23b\n" \
    "ds_read_b128 v[32:35], %[a00] offset:0\n" \
    "ds_read_b128 v[36:39], %[a01] offset:0\n" \
    "ds_read_b64_tr_b8 v[72:73], %[b00] offset:0\n" \
    "ds_read_b64_tr_b8 v[74:75], %[b00] offset:1024\n" \
    "ds_read_b64_tr_b8 v[76:77], %[b00] offset:8192\n" \
    "ds_read_b64_tr_b8 v[78:79], %[b00] offset:9216\n" \
    "ds_read_b128 v[40:43], %[a00] offset:2048\n" \
    "ds_read_b128 v[44:47], %[a01] offset:2048\n" \
    "ds_read_b64_tr_b8 v[80:81], %[b01] offset:0\n" \
    "ds_read_b64_tr_b8 v[82:83], %[b01] offset:1024\n" \
    "ds_read_b64_tr_b8 v[84:85], %[b01] offset:8192\n" \
    "ds_read_b64_tr_b8 v[86:87], %[b01] offset:9216\n" \
    "ds_read_b128 v[48:51], %[a00] offset:4096\n" \
    "ds_read_b128 v[52:55], %[a01] offset:4096\n" \
    "ds_read_b64_tr_b8 v[88:89], %[b02] offset:0\n" \
    "ds_read_b64_tr_b8 v[90:91], %[b02] offset:1024\n" \
    "ds_read_b64_tr_b8 v[92:93], %[b02] offset:8192\n" \
    "ds_read_b64_tr_b8 v[94:95], %[b02] offset:9216\n" \
    "ds_read_b128 v[56:59], %[a00] offset:6144\n" \
    "ds_read_b128 v[60:63], %[a01] offset:6144\n" \
    "ds_read_b64_tr_b8 v[96:97], %[b03] offset:0\n" \
    "ds_read_b64_tr_b8 v[98:99], %[b03] offset:1024\n" \
    "ds_read_b64_tr_b8 v[100:101], %[b03] offset:8192\n" \
    "ds_read_b64_tr_b8 v[102:103], %[b03] offset:9216\n" \
    "ds_read_b128 v[64:67], %[a00] offset:8192\n" \
    "ds_read_b128 v[68:71], %[a01] offset:8192\n" \
    "ds_write_b32 %[fr], %[vt] offset:0\n" \
    "ds_read_b32 %[tk], %[fu] offset:4\n" \
    "s_branch 13b\n" \
    "3:\n" \
    "s_waitcnt lgkmcnt(0)\n" \
    "s_nop 7\n" \
    "s_nop 7\n" \
    ""
#define AG_ASM_LOOP4 \
    "s_nop 1\n" \
    "0:\n" \
    "s_waitcnt lgkmcnt(14)\n" \
    "v_mfma_f32_16x16x128_f8f6f4 v[176:179], v[72:79], v[32:39], v[176:179] cbsz:1 blgp:1\n" \
    "s_add_u32 %[thr], %[u], 1\n" \
    "ds_read_b128 v[104:107], %[a00] offset:36864\n" \
    "ds_read_b128 v[108:111], %[a01] offset:36864\n" \
    "s_waitcnt lgkmcnt(14)\n" \
    "v_mfma_f32_16x16x128_f8f6f4 v[180:183], v[80:87], v[32:39], v[180:183] cbsz:1 blgp:1\n" \
    "ds_read_b64_tr_b8 v[144:145], %[b00] offset:36864\n" \
    "ds_read_b64_tr_b8 v[146:147], %[b00] offset:37888\n" \
    "s_waitcnt lgkmcnt(12)\n" \
    "v_mfma_f32_16x16x128_f8f6f4 v[184:187], v[88:95], v[32:39], v[184:187] cbsz:1 blgp:1\n" \
    "ds_read_b64_tr_b8 v[148:149], %[b00] offset:45056\n" \
    "ds_read_b64_tr_b8 v[150:151], %[b00] offset:46080\n" \
    "s_waitcnt lgkmcnt(6)\n" \
    "v_mfma_f32_16x16x128_f8f6f4 v[188:191], v[96:103], v[32:39], v[188:191] cbsz:1 blgp:1\n" \
    "v_cmp_ge_u32 vcc, %[thr], %[tk]\n" \
    "ds_read_b128 v[112:115], %[a00] offset:38912\n" \
    "ds_read_b128 v[116:119], %[a01] offset:38912\n" \
    "v_mfma_f32_16x16x128_f8f6f4 v[192:195], v[72:79], v[40:47], v[192:195] cbsz:1 blgp:1\n" \
    "ds_read_b64_tr_b8 v[152:153], %[b01] offset:36864\n" \
    "ds_read_b64_tr_b8 v[154:155], %[b01] offset:37888\n" \
    "v_mfma_f32_16x16x128_f8f6f4 v[196:199], v[80:87], v[40:47], v[196:199] cbsz:1 blgp:1\n" \
    "ds_read_b64_tr_b8 v[156:157], %[b01] offset:45056\n" \
    "ds_read_b64_tr_b8 v[158:159], %[b01] offset:46080\n" \
    "v_mfma_f32_16x16x128_f8f6f4 v[200:203], v[88:95], v[40:47], v[200:203] cbsz:1 blgp:1\n" \
    "ds_read_b128 v[120:123], %[a00] offset:40960\n" \
    "ds_read_b128 v[124:127], %[a01] offset:40960\n" \
    "v_mfma_f32_16x16x128_f8f6f4 v[204:207], v[96:103], v[40:47], v[204:207] cbsz:1 blgp:1\n" \
    "ds_read_b64_tr_b8 v[160:161], %[b02] offset:36864\n" \
    "ds_read_b64_tr_b8 v[162:163], %[b02] offset:37888\n" \
    "v_mfma_f32_16x16x128_f8f6f4 v[208:211], v[72:79], v[48:55], v[208:211] cbsz:1 blgp:1\n" \
    "ds_read_b64_tr_b8 v[164:165], %[b02] offset:45056\n" \
    "ds_read_b64_tr_b8 v[166:167], %[b02] offset:46080\n" \
    "v_mfma_f32_16x16x128_f8f6f4 v[212:215], v[80:87], v[48:55], v[212:215] cbsz:1 blgp:1\n" \
    "ds_read_b128 v[128:131], %[a00] offset:43008\n" \
    "ds_read_b128 v[132:135], %[a01] offset:43008\n" \
    "v_mfma_f32_16x16x128_f8f6f4 v[216:219], v[88:95], v[48:55], v[216:219] cbsz:1 blgp:1\n" \
    "ds_read_b64_tr_b8 v[168:169], %[b03] offset:36864\n" \
    "ds_read_b64_tr_b8 v[170:171], %[b03] offset:37888\n" \
    "v_mfma_f32_16x16x128_f8f6f4 v[220:223], v[96:103], v[48:55], v[220:223] cbsz:1 blgp:1\n" \
    "ds_read_b64_tr_b8 v[172:173], %[b03] offset:45056\n" \
    "ds_read_b64_tr_b8 v[174:175], %[b03] offset:46080\n" \
    "v_mfma_f32_16x16x128_f8f6f4 v[224:227], v[72:79], v[56:63], v[224:227] cbsz:1 blgp:1\n" \
    "ds_write_b32 %[fr], %[tk] offset:512\n" \
    "ds_read_b32 %[tk], %[fu] offset:8\n" \
    "v_mfma_f32_16x16x128_f8f6f4 v[228:231], v[80:87], v[56:63], v[228:231] cbsz:1 blgp:1\n" \
    "v_mfma_f32_16x16x128_f8f6f4 v[232:235], v[88:95], v[56:63], v[232:235] cbsz:1 blgp:1\n" \
    "v_mfma_f32_16x16x128_f8f6f4 v[236:239], v[96:103], v[56:63], v[236:239] cbsz:1 blgp:1\n" \
    "s_cbranch_vccnz 20f\n" \
    "10:\n" \
    "s_waitcnt lgkmcnt(14)\n" \
    "v_mfma_f32_16x16x128_f8f6f4 v[176:179], v[144:151], v[104:111], v[176:179] cbsz:1 blgp:1\n" \
    "s_add_u32 %[thr], %[u], 2\n" \
    "ds_read_b128 v[32:35], %[a10] offset:0\n" \
    "ds_read_b128 v[36:39], %[a11] offset:0\n" \
    "s_waitcnt lgkmcnt(14)\n" \
    "v_mfma_f32_16x16x128_f8f6f4 v[180:183], v[152:159], v[104:111], v[180:183] cbsz:1 blgp:1\n" \
    "ds_read_b64_tr_b8 v[72:73], %[b10] offset:0\n" \
    "ds_read_b64_tr_b8 v[74:75], %[b10] offset:1024\n" \
    "s_waitcnt lgkmcnt(12)\n" \
    "v_mfma_f32_16x16x128_f8f6f4 v[184:187], v[160:167], v[104:111], v[184:187] cbsz:1 blgp:1\n" \
    "ds_read_b64_tr_b8 v[76:77], %[b10] offset:8192\n" \
    "ds_read_b64_tr_b8 v[78:79], %[b10] offset:9216\n" \
    "s_waitcnt lgkmcnt(6)\n" \
    "v_mfma_f32_16x16x128_f8f6f4 v[188:191], v[168:175], v[104:111], v[188:191] cbsz:1 blgp:1\n" \
    "v_cmp_ge_u32 vcc, %[thr], %[tk]\n" \
    "ds_read_b128 v[40:43], %[a10] offset:2048\n" \
    "ds_read_b128 v[44:47], %[a11] offset:2048\n" \
    "v_mfma_f32_16x16x128_f8f6f4 v[192:195], v[144:151], v[112:119], v[192:195] cbsz:1 blgp:1\n" \
    "ds_read_b64_tr_b8 v[80:81], %[b11] offset:0\n" \
    "ds_read_b64_tr_b8 v[82:83], %[b11] offset:1024\n" \
    "v_mfma_f32_16x16x128_f8f6f4 v[196:199], v[152:159], v[112:119], v[196:199] cbsz:1 blgp:1\n" \
    "ds_read_b64_tr_b8 v[84:85], %[b11] offset:8192\n" \
    "ds_read_b64_tr_b8 v[86:87], %[b11] offset:9216\n" \
    "v_mfma_f32_16x16x128_f8f6f4 v[200:203], v[160:167], v[112:119], v[200:203] cbsz:1 blgp:1\n" \
    "ds_read_b128 v[48:51], %[a10] offset:4096\n" \
    "ds_read_b128 v[52:55], %[a11] offset:4096\n" \
    "v_mfma_f32_16x16x128_f8f6f4 v[204:207], v[168:175], v[112:119], v[204:207] cbsz:1 blgp:1\n" \
    "ds_read_b64_tr_b8 v[88:89], %[b12] offset:0\n" \
    "ds_read_b64_tr_b8 v[90:91], %[b12] offset:1024\n" \
    "v_mfma_f32_16x16x128_f8f6f4 v[208:211], v[144:151], v[120:127], v[208:211] cbsz:1 blgp:1\n" \
    "ds_read_b64_tr_b8 v[92:93], %[b12] offset:8192\n" \
    "ds_read_b64_tr_b8 v[94:95], %[b12] offset:9216\n" \
    "v_mfma_f32_16x16x128_f8f6f4 v[212:215], v[152:159], v[120:127], v[212:215] cbsz:1 blgp:1\n" \
    "ds_read_b128 v[56:59], %[a10] offset:6144\n" \
    "ds_read_b128 v[60:63], %[a11] offset:6144\n" \
    "v_mfma_f32_16x16x128_f8f6f4 v[216:219], v[160:167], v[120:127], v[216:219] cbsz:1 blgp:1\n" \
    "ds_read_b64_tr_b8 v[96:97], %[b13] offset:0\n" \
    "ds_read_b64_tr_b8 v[98:99], %[b13] offset:1024\n" \
    "v_mfma_f32_16x16x128_f8f6f4 v[220:223], v[168:175], v[120:127], v[220:223] cbsz:1 blgp:1\n" \
    "ds_read_b64_tr_b8 v[100:101], %[b13] offset:8192\n" \
    "ds_read_b64_tr_b8 v[102:103], %[b13] offset:9216\n" \
    "v_mfma_f32_16x16x128_f8f6f4 v[224:227], v[144:151], v[128:135], v[224:227] cbsz:1 blgp:1\n" \
    "ds_write_b32 %[fr], %[tk] offset:1024\n" \
    "ds_read_b32 %[tk], %[fu] offset:12\n" \
    "v_mfma_f32_16x16x128_f8f6f4 v[228:231], v[152:159], v[128:135], v[228:231] cbsz:1 blgp:1\n" \
    "v_mfma_f32_16x16x128_f8f6f4 v[232:235], v[160:167], v[128:135], v[232:235] cbsz:1 blgp:1\n" \
    "v_mfma_f32_16x16x128_f8f6f4 v[236:239], v[168:175], v[128:135], v[236:239] cbsz:1 blgp:1\n" \
    "s_cbranch_vccnz 21f\n" \
    "11:\n" \
    "s_waitcnt lgkmcnt(14)\n" \
    "v_mfma_f32_16x16x128_f8f6f4 v[176:179], v[72:79], v[32:39], v[176:179] cbsz:1 blgp:1\n" \
    "s_add_u32 %[thr], %[u], 3\n" \
    "ds_read_b128 v[104:107], %[a10] offset:36864\n" \
    "ds_read_b128 v[108:111], %[a11] offset:36864\n" \
    "s_waitcnt lgkmcnt(14)\n" \
    "v_mfma_f32_16x16x128_f8f6f4 v[180:183], v[80:87], v[32:39], v[180:183] cbsz:1 blgp:1\n" \
    "ds_read_b64_tr_b8 v[144:145], %[b10] offset:36864\n" \
    "ds_read_b64_tr_b8 v[146:147], %[b10] offset:37888\n" \
    "s_waitcnt lgkmcnt(12)\n" \
    "v_mfma_f32_16x16x128_f8f6f4 v[184:187], v[88:95], v[32:39], v[184:187] cbsz:1 blgp:1\n" \
    "ds_read_b64_tr_b8 v[148:149], %[b10] offset:45056\n" \
    "ds_read_b64_tr_b8 v[150:151], %[b10] offset:46080\n" \
    "s_waitcnt lgkmcnt(6)\n" \
    "v_mfma_f32_16x16x128_f8f6f4 v[188:191], v[96:103], v[32:39], v[188:191] cbsz:1 blgp:1\n" \
    "v_cmp_ge_u32 vcc, %[thr], %[tk]\n" \
    "ds_read_b128 v[112:115], %[a10] offset:38912\n" \
    "ds_read_b128 v[116:119], %[a11] offset:38912\n" \
    "v_mfma_f32_16x16x128_f8f6f4 v[192:195], v[72:79], v[40:47], v[192:195] cbsz:1 blgp:1\n" \
    "ds_read_b64_tr_b8 v[152:153], %[b11] offset:36864\n" \
    "ds_read_b64_tr_b8 v[154:155], %[b11] offset:37888\n" \
    "v_mfma_f32_16x16x128_f8f6f4 v[196:199], v[80:87], v[40:47], v[196:199] cbsz:1 blgp:1\n" \
    "ds_read_b64_tr_b8 v[156:157], %[b11] offset:45056\n" \
    "ds_read_b64_tr_b8 v[158:159], %[b11] offset:46080\n" \
    "v_mfma_f32_16x16x128_f8f6f4 v[200:203], v[88:95], v[40:47], v[200:203] cbsz:1 blgp:1\n" \
    "ds_read_b128 v[120:123], %[a10] offset:40960\n" \
    "ds_read_b128 v[124:127], %[a11] offset:40960\n" \
    "v_mfma_f32_16x16x128_f8f6f4 v[204:207], v[96:103], v[40:47], v[204:207] cbsz:1 blgp:1\n" \
    "ds_read_b64_tr_b8 v[160:161], %[b12] offset:36864\n" \
    "ds_read_b64_tr_b8 v[162:163], %[b12] offset:37888\n" \
    "v_mfma_f32_16x16x128_f8f6f4 v[208:211], v[72:79], v[48:55], v[208:211] cbsz:1 blgp:1\n" \
    "ds_read_b64_tr_b8 v[164:165], %[b12] offset:45056\n" \
    "ds_read_b64_tr_b8 v[166:167], %[b12] offset:46080\n" \
    "v_mfma_f32_16x16x128_f8f6f4 v[212:215], v[80:87], v[48:55], v[212:215] cbsz:1 blgp:1\n" \
    "ds_read_b128 v[128:131], %[a10] offset:43008\n" \
    "ds_read_b128 v[132:135], %[a11] offset:43008\n" \
    "v_mfma_f32_16x16x128_f8f6f4 v[216:219], v[88:95], v[48:55], v[216:219] cbsz:1 blgp:1\n" \
    "ds_read_b64_tr_b8 v[168:169], %[b13] offset:36864\n" \
    "ds_read_b64_tr_b8 v[170:171], %[b13] offset:37888\n" \
    "v_mfma_f32_16x16x128_f8f6f4 v[220:223], v[96:103], v[48:55], v[220:223] cbsz:1 blgp:1\n" \
    "ds_read_b64_tr_b8 v[172:173], %[b13] offset:45056\n" \
    "ds_read_b64_tr_b8 v[174:175], %[b13] offset:46080\n" \
    "v_mfma_f32_16x16x128_f8f6f4 v[224:227], v[72:79], v[56:63], v[224:227] cbsz:1 blgp:1\n" \
    "ds_write_b32 %[fr], %[tk] offset:1536\n" \
    "ds_read_b32 %[tk], %[fu] offset:0\n" \
    "v_mfma_f32_16x16x128_f8f6f4 v[228:231], v[80:87], v[56:63], v[228:231] cbsz:1 blgp:1\n" \
    "v_mfma_f32_16x16x128_f8f6f4 v[232:235], v[88:95], v[56:63], v[232:235] cbsz:1 blgp:1\n" \
    "v_mfma_f32_16x16x128_f8f6f4 v[236:239], v[96:103], v[56:63], v[236:239] cbsz:1 blgp:1\n" \
    "s_cbranch_vccnz 22f\n" \
    "12:\n" \
    "s_waitcnt lgkmcnt(14)\n" \
    "v_mfma_f32_16x16x128_f8f6f4 v[176:179], v[144:151], v[104:111], v[176:179] cbsz:1 blgp:1\n" \
    "s_add_u32 %[thr], %[u], 4\n" \
    "ds_read_b128 v[32:35], %[a00] offset:0\n" \
    "ds_read_b128 v[36:39], %[a01] offset:0\n" \
    "s_waitcnt lgkmcnt(14)\n" \
    "v_mfma_f32_16x16x128_f8f6f4 v[180:183], v[152:159], v[104:111], v[180:183] cbsz:1 blgp:1\n" \
    "s_cmp_lt_u32 %[thr], %[ns]\n" \
    "s_cselect_b32 %[thr], %[thr], 0\n" \
    "ds_read_b64_tr_b8 v[72:73], %[b00] offset:0\n" \
    "ds_read_b64_tr_b8 v[74:75], %[b00] offset:1024\n" \
    "s_waitcnt lgkmcnt(12)\n" \
    "v_mfma_f32_16x16x128_f8f6f4 v[184:187], v[160:167], v[104:111], v[184:187] cbsz:1 blgp:1\n" \
    "ds_read_b64_tr_b8 v[76:77], %[b00] offset:8192\n" \
    "ds_read_b64_tr_b8 v[78:79], %[b00] offset:9216\n" \
    "s_waitcnt lgkmcnt(6)\n" \
    "v_mfma_f32_16x16x128_f8f6f4 v[188:191], v[168:175], v[104:111], v[188:191] cbsz:1 blgp:1\n" \
    "v_cmp_ge_u32 vcc, %[thr], %[tk]\n" \
    "ds_read_b128 v[40:43], %[a00] offset:2048\n" \
    "ds_read_b128 v[44:47], %[a01] offset:2048\n" \
    "v_mfma_f32_16x16x128_f8f6f4 v[192:195], v[144:151], v[112:119], v[192:195] cbsz:1 blgp:1\n" \
    "ds_read_b64_tr_b8 v[80:81], %[b01] offset:0\n" \
    "ds_read_b64_tr_b8 v[82:83], %[b01] offset:1024\n" \
    "v_mfma_f32_16x16x128_f8f6f4 v[196:199], v[152:159], v[112:119], v[196:199] cbsz:1 blgp:1\n" \
    "ds_read_b64_tr_b8 v[84:85], %[b01] offset:8192\n" \
    "ds_read_b64_tr_b8 v[86:87], %[b01] offset:9216\n" \
    "v_mfma_f32_16x16x128_f8f6f4 v[200:203], v[160:167], v[112:119], v[200:203] cbsz:1 blgp:1\n" \
    "ds_read_b128 v[48:51], %[a00] offset:4096\n" \
    "ds_read_b128 v[52:55], %[a01] offset:4096\n" \
    "v_mfma_f32_16x16x128_f8f6f4 v[204:207], v[168:175], v[112:119], v[204:207] cbsz:1 blgp:1\n" \
    "ds_read_b64_tr_b8 v[88:89], %[b02] offset:0\n" \
    "ds_read_b64_tr_b8 v[90:91], %[b02] offset:1024\n" \
    "v_mfma_f32_16x16x128_f8f6f4 v[208:211], v[144:151], v[120:127], v[208:211] cbsz:1 blgp:1\n" \
    "ds_read_b64_tr_b8 v[92:93], %[b02] offset:8192\n" \
    "ds_read_b64_tr_b8 v[94:95], %[b02] offset:9216\n" \
    "v_mfma_f32_16x16x128_f8f6f4 v[212:215], v[152:159], v[120:127], v[212:215] cbsz:1 blgp:1\n" \
    "ds_read_b128 v[56:59], %[a00] offset:6144\n" \
    "ds_read_b128 v[60:63], %[a01] offset:6144\n" \
    "v_mfma_f32_16x16x128_f8f6f4 v[216:219], v[160:167], v[120:127], v[216:219] cbsz:1 blgp:1\n" \
    "ds_read_b64_tr_b8 v[96:97], %[b03] offset:0\n" \
    "ds_read_b64_tr_b8 v[98:99], %[b03] offset:1024\n" \
    "v_mfma_f32_16x16x128_f8f6f4 v[220:223], v[168:175], v[120:127], v[220:223] cbsz:1 blgp:1\n" \
    "ds_read_b64_tr_b8 v[100:101], %[b03] offset:8192\n" \
    "ds_read_b64_tr_b8 v[102:103], %[b03] offset:9216\n" \
    "v_mfma_f32_16x16x128_f8f6f4 v[224:227], v[144:151], v[128:135], v[224:227] cbsz:1 blgp:1\n" \
    "ds_write_b32 %[fr], %[tk] offset:0\n" \
    "ds_read_b32 %[tk], %[fu] offset:4\n" \
    "v_mfma_f32_16x16x128_f8f6f4 v[228:231], v[152:159], v[128:135], v[228:231] cbsz:1 blgp:1\n" \
    "s_add_u32 %[u], %[u], 4\n" \
    "v_mfma_f32_16x16x128_f8f6f4 v[232:235], v[160:167], v[128:135], v[232:235] cbsz:1 blgp:1\n" \
    "s_cmp_lt_u32 %[u], %[uend]\n" \
    "v_mfma_f32_16x16x128_f8f6f4 v[236:239], v[168:175], v[128:135], v[236:239] cbsz:1 blgp:1\n" \
    "s_cbranch_vccnz 23f\n" \
    "13:\n" \
    "s_cbranch_scc1 0b\n" \
    "s_branch 3f\n" \
    "20:\n" \
    "ds_read_b32 %[vt], %[fu] offset:4\n" \
    "s_waitcnt lgkmcnt(0)\n" \
    "v_cmp_ge_u32 vcc, %[thr], %[vt]\n" \
    "s_cbranch_vccnz 20b\n" \
    "ds_read_b128 v[104:107], %[a00] offset:36864\n" \
    "ds_read_b128 v[108:111], %[a01] offset:36864\n" \
    "ds_read_b64_tr_b8 v[144:145], %[b00] offset:36864\n" \
    "ds_read_b64_tr_b8 v[146:147], %[b00] offset:37888\n" \
    "ds_read_b64_tr_b8 v[148:149], %[b00] offset:45056\n" \
    "ds_read_b64_tr_b8 v[150:151], %[b00] offset:46080\n" \
    "ds_read_b128 v[112:115], %[a00] offset:38912\n" \
    "ds_read_b128 v[116:119], %[a01] offset:38912\n" \
    "ds_read_b64_tr_b8 v[152:153], %[b01] offset:36864\n" \
    "ds_read_b64_tr_b8 v[154:155], %[b01] offset:37888\n" \
    "ds_read_b64_tr_b8 v[156:157], %[b01] offset:45056\n" \
    "ds_read_b64_tr_b8 v[158:159], %[b01] offset:46080\n" \
    "ds_read_b128 v[120:123], %[a00] offset:40960\n" \
    "ds_read_b128 v[124:127], %[a01] offset:40960\n" \
    "ds_read_b64_tr_b8 v[160:161], %[b02] offset:36864\n" \
    "ds_read_b64_tr_b8 v[162:163], %[b02] offset:37888\n" \
    "ds_read_b64_tr_b8 v[164:165], %[b02] offset:45056\n" \
    "ds_read_b64_tr_b8 v[166:167], %[b02] offset:46080\n" \
    "ds_read_b128 v[128:131], %[a00] offset:43008\n" \
    "ds_read_b128 v[132:135], %[a01] offset:43008\n" \
    "ds_read_b64_tr_b8 v[168:169], %[b03] offset:36864\n" \
    "ds_read_b64_tr_b8 v[170:171], %[b03] offset:37888\n" \
    "ds_read_b64_tr_b8 v[172:173], %[b03] offset:45056\n" \
    "ds_read_b64_tr_b8 v[174:175], %[b03] offset:46080\n" \
    "ds_write_b32 %[fr], %[vt] offset:512\n" \
    "ds_read_b32 %[tk], %[fu] offset:8\n" \
    "s_branch 10b\n" \
    "21:\n" \
    "ds_read_b32 %[vt], %[fu] offset:8\n" \
    "s_waitcnt lgkmcnt(0)\n" \
    "v_cmp_ge_u32 vcc, %[thr], %[vt]\n" \
    "s_cbranch_vccnz 21b\n" \
    "ds_read_b128 v[32:35], %[a10] offset:0\n" \
    "ds_read_b128 v[36:39], %[a11] offset:0\n" \
    "ds_read_b64_tr_b8 v[72:73], %[b10] offset:0\n" \
    "ds_read_b64_tr_b8 v[74:75], %[b10] offset:1024\n" \
    "ds_read_b64_tr_b8 v[76:77], %[b10] offset:8192\n" \
    "ds_read_b64_tr_b8 v[78:79], %[b10] offset:9216\n" \
    "ds_read_b128 v[40:43], %[a10] offset:2048\n" \
    "ds_read_b128 v[44:47], %[a11] offset:2048\n" \
    "ds_read_b64_tr_b8 v[80:81], %[b11] offset:0\n" \
    "ds_read_b64_tr_b8 v[82:83], %[b11] offset:1024\n" \
    "ds_read_b64_tr_b8 v[84:85], %[b11] offset:8192\n" \
    "ds_read_b64_tr_b8 v[86:87], %[b11] offset:9216\n" \
    "ds_read_b128 v[48:51], %[a10] offset:4096\n" \
    "ds_read_b128 v[52:55], %[a11] offset:4096\n" \
    "ds_read_b64_tr_b8 v[88:89], %[b12] offset:0\n" \
    "ds_read_b64_tr_b8 v[90:91], %[b12] offset:1024\n" \
    "ds_read_b64_tr_b8 v[92:93], %[b12] offset:8192\n" \
    "ds_read_b64_tr_b8 v[94:95], %[b12] offset:9216\n" \
    "ds_read_b128 v[56:59], %[a10] offset:6144\n" \
    "ds_read_b128 v[60:63], %[a11] offset:6144\n" \
    "ds_read_b64_tr_b8 v[96:97], %[b13] offset:0\n" \
    "ds_read_b64_tr_b8 v[98:99], %[b13] offset:1024\n" \
    "ds_read_b64_tr_b8 v[100:101], %[b13] offset:8192\n" \
    "ds_read_b64_tr_b8 v[102:103], %[b13] offset:9216\n" \
    "ds_write_b32 %[fr], %[vt] offset:1024\n" \
    "ds_read_b32 %[tk], %[fu] offset:12\n" \
    "s_branch 11b\n" \
    "22:\n" \
    "ds_read_b32 %[vt], %[fu] offset:12\n" \
    "s_waitcnt lgkmcnt(0)\n" \
    "v_cmp_ge_u32 vcc, %[thr], %[vt]\n" \
    "s_cbranch_vccnz 22b\n" \
    "ds_read_b128 v[104:107], %[a10] offset:36864\n" \
    "ds_read_b128 v[108:111], %[a11] offset:36864\n" \
    "ds_read_b64_tr_b8 v[144:145], %[b10] offset:36864\n" \
    "ds_read_b64_tr_b8 v[146:147], %[b10] offset:37888\n" \
    "ds_read_b64_tr_b8 v[148:149], %[b10] offset:45056\n" \
    "ds_read_b64_tr_b8 v[150:151], %[b10] offset:46080\n" \
    "ds_read_b128 v[112:115], %[a10] offset:38912\n" \
    "ds_read_b128 v[116:119], %[a11] offset:38912\n" \
    "ds_read_b64_tr_b8 v[152:153], %[b11] offset:36864\n" \
    "ds_read_b64_tr_b8 v[154:155], %[b11] offset:37888\n" \
    "ds_read_b64_tr_b8 v[156:157], %[b11] offset:45056\n" \
    "ds_read_b64_tr_b8 v[158:159], %[b11] offset:46080\n" \
    "ds_read_b128 v[120:123], %[a10] offset:40960\n" \
    "ds_read_b128 v[124:127], %[a11] offset:40960\n" \
    "ds_read_b64_tr_b8 v[160:161], %[b12] offset:36864\n" \
    "ds_read_b64_tr_b8 v[162:163], %[b12] offset:37888\n" \
    "ds_read_b64_tr_b8 v[164:165], %[b12] offset:45056\n" \
    "ds_read_b64_tr_b8 v[166:167], %[b12] offset:46080\n" \
    "ds_read_b128 v[128:131], %[a10] offset:43008\n" \
    "ds_read_b128 v[132:135], %[a11] offset:43008\n" \
    "ds_read_b64_tr_b8 v[168:169], %[b13] offset:36864\n" \
    "ds_read_b64_tr_b8 v[170:171], %[b13] offset:37888\n" \
    "ds_read_b64_tr_b8 v[172:173], %[b13] offset:45056\n" \
    "ds_read_b64_tr_b8 v[174:175], %[b13] offset:46080\n" \
    "ds_write_b32 %[fr], %[vt] offset:1536\n" \
    "ds_read_b32 %[tk], %[fu] offset:0\n" \
    "s_branch 12b\n" \
    "23:\n" \
    "ds_read_b32 %[vt], %[fu] offset:0\n" \
    "s_waitcnt lgkmcnt(0)\n" \
    "v_cmp_ge_u32 vcc, %[thr], %[vt]\n" \
    "s_cbranch_vccnz 23b\n" \
    "ds_read_b128 v[32:35], %[a00] offset:0\n" \
    "ds_read_b128 v[36:39], %[a01] offset:0\n" \
    "ds_read_b64_tr_b8 v[72:73], %[b00] offset:0\n" \
    "ds_read_b64_tr_b8 v[74:75], %[b00] offset:1024\n" \
    "ds_read_b64_tr_b8 v[76:77], %[b00] offset:8192\n" \
    "ds_read_b64_tr_b8 v[78:79], %[b00] offset:9216\n" \
    "ds_read_b128 v[40:43], %[a00] offset:2048\n" \
    "ds_read_b128 v[44:47], %[a01] offset:2048\n" \
    "ds_read_b64_tr_b8 v[80:81], %[b01] offset:0\n" \
    "ds_read_b64_tr_b8 v[82:83], %[b01] offset:1024\n" \
    "ds_read_b64_tr_b8 v[84:85], %[b01] offset:8192\n" \
    "ds_read_b64_tr_b8 v[86:87], %[b01] offset:9216\n" \
    "ds_read_b128 v[48:51], %[a00] offset:4096\n" \
    "ds_read_b128 v[52:55], %[a01] offset:4096\n" \
    "ds_read_b64_tr_b8 v[88:89], %[b02] offset:0\n" \
    "ds_read_b64_tr_b8 v[90:91], %[b02] offset:1024\n" \
    "ds_read_b64_tr_b8 v[92:93], %[b02] offset:8192\n" \
    "ds_read_b64_tr_b8 v[94:95], %[b02] offset:9216\n" \
    "ds_read_b128 v[56:59], %[a00] offset:6144\n" \
    "ds_read_b128 v[60:63], %[a01] offset:6144\n" \
    "ds_read_b64_tr_b8 v[96:97], %[b03] offset:0\n" \
    "ds_read_b64_tr_b8 v[98:99], %[b03] offset:1024\n" \
    "ds_read_b64_tr_b8 v[100:101], %[b03] offset:8192\n" \
    "ds_read_b64_tr_b8 v[102:103], %[b03] offset:9216\n" \
    "ds_write_b32 %[fr], %[vt] offset:0\n" \
    "ds_read_b32 %[tk], %[fu] offset:4\n" \
    "s_branch 13b\n" \
    "3:\n" \
    "s_waitcnt lgkmcnt(0)\n" \
    "s_nop 7\n" \
    "s_nop 7\n" \
    ""
#define AG_ASM_CLOBBERS "v104", "v105", "v106", "v107", "v108", "v109", "v110", "v111", "v112", "v113", "v114", "v115", "v116", "v117", "v118", "v119", "v120", "v121", "v122", "v123", "v124", "v125", "v126", "v127", "v128", "v129", "v130", "v131", "v132", "v133", "v134", "v135", "v136", "v137", "v138", "v139", "v140", "v141", "v142", "v143", "v144", "v145", "v146", "v147", "v148", "v149", "v150", "v151", "v152", "v153", "v154", "v155", "v156", "v157", "v158", "v159", "v160", "v161", "v162", "v163", "v164", "v165", "v166", "v167", "v168", "v169", "v170", "v171", "v172", "v173", "v174", "v175"

__global__ void __launch_bounds__(NTHREADS, 2) __attribute__((amdgpu_waves_per_eu(2, 2))) ag_gemm_kernel(Args a) {
    extern __shared__ __attribute__((aligned(16))) unsigned char smem[];
    LAS unsigned char* lds = (LAS unsigned char*)smem;
    const int tid = threadIdx.x, lane = tid & 63;
    const int wid = __builtin_amdgcn_readfirstlane(tid >> 6);
    const int b = blockIdx.x, xcd = b & 7, bi = b >> 3;
    const int rank = a.rank;
    unsigned char* myws = a.ws[rank];
    GAS unsigned* callp = (GAS unsigned*)(myws + WS_CALL) + (xcd * 32 + bi);

    if (bi < 4) {
        if (xcd == 7) return;
        if (wid == 0) STAMP(0);
        const unsigned lo = (unsigned)(lane >> 3) * KDIM + (unsigned)((lane & 7) ^ (lane >> 4) ^ ((wid & 1) << 2)) * 16u;
        const unsigned char* xs = a.x + (size_t)(bi * 64 + wid * 8) * KDIM;
        u32x4 buf[PWIN];
        auto ld = [&](int s, u32x4& d0) {
            const unsigned char* p = xs + (size_t)(s >> 5) * 256 * KDIM + (s & 31) * 128;
            asm volatile("s_nop 4\n\tglobal_load_dwordx4 %0, %1, %2" : "=&v"(d0) : "v"(lo), "s"(p) : "memory");
        };
        unsigned callv;
        asm volatile("s_nop 4\n\tglobal_load_dword %0, %1, %2 sc1" : "=&v"(callv) : "v"(0u), "s"(callp) : "memory");
#pragma unroll
        for (int i = 0; i < PWIN; ++i) ld(i, buf[i]);
        asm volatile("s_waitcnt vmcnt(%1)" : "+v"(callv) : "n"(PWIN) : "memory");
        const unsigned call = uni(callv) + 1u;
        const unsigned peer = (unsigned)(rank + 1 + xcd) & 7u;
        unsigned char* pws = a.ws[peer];
        const unsigned so = (unsigned)lane * 16u + (unsigned)wid * 1024u;
        unsigned char* xd = pws + WS_GATHER + (size_t)(call & 1u) * NR * SHARD_BYTES + (size_t)rank * SHARD_BYTES +
                            (size_t)(bi >> 1) * NKT * 16384 + (bi & 1) * 8192;
        GAS unsigned* fl = (GAS unsigned*)(pws + WS_FLAGS) + rank * 256 + (bi >> 1) * 64 + (bi & 1);
        auto st = [&](int s, u32x4& d0) {
            unsigned char* p = xd + (size_t)(s >> 5) * 2 * NKT * 16384 + (size_t)(s & 31) * 16384;
            asm volatile("s_nop 4\n\tglobal_store_dwordx4 %1, %0, %2 sc0 sc1\n\ts_nop 1" : "+v"(d0) : "v"(so), "s"(p) : "memory");
        };
        auto done = [&](int s) {
            __builtin_amdgcn_s_barrier();
            if (wid == (s & 7)) {
                unsigned t0, t1;
                const GAS unsigned* fp = fl + (s >> 5) * 128 + (s & 31) * 2;
                asm volatile("s_nop 4\n\t"
                             "v_mov_b32 %0, 0\n\t"
                             "v_mov_b32 %1, %2\n\t"
                             "s_mov_b64 exec, 1\n\t"
                             "global_store_dword %0, %1, %3 sc0 sc1\n\t"
                             "s_mov_b64 exec, -1"
                             : "=&v"(t0), "=&v"(t1) : "s"(call), "s"(fp) : "memory");
                STAMP(16 + s);
            }
        };
        static_assert(PWIN == 8 || PWIN == 4 || PWIN == 2, "windows written for 2, 4 or 8");
#define AG_FIRST(i) if constexpr ((i) < PWIN) { wait_vm<(PWIN - 1) + (i)>(); st(i, buf[i]); ld((i) + PWIN, buf[i]); }
        AG_FIRST(0) AG_FIRST(1) AG_FIRST(2) AG_FIRST(3) AG_FIRST(4) AG_FIRST(5) AG_FIRST(6) AG_FIRST(7)
#undef AG_FIRST
#pragma unroll 1
        for (int s0 = PWIN; s0 < 64 - PWIN; s0 += PWIN) {
#pragma unroll
            for (int i = 0; i < PWIN; ++i) {
                wait_vm<2 * (PWIN - 1)>();
                done(s0 + i - PWIN);
                st(s0 + i, buf[i]);
                ld(s0 + i + PWIN, buf[i]);
            }
        }
#define AG_LAST(i) if constexpr ((i) < PWIN) { wait_vm<2 * (PWIN - 1 - (i)) + (i)>(); done(64 - 2 * PWIN + (i)); st(64 - PWIN + (i), buf[i]); }
        AG_LAST(0) AG_LAST(1) AG_LAST(2) AG_LAST(3) AG_LAST(4) AG_LAST(5) AG_LAST(6) AG_LAST(7)
#undef AG_LAST
#define AG_DRAIN(i) if constexpr ((i) < PWIN) { wait_vm<(PWIN - 1 - (i))>(); done(64 - PWIN + (i)); }
        AG_DRAIN(0) AG_DRAIN(1) AG_DRAIN(2) AG_DRAIN(3) AG_DRAIN(4) AG_DRAIN(5) AG_DRAIN(6) AG_DRAIN(7)
#undef AG_DRAIN
        if (tid == 0) __hip_atomic_store(callp, call, __ATOMIC_RELAXED, __HIP_MEMORY_SCOPE_AGENT);
        if (wid == 0) STAMP(1);
        return;
    }

    const int pj = (bi - 4) >> 2, par = xcd & 1;
    const int src = (rank + 1 + pj) & 7;
    const int ct = (xcd >> 1) * 4 + (bi & 3);
    const int grp0 = par * 7 + pj, grp1 = 14 + par;
    const int next = pj == 0 ? 2 : 1;
    if (wid == 0) STAMP(0);
    const unsigned call = uni(__hip_atomic_load(callp, __ATOMIC_RELAXED, __HIP_MEMORY_SCOPE_AGENT)) + 1u;
    if (tid < 128) *(LAS unsigned*)(lds + LDS_CTRL + tid * 4) = 0u;
    *(LAS unsigned*)(lds + C_FREE + tid * 4) = 0u;
    __syncthreads();
    if (tid == 0) __hip_atomic_store(callp, call, __ATOMIC_RELAXED, __HIP_MEMORY_SCOPE_AGENT);

    if (wid >= 4) {
        const int j = wid - 4;
        GAS unsigned* tmo = (GAS unsigned*)(myws + WS_TMO);
        const unsigned sw0 = (unsigned)((lane & 7) ^ (lane >> 4));
        const unsigned wo0 = (unsigned)(lane >> 3) * NL + sw0 * 16u, wo1 = (unsigned)(lane >> 3) * NL + (sw0 ^ 4u) * 16u;
        const unsigned xo0 = (unsigned)(lane >> 3) * KDIM + sw0 * 16u, xo1 = (unsigned)(8 + (lane >> 3)) * KDIM + (sw0 ^ 4u) * 16u;
        const unsigned ro = (unsigned)lane * 16u;
        const unsigned freev = C_FREE + (unsigned)j * 512u + (unsigned)(lane & 3) * 128u;
        LAS unsigned char* sa = lds + j * SLOT_BYTES;
        LAS unsigned char* sb = sa + SLOT_A;
        const unsigned char* gx = myws + WS_GATHER + (size_t)(call & 1u) * NR * SHARD_BYTES + (size_t)src * SHARD_BYTES;
        const unsigned long long t0 = __builtin_amdgcn_s_memrealtime();
        unsigned spins = 0;
#pragma unroll 1
        for (int u = j; u < NSTEPS; u += 4) {
            const int t = u >> 5, kt = u & 31, rt = 2 * t + par;
            if (u >= NSLOT) lds_wait_all_ge(freev, (unsigned)(u - NSLOT + 1));
            const unsigned char* wp = a.w + ct * 128 + (size_t)kt * 128 * NL;
            const unsigned char* xp = gx + (size_t)(rt * NKT + kt) * 16384;
            auto load_w = [&]() {
#pragma unroll
                for (int p = 0; p < 16; ++p)
                    __builtin_amdgcn_global_load_lds((const GAS void*)(wp + (size_t)p * 8 * NL + (((p >> 1) & 1) ? wo1 : wo0)),
                                                     (LAS void*)(sb + p * 1024), 16, 0, 0);
                if (t < next) {
                    const unsigned char* lp = a.x + (size_t)((t == 0 ? grp0 : grp1) * 32) * KDIM + kt * 128;
                    __builtin_amdgcn_global_load_lds((const GAS void*)(lp + xo0), (LAS void*)(sa + 8192), 16, 0, 0);
                    __builtin_amdgcn_global_load_lds((const GAS void*)(lp + xo1), (LAS void*)(sa + 9216), 16, 0, 0);
                    __builtin_amdgcn_global_load_lds((const GAS void*)(lp + 16 * KDIM + xo0), (LAS void*)(sa + 18432), 16, 0, 0);
                    __builtin_amdgcn_global_load_lds((const GAS void*)(lp + 16 * KDIM + xo1), (LAS void*)(sa + 19456), 16, 0, 0);
                }
            };
            auto load_x = [&]() {
#define AG_LDX(p) __builtin_amdgcn_global_load_lds((const GAS void*)(xp + ((p) & 12) * 1024 + ro), \
                                                   (LAS void*)(sa + ((p) & 12) * 1024 + ((p) >> 3) * 2048), 16, ((p) & 3) * 1024, 17);
                AG_LDX(0) AG_LDX(1) AG_LDX(2) AG_LDX(3) AG_LDX(4) AG_LDX(5) AG_LDX(6) AG_LDX(7)
                AG_LDX(8) AG_LDX(9) AG_LDX(10) AG_LDX(11) AG_LDX(12) AG_LDX(13) AG_LDX(14) AG_LDX(15)
#undef AG_LDX
            };
            if (lds_read_uni(C_READY) > (unsigned)u) {
                load_x();
                load_w();
            } else {
                load_w();
                const GAS unsigned* fl = (const GAS unsigned*)(myws + WS_FLAGS) + src * 256 + rt * 64 + lane;
                for (;;) {
                    const unsigned rdy = lds_read_uni(C_READY);
                    if (rdy > (unsigned)u) break;
                    if (rdy == (unsigned)u) {
                        const unsigned f = __hip_atomic_load(fl, __ATOMIC_RELAXED, __HIP_MEMORY_SCOPE_SYSTEM);
                        const unsigned long long m = __ballot(f >= call);
                        const unsigned n = (m == ~0ull) ? 32u : (unsigned)(__builtin_ctzll(~m) >> 1);
                        if (n > (unsigned)kt) { lds_write_uni(C_READY, (unsigned)(t * 32) + n); break; }
                        if ((++spins & 63u) == 0u &&
                            (uni(__hip_atomic_load(tmo, __ATOMIC_RELAXED, __HIP_MEMORY_SCOPE_AGENT)) == call ||
                             __builtin_amdgcn_s_memrealtime() - t0 > DEADLINE_TICKS)) {
                            __hip_atomic_store(tmo, call, __ATOMIC_RELAXED, __HIP_MEMORY_SCOPE_AGENT);
                            lds_write_uni(C_BAD, 1u);
                            lds_write_uni(C_READY, 1024u);
                            break;
                        }
                    }
                    __builtin_amdgcn_s_sleep(1);
                }
                STAMPL(16 + u);
                load_x();
            }
            wait_vm<0>();
            lds_write_uni(C_FULL + (unsigned)j * 4u, (unsigned)(u + 1));
            STAMPL(112 + u);
        }
        return;
    }

    {
        const int wm = wid >> 1, wn = wid & 1;
        const int fr = lane & 15, g = lane >> 4;
        const float scale = *(const CAS float*)a.sx * *(const CAS float*)a.sw;
        const unsigned a0 = (unsigned)((wm * 80 + fr) * 128 + ((g ^ (fr >> 1)) * 16));
        const unsigned a1 = a0 ^ 64u;
        const int q = fr >> 1, p2 = fr & 1;
        const unsigned F = (unsigned)(((q >> 1) & 3) | ((g & 1) << 2));
        unsigned bo[4];
#pragma unroll
        for (int nb = 0; nb < 4; ++nb) bo[nb] = (unsigned)SLOT_A + (unsigned)((16 * g + q) * 128) + (((unsigned)(wn * 4 + nb) ^ F) * 16u) + (unsigned)p2 * 8u;
        __builtin_amdgcn_s_setprio(2);
        constexpr unsigned HI = 2u * SLOT_BYTES;
        const unsigned freev = C_FREE + (unsigned)wid * 128u + (unsigned)(lane & 31) * 4u;
        const unsigned fullv = C_FULL;
        i32x32 f0, f1;
        i32x8 f2;
        unsigned tk, vt;
        asm volatile(AG_ASM_PROLOGUE
                     : "=&{v[32:63]}"(f0), "=&{v[64:95]}"(f1), "=&{v[96:103]}"(f2), [tk] "=&v"(tk), [vt] "=&v"(vt)
                     : [a00] "v"(a0), [a01] "v"(a1), [b00] "v"(bo[0]), [b01] "v"(bo[1]), [b02] "v"(bo[2]), [b03] "v"(bo[3]),
                       [fr] "v"(freev), [fu] "v"(fullv)
                     : "memory", "vcc");
        unsigned u = 0, thr;
        f32x32 acc0, acc1;
        f32x16 acc2;
#define AG_OPERANDS(UEND) \
            : "+{v[176:207]}"(acc0), "+{v[208:239]}"(acc1), "+{v[240:255]}"(acc2), "+{v[32:63]}"(f0), "+{v[64:95]}"(f1), "+{v[96:103]}"(f2), \
              [tk] "+v"(tk), [vt] "=&v"(vt), [u] "+s"(u), [thr] "=&s"(thr) \
            : [a00] "v"(a0), [a01] "v"(a1), [a10] "v"(a0 + HI), [a11] "v"(a1 + HI), \
              [b00] "v"(bo[0]), [b01] "v"(bo[1]), [b02] "v"(bo[2]), [b03] "v"(bo[3]), \
              [b10] "v"(bo[0] + HI), [b11] "v"(bo[1] + HI), [b12] "v"(bo[2] + HI), [b13] "v"(bo[3] + HI), \
              [fr] "v"(freev), [fu] "v"(fullv), [uend] "s"((unsigned)(UEND)), [ns] "s"((unsigned)NSTEPS), [nx] "s"(next) \
            : "memory", "vcc", "scc", AG_ASM_CLOBBERS
        auto frag = [&](int m, int n) -> f32x4 {
            const int e = (m & 1) * 16 + n * 4;
            return m < 2 ? (f32x4){acc0[e], acc0[e + 1], acc0[e + 2], acc0[e + 3]}
                 : m < 4 ? (f32x4){acc1[e], acc1[e + 1], acc1[e + 2], acc1[e + 3]}
                         : (f32x4){acc2[e], acc2[e + 1], acc2[e + 2], acc2[e + 3]};
        };
        float* const ob = a.out + ct * 128 + wn * 64;
        acc0 = 0.f; acc1 = 0.f; acc2 = 0.f;
        asm volatile(AG_ASM_LOOP5 AG_OPERANDS(NKT));
        if (wid == 0) STAMP(2);
        {
            const float sc = lds_read_uni(C_BAD) != 0u ? __builtin_nanf("") : scale;
            float* op = ob + (size_t)(src * MS + par * 128 + wm * 64 + fr) * NL + g * 4;
#pragma unroll
            for (int m = 0; m < 4; ++m)
#pragma unroll
                for (int n = 0; n < 4; ++n) store_wt(op + (size_t)m * 16 * NL + n * 16, frag(m, n) * sc);
            op = ob + (size_t)(rank * MS + grp0 * 32 + wm * 16 + fr) * NL + g * 4;
#pragma unroll
            for (int n = 0; n < 4; ++n) store_wt(op + n * 16, frag(4, n) * sc);
        }
        if (wid == 0) STAMP(3);
        acc0 = 0.f; acc1 = 0.f; acc2 = 0.f;
        asm volatile("s_cmp_lt_u32 %[nx], 2\n"
                     "s_cbranch_scc1 4f\n"
                     AG_ASM_LOOP5
                     "s_branch 5f\n"
                     "4:\n"
                     AG_ASM_LOOP4
                     "5:\n"
                     AG_OPERANDS(NSTEPS));
#undef AG_OPERANDS
        if (wid == 0) STAMP(4);
        {
            const float sc = lds_read_uni(C_BAD) != 0u ? __builtin_nanf("") : scale;
            const bool ext = next == 2;
            lds_wait_all_ge(C_FREE + (unsigned)(lane & 3) * 128u, (unsigned)(NSTEPS - 3));
            lds_wait_all_ge(C_FREE + (unsigned)(lane & 3) * 128u + 512u, (unsigned)(NSTEPS - 2));
            lds_wait_all_ge(C_FREE + (unsigned)(lane & 3) * 128u + 1024u, (unsigned)(NSTEPS - 1));
            LAS unsigned char* stg = lds + wid * 20480;
            const unsigned wo = (unsigned)fr * 256u;
#pragma unroll
            for (int m = 0; m < 4; ++m)
#pragma unroll
                for (int n = 0; n < 4; ++n) *(LAS f32x4*)(stg + m * 4096 + wo + (unsigned)(((n * 4 + g) ^ fr) * 16)) = frag(m, n) * sc;
            if (ext) {
#pragma unroll
                for (int n = 0; n < 4; ++n) *(LAS f32x4*)(stg + 4 * 4096 + wo + (unsigned)(((n * 4 + g) ^ fr) * 16)) = frag(4, n) * sc;
            }
            float* op = ob + (size_t)(src * MS + (2 + par) * 128 + wm * 64 + g) * NL + fr * 4;
#pragma unroll
            for (int i = 0; i < 16; ++i) {
                const int r4 = (i & 3) * 4;
                const f32x4 v = *(LAS f32x4*)(stg + (i >> 2) * 4096 + (unsigned)(r4 + g) * 256u + (unsigned)((fr ^ (r4 + g)) * 16));
                store_wt(op + (size_t)i * 4 * NL, v);
            }
            if (ext) {
                op = ob + (size_t)(rank * MS + grp1 * 32 + wm * 16 + g) * NL + fr * 4;
#pragma unroll
                for (int i = 0; i < 4; ++i) {
                    const f32x4 v = *(LAS f32x4*)(stg + 4 * 4096 + (unsigned)(i * 4 + g) * 256u + (unsigned)((fr ^ (i * 4 + g)) * 16));
                    store_wt(op + (size_t)i * 4 * NL, v);
                }
            }
        }
        if (wid == 0) STAMP(5);
    }
}

}

extern "C" void kernel_launch(int rank, int nranks, ncclComm_t comm,
                              hipStream_t stream,
                              void* const* d_in, const int* in_sizes, int n_in,
                              void* d_out, int out_size,
                              const int* in_dtypes, int out_dtype,
                              void* const* d_out_all,
                              void* const* d_ws_all, size_t ws_bytes) {
    using namespace agk;
    (void)comm; (void)in_dtypes; (void)out_dtype; (void)d_out_all;
    if (nranks != NR || n_in != 4 || in_sizes[0] != MS * KDIM || in_sizes[1] != KDIM * NL || out_size != NR * MS * NL ||
        ws_bytes < WS_NEEDED || (unsigned)rank >= (unsigned)NR) {
        fprintf(stderr, "ag_gemm: unexpected problem shape\n");
        return;
    }
    static bool attr_set = false;
    if (!attr_set) {
        HIP_CHECK(hipFuncSetAttribute((const void*)ag_gemm_kernel, hipFuncAttributeMaxDynamicSharedMemorySize, LDS_BYTES));
        attr_set = true;
    }
    Args a;
    a.x = (const unsigned char*)d_in[0];
    a.w = (const unsigned char*)d_in[1];
    a.sx = (const float*)d_in[2];
    a.sw = (const float*)d_in[3];
    a.out = (float*)d_out;
    for (int p = 0; p < NR; ++p) a.ws[p] = (unsigned char*)d_ws_all[p];
    a.rank = rank;
    hipLaunchKernelGGL(ag_gemm_kernel, dim3(NWG), dim3(NTHREADS), LDS_BYTES, stream, a);
    HIP_CHECK(hipGetLastError());
}
```

```cpp
#include <hip/hip_runtime.h>
#include <rccl/rccl.h>
#include <cstdio>
#include <cstdint>

#define HIP_CHECK(expr) do { hipError_t _e = (expr); if (_e != hipSuccess) { \
    fprintf(stderr, "HIP error %s at %s:%d\n", hipGetErrorString(_e), __FILE__, __LINE__); \
    return; } } while (0)

namespace agk {

#define LAS __attribute__((address_space(3)))
#define GAS __attribute__((address_space(1)))
#define CAS __attribute__((address_space(4)))
typedef int i32x8 __attribute__((ext_vector_type(8)));
typedef int i32x32 __attribute__((ext_vector_type(32)));
typedef unsigned u32x4 __attribute__((ext_vector_type(4)));
typedef float f32x4 __attribute__((ext_vector_type(4)));
typedef float f32x32 __attribute__((ext_vector_type(32)));

constexpr int NR = 8;
constexpr int KDIM = 4096, NL = 2048;
constexpr int MS = 512;
constexpr int NKT = 32;
constexpr int NSTEPS = 2 * NKT;
constexpr int NWG = 256, NTHREADS = 512;
constexpr int NSLOT = 6;
constexpr unsigned L_W = 0;
constexpr unsigned L_OWN = NSLOT * 16384;
constexpr unsigned LDS_CTRL = L_OWN + 32768;
constexpr int LDS_BYTES = LDS_CTRL + 2 * NSLOT * 512;
constexpr unsigned C_FULL = LDS_CTRL;
constexpr unsigned C_OPT = LDS_CTRL + 16;
constexpr unsigned C_READY = LDS_CTRL + 24;
constexpr unsigned C_BAD = LDS_CTRL + 20;
constexpr unsigned C_FREE = LDS_CTRL + NSLOT * 512;
constexpr size_t WS_FLAGS = 0;
constexpr size_t WS_CLR = 8192;
constexpr size_t WS_CALL = 16384;
constexpr size_t WS_TMO = 32768;
constexpr size_t WS_STAMP = 65536;
constexpr size_t WS_GATHER = 2u << 20;
constexpr size_t SHARD_BYTES = (size_t)MS * KDIM;
constexpr size_t WS_NEEDED = WS_GATHER + 2 * NR * SHARD_BYTES;
constexpr unsigned long long DEADLINE_TICKS = 100000000ull;
#define AG_PWIN 3
#define AG_LWIN 210
#define AG_LEAD 6
#define AG_XWARM 40
#define AG_WDELAY 100
#define AG_CODEPF 1
constexpr int PWIN = AG_PWIN;
static_assert(PWIN >= 2 && PWIN <= 8, "");

struct Args {
    const unsigned char* x;
    unsigned char* ws;
    const unsigned char* w;
    const float* sx;
    const float* sw;
    float* out;
    int rank;
    unsigned peers;
    unsigned char* pws[NR];
};

inline unsigned peer_map(int rank) {
    const unsigned grp = 0xB2u;
    unsigned m = 8u;
    int near = 1, far = 4;
    for (int i = 1; i < NR; ++i) {
        const int p = (rank + i) & 7;
        const int x = (((grp >> p) ^ (grp >> rank)) & 1u) ? far++ : near++;
        m |= (unsigned)p << (4 * x);
    }
    return m;
}

template <int N> __device__ __forceinline__ void wait_vm() { asm volatile("s_waitcnt vmcnt(%0)" ::"n"(N) : "memory"); }
__device__ __forceinline__ int lane_id() {
    int l;
    asm volatile("v_mbcnt_lo_u32_b32 %0, -1, 0\n\tv_mbcnt_hi_u32_b32 %0, -1, %0" : "=v"(l));
    return l;
}
__device__ __forceinline__ unsigned uni(unsigned v) { return (unsigned)__builtin_amdgcn_readfirstlane((int)v); }
__device__ __forceinline__ unsigned lds_read_uni(unsigned addr) {
    unsigned t, sv;
    asm volatile("v_mov_b32 %0, %2\n\t"
                 "ds_read_b32 %0, %0\n\t"
                 "s_waitcnt lgkmcnt(0)\n\t"
                 "v_readfirstlane_b32 %1, %0"
                 : "=&v"(t), "=s"(sv) : "s"(addr) : "memory");
    return sv;
}
__device__ __forceinline__ bool lds_all_ge(unsigned vaddr, unsigned vthr) {
    unsigned t;
    asm volatile("ds_read_b32 %0, %1\n\ts_waitcnt lgkmcnt(0)" : "=&v"(t) : "v"(vaddr) : "memory");
    return __ballot(t < vthr) == 0ull;
}
__device__ __forceinline__ void lds_write_uni(unsigned addr, unsigned val) {
    unsigned t0, t1;
    asm volatile("s_nop 1\n\t"
                 "v_mov_b32 %0, %2\n\t"
                 "v_mov_b32 %1, %3\n\t"
                 "s_mov_b64 exec, 1\n\t"
                 "ds_write_b32 %0, %1\n\t"
                 "s_mov_b64 exec, -1"
                 : "=&v"(t0), "=&v"(t1) : "s"(addr), "s"(val) : "memory");
}

#define STAMP(idx) do { } while (0)
#define STAMPL(idx) do { } while (0)

#define AG_LL0 48
#define AG_ASM_CONSUMER \
    "s_cmp_ge_u32 %[cj], 65536\n" \
    "s_cbranch_scc1 88f\n" \
    "s_getpc_b64 s[76:77]\n" \
    "89:\n" \
    "s_min_u32 %[c], %[cj], 95f-89b-16384\n" \
    "s_add_u32 s76, s76, %[c]\n" \
    "s_addc_u32 s77, s77, 0\n" \
    "global_load_dwordx4 v[32:35], v20, s[76:77] offset:0\n" \
    "global_load_dwordx4 v[32:35], v20, s[76:77] offset:1024\n" \
    "global_load_dwordx4 v[32:35], v20, s[76:77] offset:2048\n" \
    "global_load_dwordx4 v[32:35], v20, s[76:77] offset:3072\n" \
    "88:\n" \
    AG_WWARM \
    "v_mov_b64 v[160:161], 0\n" \
    "v_mov_b64 v[162:163], 0\n" \
    "v_mov_b64 v[164:165], 0\n" \
    "v_mov_b64 v[166:167], 0\n" \
    "v_mov_b64 v[168:169], 0\n" \
    "v_mov_b64 v[170:171], 0\n" \
    "v_mov_b64 v[172:173], 0\n" \
    "v_mov_b64 v[174:175], 0\n" \
    "v_mov_b64 v[176:177], 0\n" \
    "v_mov_b64 v[178:179], 0\n" \
    "v_mov_b64 v[180:181], 0\n" \
    "v_mov_b64 v[182:183], 0\n" \
    "v_mov_b64 v[184:185], 0\n" \
    "v_mov_b64 v[186:187], 0\n" \
    "v_mov_b64 v[188:189], 0\n" \
    "v_mov_b64 v[190:191], 0\n" \
    "v_mov_b64 v[192:193], 0\n" \
    "v_mov_b64 v[194:195], 0\n" \
    "v_mov_b64 v[196:197], 0\n" \
    "v_mov_b64 v[198:199], 0\n" \
    "v_mov_b64 v[200:201], 0\n" \
    "v_mov_b64 v[202:203], 0\n" \
    "v_mov_b64 v[204:205], 0\n" \
    "v_mov_b64 v[206:207], 0\n" \
    "v_mov_b64 v[208:209], 0\n" \
    "v_mov_b64 v[210:211], 0\n" \
    "v_mov_b64 v[212:213], 0\n" \
    "v_mov_b64 v[214:215], 0\n" \
    "v_mov_b64 v[216:217], 0\n" \
    "v_mov_b64 v[218:219], 0\n" \
    "v_mov_b64 v[220:221], 0\n" \
    "v_mov_b64 v[222:223], 0\n" \
    "v_mov_b64 v[224:225], 0\n" \
    "v_mov_b64 v[226:227], 0\n" \
    "v_mov_b64 v[228:229], 0\n" \
    "v_mov_b64 v[230:231], 0\n" \
    "v_mov_b64 v[232:233], 0\n" \
    "v_mov_b64 v[234:235], 0\n" \
    "v_mov_b64 v[236:237], 0\n" \
    "v_mov_b64 v[238:239], 0\n" \
    "v_mov_b64 v[240:241], 0\n" \
    "v_mov_b64 v[242:243], 0\n" \
    "v_mov_b64 v[244:245], 0\n" \
    "v_mov_b64 v[246:247], 0\n" \
    "v_mov_b64 v[248:249], 0\n" \
    "v_mov_b64 v[250:251], 0\n" \
    "v_mov_b64 v[252:253], 0\n" \
    "v_mov_b64 v[254:255], 0\n" \
    "90:\n" \
    "s_cmp_eq_u32 %[three], 0\n" \
    "s_cbranch_scc1 98f\n" \
    "s_cmp_lt_i32 %[u], 0\n" \
    "s_cbranch_scc0 0f\n" \
    "s_mov_b32 %[thr], 0\n" \
    "s_mov_b64 vcc, -1\n" \
    "s_branch 23f\n" \
    "0:\n" \
    "s_waitcnt lgkmcnt(14)\n" \
    "v_mfma_f32_16x16x128_f8f6f4 v[160:163], v[96:103], v[32:39], v[160:163] cbsz:1 blgp:1\n" \
    "s_add_u32 %[thr], %[u], 1\n" \
    "s_add_u32 %[n1], %[u], 3\n" \
    "v_mfma_f32_16x16x128_f8f6f4 v[192:195], v[96:103], v[40:47], v[192:195] cbsz:1 blgp:1\n" \
    "s_lshl_b32 %[so], %[n1], 14\n" \
    "s_waitcnt lgkmcnt(3)\n" \
    "v_mfma_f32_16x16x128_f8f6f4 v[224:227], v[96:103], v[24:31], v[224:227] cbsz:1 blgp:1\n" \
    "ds_read_b64_tr_b8 v[96:97], v8 offset:16384\n" \
    "ds_read_b64_tr_b8 v[98:99], v8 offset:17408\n" \
    "v_mfma_f32_16x16x128_f8f6f4 v[164:167], v[104:111], v[32:39], v[164:167] cbsz:1 blgp:1\n" \
    "ds_read_b64_tr_b8 v[100:101], v8 offset:24576\n" \
    "ds_read_b64_tr_b8 v[102:103], v8 offset:25600\n" \
    "s_waitcnt lgkmcnt(4)\n" \
    "v_mfma_f32_16x16x128_f8f6f4 v[196:199], v[104:111], v[40:47], v[196:199] cbsz:1 blgp:1\n" \
    "v_cmp_ge_u32 vcc, %[thr], v22\n" \
    "v_readfirstlane_b32 %[rv], v23\n" \
    "s_cmp_eq_u32 %[xi], %[n1]\n" \
    "s_cselect_b32 %[rv], %[rv], 0\n" \
    "v_mfma_f32_16x16x128_f8f6f4 v[228:231], v[104:111], v[24:31], v[228:231] cbsz:1 blgp:1\n" \
    "ds_read_b64_tr_b8 v[104:105], v9 offset:16384\n" \
    "ds_read_b64_tr_b8 v[106:107], v9 offset:17408\n" \
    "s_waitcnt lgkmcnt(14)\n" \
    "v_mfma_f32_16x16x128_f8f6f4 v[168:171], v[112:119], v[32:39], v[168:171] cbsz:1 blgp:1\n" \
    "ds_read_b64_tr_b8 v[108:109], v9 offset:24576\n" \
    "ds_read_b64_tr_b8 v[110:111], v9 offset:25600\n" \
    "s_cmp_gt_u32 %[rv], %[n1]\n" \
    "s_cselect_b32 s62, %[sz], 0\n" \
    "s_cselect_b32 s73, %[vc], s73\n" \
    "s_addc_u32 %[xi], %[xi], 0\n" \
    "v_mfma_f32_16x16x128_f8f6f4 v[200:203], v[112:119], v[40:47], v[200:203] cbsz:1 blgp:1\n" \
    "buffer_load_dwordx4 v[80:83], v20, s[60:63], %[so] offen offset:0 sc0 sc1\n" \
    "v_mfma_f32_16x16x128_f8f6f4 v[232:235], v[112:119], v[24:31], v[232:235] cbsz:1 blgp:1\n" \
    "ds_read_b64_tr_b8 v[112:113], v10 offset:16384\n" \
    "ds_read_b64_tr_b8 v[114:115], v10 offset:17408\n" \
    "buffer_load_dwordx4 v[84:87], v20, s[60:63], %[so] offen offset:1024 sc0 sc1\n" \
    "v_mfma_f32_16x16x128_f8f6f4 v[172:175], v[120:127], v[32:39], v[172:175] cbsz:1 blgp:1\n" \
    "ds_read_b64_tr_b8 v[116:117], v10 offset:24576\n" \
    "ds_read_b64_tr_b8 v[118:119], v10 offset:25600\n" \
    "buffer_load_dwordx4 v[88:91], v20, s[60:63], %[so] offen offset:2048 sc0 sc1\n" \
    "s_add_u32 %[vc], %[vc], 1\n" \
    "v_mfma_f32_16x16x128_f8f6f4 v[204:207], v[120:127], v[40:47], v[204:207] cbsz:1 blgp:1\n" \
    "buffer_load_dwordx4 v[92:95], v20, s[60:63], %[so] offen offset:3072 sc0 sc1\n" \
    "v_mfma_f32_16x16x128_f8f6f4 v[236:239], v[120:127], v[24:31], v[236:239] cbsz:1 blgp:1\n" \
    "ds_read_b64_tr_b8 v[120:121], v11 offset:16384\n" \
    "ds_read_b64_tr_b8 v[122:123], v11 offset:17408\n" \
    "s_waitcnt lgkmcnt(14)\n" \
    "v_mfma_f32_16x16x128_f8f6f4 v[176:179], v[128:135], v[32:39], v[176:179] cbsz:1 blgp:1\n" \
    "ds_read_b32 v23, v19 offset:16\n" \
    "ds_read_b64_tr_b8 v[124:125], v11 offset:24576\n" \
    "ds_read_b64_tr_b8 v[126:127], v11 offset:25600\n" \
    "v_mfma_f32_16x16x128_f8f6f4 v[208:211], v[128:135], v[40:47], v[208:211] cbsz:1 blgp:1\n" \
    "v_mfma_f32_16x16x128_f8f6f4 v[240:243], v[128:135], v[24:31], v[240:243] cbsz:1 blgp:1\n" \
    "ds_read_b64_tr_b8 v[128:129], v12 offset:16384\n" \
    "ds_read_b64_tr_b8 v[130:131], v12 offset:17408\n" \
    "v_mfma_f32_16x16x128_f8f6f4 v[180:183], v[136:143], v[32:39], v[180:183] cbsz:1 blgp:1\n" \
    "ds_read_b64_tr_b8 v[132:133], v12 offset:24576\n" \
    "ds_read_b64_tr_b8 v[134:135], v12 offset:25600\n" \
    "s_sub_u32 %[dl], %[sb], %[sa]\n" \
    "s_lshl_b32 %[dl], %[dl], 5\n" \
    "v_mfma_f32_16x16x128_f8f6f4 v[212:215], v[136:143], v[40:47], v[212:215] cbsz:1 blgp:1\n" \
    "s_cmp_le_u32 %[xi], %[n1]\n" \
    "s_cbranch_scc1 300f\n" \
    "310:\n" \
    "v_mfma_f32_16x16x128_f8f6f4 v[244:247], v[136:143], v[24:31], v[244:247] cbsz:1 blgp:1\n" \
    "ds_read_b64_tr_b8 v[136:137], v13 offset:16384\n" \
    "ds_read_b64_tr_b8 v[138:139], v13 offset:17408\n" \
    "s_waitcnt lgkmcnt(14)\n" \
    "v_mfma_f32_16x16x128_f8f6f4 v[184:187], v[144:151], v[32:39], v[184:187] cbsz:1 blgp:1\n" \
    "ds_read_b64_tr_b8 v[140:141], v13 offset:24576\n" \
    "ds_read_b64_tr_b8 v[142:143], v13 offset:25600\n" \
    "v_add_u32 v21, %[sb], v19\n" \
    "v_mfma_f32_16x16x128_f8f6f4 v[216:219], v[144:151], v[40:47], v[216:219] cbsz:1 blgp:1\n" \
    "s_sub_u32 %[c], %[vc], s71\n" \
    "s_add_u32 %[n1], %[u], 2\n" \
    "s_sub_u32 %[c], %[c], 3\n" \
    "s_sub_u32 %[n1], %[xi], %[n1]\n" \
    "v_mfma_f32_16x16x128_f8f6f4 v[248:251], v[144:151], v[24:31], v[248:251] cbsz:1 blgp:1\n" \
    "ds_read_b64_tr_b8 v[144:145], v14 offset:16384\n" \
    "ds_read_b64_tr_b8 v[146:147], v14 offset:17408\n" \
    "v_mfma_f32_16x16x128_f8f6f4 v[188:191], v[152:159], v[32:39], v[188:191] cbsz:1 blgp:1\n" \
    "ds_read_b64_tr_b8 v[148:149], v14 offset:24576\n" \
    "ds_read_b64_tr_b8 v[150:151], v14 offset:25600\n" \
    "s_andn2_b32 %[c], %[c], 3\n" \
    "s_lshr_b32 %[n1], %[n1], 31\n" \
    "v_mfma_f32_16x16x128_f8f6f4 v[220:223], v[152:159], v[40:47], v[220:223] cbsz:1 blgp:1\n" \
    "s_or_b32 %[c], %[c], %[n1]\n" \
    "v_mfma_f32_16x16x128_f8f6f4 v[252:255], v[152:159], v[24:31], v[252:255] cbsz:1 blgp:1\n" \
    "ds_read_b64_tr_b8 v[152:153], v15 offset:16384\n" \
    "ds_read_b64_tr_b8 v[154:155], v15 offset:17408\n" \
    "ds_read_b64_tr_b8 v[156:157], v15 offset:24576\n" \
    "ds_read_b64_tr_b8 v[158:159], v15 offset:25600\n" \
    "ds_read_b128 v[24:27], v16 offset:2048\n" \
    "ds_read_b128 v[28:31], v17 offset:2048\n" \
    "ds_write_b32 v7, v22 offset:512\n" \
    "ds_read_b32 v22, v21 offset:0\n" \
    "ds_read_b32 v23, v19 offset:16\n" \
    "s_cmp_lg_u32 %[c], 0\n" \
    "s_cbranch_scc1 20f\n" \
    "s_cbranch_vccnz 20f\n" \
    "s_waitcnt vmcnt(8)\n" \
    "10:\n" \
    "s_waitcnt lgkmcnt(14)\n" \
    "v_mfma_f32_16x16x128_f8f6f4 v[160:163], v[96:103], v[48:55], v[160:163] cbsz:1 blgp:1\n" \
    "s_add_u32 %[thr], %[u], 2\n" \
    "s_add_u32 %[n1], %[u], 4\n" \
    "v_mfma_f32_16x16x128_f8f6f4 v[192:195], v[96:103], v[56:63], v[192:195] cbsz:1 blgp:1\n" \
    "v_add_u32 v8, %[dl], v8\n" \
    "s_lshl_b32 %[so], %[n1], 14\n" \
    "s_waitcnt lgkmcnt(3)\n" \
    "v_mfma_f32_16x16x128_f8f6f4 v[224:227], v[96:103], v[24:31], v[224:227] cbsz:1 blgp:1\n" \
    "ds_read_b64_tr_b8 v[96:97], v8 offset:0\n" \
    "ds_read_b64_tr_b8 v[98:99], v8 offset:1024\n" \
    "v_mfma_f32_16x16x128_f8f6f4 v[164:167], v[104:111], v[48:55], v[164:167] cbsz:1 blgp:1\n" \
    "ds_read_b64_tr_b8 v[100:101], v8 offset:8192\n" \
    "ds_read_b64_tr_b8 v[102:103], v8 offset:9216\n" \
    "s_waitcnt lgkmcnt(4)\n" \
    "v_mfma_f32_16x16x128_f8f6f4 v[196:199], v[104:111], v[56:63], v[196:199] cbsz:1 blgp:1\n" \
    "v_add_u32 v9, %[dl], v9\n" \
    "v_cmp_ge_u32 vcc, %[thr], v22\n" \
    "v_readfirstlane_b32 %[rv], v23\n" \
    "s_cmp_eq_u32 %[xi], %[n1]\n" \
    "s_cselect_b32 %[rv], %[rv], 0\n" \
    "v_mfma_f32_16x16x128_f8f6f4 v[228:231], v[104:111], v[24:31], v[228:231] cbsz:1 blgp:1\n" \
    "ds_read_b64_tr_b8 v[104:105], v9 offset:0\n" \
    "ds_read_b64_tr_b8 v[106:107], v9 offset:1024\n" \
    "s_waitcnt lgkmcnt(14)\n" \
    "v_mfma_f32_16x16x128_f8f6f4 v[168:171], v[112:119], v[48:55], v[168:171] cbsz:1 blgp:1\n" \
    "ds_read_b64_tr_b8 v[108:109], v9 offset:8192\n" \
    "ds_read_b64_tr_b8 v[110:111], v9 offset:9216\n" \
    "s_cmp_gt_u32 %[rv], %[n1]\n" \
    "s_cselect_b32 s62, %[sz], 0\n" \
    "s_cselect_b32 s70, %[vc], s70\n" \
    "s_addc_u32 %[xi], %[xi], 0\n" \
    "v_mfma_f32_16x16x128_f8f6f4 v[200:203], v[112:119], v[56:63], v[200:203] cbsz:1 blgp:1\n" \
    "v_add_u32 v10, %[dl], v10\n" \
    "buffer_load_dwordx4 v[32:35], v20, s[60:63], %[so] offen offset:0 sc0 sc1\n" \
    "v_mfma_f32_16x16x128_f8f6f4 v[232:235], v[112:119], v[24:31], v[232:235] cbsz:1 blgp:1\n" \
    "ds_read_b64_tr_b8 v[112:113], v10 offset:0\n" \
    "ds_read_b64_tr_b8 v[114:115], v10 offset:1024\n" \
    "buffer_load_dwordx4 v[36:39], v20, s[60:63], %[so] offen offset:1024 sc0 sc1\n" \
    "v_mfma_f32_16x16x128_f8f6f4 v[172:175], v[120:127], v[48:55], v[172:175] cbsz:1 blgp:1\n" \
    "ds_read_b64_tr_b8 v[116:117], v10 offset:8192\n" \
    "ds_read_b64_tr_b8 v[118:119], v10 offset:9216\n" \
    "buffer_load_dwordx4 v[40:43], v20, s[60:63], %[so] offen offset:2048 sc0 sc1\n" \
    "s_add_u32 %[vc], %[vc], 1\n" \
    "v_mfma_f32_16x16x128_f8f6f4 v[204:207], v[120:127], v[56:63], v[204:207] cbsz:1 blgp:1\n" \
    "v_add_u32 v11, %[dl], v11\n" \
    "buffer_load_dwordx4 v[44:47], v20, s[60:63], %[so] offen offset:3072 sc0 sc1\n" \
    "v_mfma_f32_16x16x128_f8f6f4 v[236:239], v[120:127], v[24:31], v[236:239] cbsz:1 blgp:1\n" \
    "ds_read_b64_tr_b8 v[120:121], v11 offset:0\n" \
    "ds_read_b64_tr_b8 v[122:123], v11 offset:1024\n" \
    "s_waitcnt lgkmcnt(14)\n" \
    "v_mfma_f32_16x16x128_f8f6f4 v[176:179], v[128:135], v[48:55], v[176:179] cbsz:1 blgp:1\n" \
    "ds_read_b32 v23, v19 offset:16\n" \
    "ds_read_b64_tr_b8 v[124:125], v11 offset:8192\n" \
    "ds_read_b64_tr_b8 v[126:127], v11 offset:9216\n" \
    "s_mov_b32 %[sa], %[sb]\n" \
    "s_add_u32 %[sb], %[sb], 1024\n" \
    "v_mfma_f32_16x16x128_f8f6f4 v[208:211], v[128:135], v[56:63], v[208:211] cbsz:1 blgp:1\n" \
    "v_add_u32 v12, %[dl], v12\n" \
    "s_cmp_eq_u32 %[sb], 3072\n" \
    "s_cselect_b32 %[sb], 0, %[sb]\n" \
    "v_mfma_f32_16x16x128_f8f6f4 v[240:243], v[128:135], v[24:31], v[240:243] cbsz:1 blgp:1\n" \
    "ds_read_b64_tr_b8 v[128:129], v12 offset:0\n" \
    "ds_read_b64_tr_b8 v[130:131], v12 offset:1024\n" \
    "v_mfma_f32_16x16x128_f8f6f4 v[180:183], v[136:143], v[48:55], v[180:183] cbsz:1 blgp:1\n" \
    "ds_read_b64_tr_b8 v[132:133], v12 offset:8192\n" \
    "ds_read_b64_tr_b8 v[134:135], v12 offset:9216\n" \
    "v_mfma_f32_16x16x128_f8f6f4 v[212:215], v[136:143], v[56:63], v[212:215] cbsz:1 blgp:1\n" \
    "v_add_u32 v13, %[dl], v13\n" \
    "s_cmp_le_u32 %[xi], %[n1]\n" \
    "s_cbranch_scc1 301f\n" \
    "311:\n" \
    "v_mfma_f32_16x16x128_f8f6f4 v[244:247], v[136:143], v[24:31], v[244:247] cbsz:1 blgp:1\n" \
    "ds_read_b64_tr_b8 v[136:137], v13 offset:0\n" \
    "ds_read_b64_tr_b8 v[138:139], v13 offset:1024\n" \
    "s_waitcnt lgkmcnt(14)\n" \
    "v_mfma_f32_16x16x128_f8f6f4 v[184:187], v[144:151], v[48:55], v[184:187] cbsz:1 blgp:1\n" \
    "ds_read_b64_tr_b8 v[140:141], v13 offset:8192\n" \
    "ds_read_b64_tr_b8 v[142:143], v13 offset:9216\n" \
    "v_add_u32 v7, %[sa], v18\n" \
    "v_mfma_f32_16x16x128_f8f6f4 v[216:219], v[144:151], v[56:63], v[216:219] cbsz:1 blgp:1\n" \
    "v_add_u32 v14, %[dl], v14\n" \
    "s_sub_u32 %[c], %[vc], s72\n" \
    "s_add_u32 %[n1], %[u], 3\n" \
    "s_sub_u32 %[c], %[c], 3\n" \
    "s_sub_u32 %[n1], %[xi], %[n1]\n" \
    "v_mfma_f32_16x16x128_f8f6f4 v[248:251], v[144:151], v[24:31], v[248:251] cbsz:1 blgp:1\n" \
    "ds_read_b64_tr_b8 v[144:145], v14 offset:0\n" \
    "ds_read_b64_tr_b8 v[146:147], v14 offset:1024\n" \
    "v_mfma_f32_16x16x128_f8f6f4 v[188:191], v[152:159], v[48:55], v[188:191] cbsz:1 blgp:1\n" \
    "ds_read_b64_tr_b8 v[148:149], v14 offset:8192\n" \
    "ds_read_b64_tr_b8 v[150:151], v14 offset:9216\n" \
    "s_andn2_b32 %[c], %[c], 3\n" \
    "s_lshr_b32 %[n1], %[n1], 31\n" \
    "v_mfma_f32_16x16x128_f8f6f4 v[220:223], v[152:159], v[56:63], v[220:223] cbsz:1 blgp:1\n" \
    "v_add_u32 v15, %[dl], v15\n" \
    "s_or_b32 %[c], %[c], %[n1]\n" \
    "v_mfma_f32_16x16x128_f8f6f4 v[252:255], v[152:159], v[24:31], v[252:255] cbsz:1 blgp:1\n" \
    "ds_read_b64_tr_b8 v[152:153], v15 offset:0\n" \
    "ds_read_b64_tr_b8 v[154:155], v15 offset:1024\n" \
    "ds_read_b64_tr_b8 v[156:157], v15 offset:8192\n" \
    "ds_read_b64_tr_b8 v[158:159], v15 offset:9216\n" \
    "ds_read_b128 v[24:27], v16 offset:4096\n" \
    "ds_read_b128 v[28:31], v17 offset:4096\n" \
    "ds_write_b32 v7, v22 offset:0\n" \
    "ds_read_b32 v22, v21 offset:512\n" \
    "ds_read_b32 v23, v19 offset:16\n" \
    "s_cmp_lg_u32 %[c], 0\n" \
    "s_cbranch_scc1 21f\n" \
    "s_cbranch_vccnz 21f\n" \
    "s_waitcnt vmcnt(8)\n" \
    "11:\n" \
    "s_waitcnt lgkmcnt(14)\n" \
    "v_mfma_f32_16x16x128_f8f6f4 v[160:163], v[96:103], v[64:71], v[160:163] cbsz:1 blgp:1\n" \
    "s_add_u32 %[thr], %[u], 3\n" \
    "s_add_u32 %[n1], %[u], 5\n" \
    "v_mfma_f32_16x16x128_f8f6f4 v[192:195], v[96:103], v[72:79], v[192:195] cbsz:1 blgp:1\n" \
    "s_lshl_b32 %[so], %[n1], 14\n" \
    "s_waitcnt lgkmcnt(3)\n" \
    "v_mfma_f32_16x16x128_f8f6f4 v[224:227], v[96:103], v[24:31], v[224:227] cbsz:1 blgp:1\n" \
    "ds_read_b64_tr_b8 v[96:97], v8 offset:16384\n" \
    "ds_read_b64_tr_b8 v[98:99], v8 offset:17408\n" \
    "v_mfma_f32_16x16x128_f8f6f4 v[164:167], v[104:111], v[64:71], v[164:167] cbsz:1 blgp:1\n" \
    "ds_read_b64_tr_b8 v[100:101], v8 offset:24576\n" \
    "ds_read_b64_tr_b8 v[102:103], v8 offset:25600\n" \
    "s_waitcnt lgkmcnt(4)\n" \
    "v_mfma_f32_16x16x128_f8f6f4 v[196:199], v[104:111], v[72:79], v[196:199] cbsz:1 blgp:1\n" \
    "v_cmp_ge_u32 vcc, %[thr], v22\n" \
    "v_readfirstlane_b32 %[rv], v23\n" \
    "s_cmp_eq_u32 %[xi], %[n1]\n" \
    "s_cselect_b32 %[rv], %[rv], 0\n" \
    "v_mfma_f32_16x16x128_f8f6f4 v[228:231], v[104:111], v[24:31], v[228:231] cbsz:1 blgp:1\n" \
    "ds_read_b64_tr_b8 v[104:105], v9 offset:16384\n" \
    "ds_read_b64_tr_b8 v[106:107], v9 offset:17408\n" \
    "s_waitcnt lgkmcnt(14)\n" \
    "v_mfma_f32_16x16x128_f8f6f4 v[168:171], v[112:119], v[64:71], v[168:171] cbsz:1 blgp:1\n" \
    "ds_read_b64_tr_b8 v[108:109], v9 offset:24576\n" \
    "ds_read_b64_tr_b8 v[110:111], v9 offset:25600\n" \
    "s_cmp_gt_u32 %[rv], %[n1]\n" \
    "s_cselect_b32 s62, %[sz], 0\n" \
    "s_cselect_b32 s71, %[vc], s71\n" \
    "s_addc_u32 %[xi], %[xi], 0\n" \
    "v_mfma_f32_16x16x128_f8f6f4 v[200:203], v[112:119], v[72:79], v[200:203] cbsz:1 blgp:1\n" \
    "buffer_load_dwordx4 v[48:51], v20, s[60:63], %[so] offen offset:0 sc0 sc1\n" \
    "v_mfma_f32_16x16x128_f8f6f4 v[232:235], v[112:119], v[24:31], v[232:235] cbsz:1 blgp:1\n" \
    "ds_read_b64_tr_b8 v[112:113], v10 offset:16384\n" \
    "ds_read_b64_tr_b8 v[114:115], v10 offset:17408\n" \
    "buffer_load_dwordx4 v[52:55], v20, s[60:63], %[so] offen offset:1024 sc0 sc1\n" \
    "v_mfma_f32_16x16x128_f8f6f4 v[172:175], v[120:127], v[64:71], v[172:175] cbsz:1 blgp:1\n" \
    "ds_read_b64_tr_b8 v[116:117], v10 offset:24576\n" \
    "ds_read_b64_tr_b8 v[118:119], v10 offset:25600\n" \
    "buffer_load_dwordx4 v[56:59], v20, s[60:63], %[so] offen offset:2048 sc0 sc1\n" \
    "s_add_u32 %[vc], %[vc], 1\n" \
    "v_mfma_f32_16x16x128_f8f6f4 v[204:207], v[120:127], v[72:79], v[204:207] cbsz:1 blgp:1\n" \
    "buffer_load_dwordx4 v[60:63], v20, s[60:63], %[so] offen offset:3072 sc0 sc1\n" \
    "v_mfma_f32_16x16x128_f8f6f4 v[236:239], v[120:127], v[24:31], v[236:239] cbsz:1 blgp:1\n" \
    "ds_read_b64_tr_b8 v[120:121], v11 offset:16384\n" \
    "ds_read_b64_tr_b8 v[122:123], v11 offset:17408\n" \
    "s_waitcnt lgkmcnt(14)\n" \
    "v_mfma_f32_16x16x128_f8f6f4 v[176:179], v[128:135], v[64:71], v[176:179] cbsz:1 blgp:1\n" \
    "ds_read_b32 v23, v19 offset:16\n" \
    "ds_read_b64_tr_b8 v[124:125], v11 offset:24576\n" \
    "ds_read_b64_tr_b8 v[126:127], v11 offset:25600\n" \
    "v_mfma_f32_16x16x128_f8f6f4 v[208:211], v[128:135], v[72:79], v[208:211] cbsz:1 blgp:1\n" \
    "v_mfma_f32_16x16x128_f8f6f4 v[240:243], v[128:135], v[24:31], v[240:243] cbsz:1 blgp:1\n" \
    "ds_read_b64_tr_b8 v[128:129], v12 offset:16384\n" \
    "ds_read_b64_tr_b8 v[130:131], v12 offset:17408\n" \
    "v_mfma_f32_16x16x128_f8f6f4 v[180:183], v[136:143], v[64:71], v[180:183] cbsz:1 blgp:1\n" \
    "ds_read_b64_tr_b8 v[132:133], v12 offset:24576\n" \
    "ds_read_b64_tr_b8 v[134:135], v12 offset:25600\n" \
    "s_sub_u32 %[dl], %[sb], %[sa]\n" \
    "s_lshl_b32 %[dl], %[dl], 5\n" \
    "v_mfma_f32_16x16x128_f8f6f4 v[212:215], v[136:143], v[72:79], v[212:215] cbsz:1 blgp:1\n" \
    "s_cmp_le_u32 %[xi], %[n1]\n" \
    "s_cbranch_scc1 302f\n" \
    "312:\n" \
    "v_mfma_f32_16x16x128_f8f6f4 v[244:247], v[136:143], v[24:31], v[244:247] cbsz:1 blgp:1\n" \
    "ds_read_b64_tr_b8 v[136:137], v13 offset:16384\n" \
    "ds_read_b64_tr_b8 v[138:139], v13 offset:17408\n" \
    "s_waitcnt lgkmcnt(14)\n" \
    "v_mfma_f32_16x16x128_f8f6f4 v[184:187], v[144:151], v[64:71], v[184:187] cbsz:1 blgp:1\n" \
    "ds_read_b64_tr_b8 v[140:141], v13 offset:24576\n" \
    "ds_read_b64_tr_b8 v[142:143], v13 offset:25600\n" \
    "v_add_u32 v21, %[sb], v19\n" \
    "v_mfma_f32_16x16x128_f8f6f4 v[216:219], v[144:151], v[72:79], v[216:219] cbsz:1 blgp:1\n" \
    "s_sub_u32 %[c], %[vc], s73\n" \
    "s_add_u32 %[n1], %[u], 4\n" \
    "s_sub_u32 %[c], %[c], 3\n" \
    "s_sub_u32 %[n1], %[xi], %[n1]\n" \
    "v_mfma_f32_16x16x128_f8f6f4 v[248:251], v[144:151], v[24:31], v[248:251] cbsz:1 blgp:1\n" \
    "ds_read_b64_tr_b8 v[144:145], v14 offset:16384\n" \
    "ds_read_b64_tr_b8 v[146:147], v14 offset:17408\n" \
    "v_mfma_f32_16x16x128_f8f6f4 v[188:191], v[152:159], v[64:71], v[188:191] cbsz:1 blgp:1\n" \
    "ds_read_b64_tr_b8 v[148:149], v14 offset:24576\n" \
    "ds_read_b64_tr_b8 v[150:151], v14 offset:25600\n" \
    "s_andn2_b32 %[c], %[c], 3\n" \
    "s_lshr_b32 %[n1], %[n1], 31\n" \
    "v_mfma_f32_16x16x128_f8f6f4 v[220:223], v[152:159], v[72:79], v[220:223] cbsz:1 blgp:1\n" \
    "s_or_b32 %[c], %[c], %[n1]\n" \
    "v_mfma_f32_16x16x128_f8f6f4 v[252:255], v[152:159], v[24:31], v[252:255] cbsz:1 blgp:1\n" \
    "ds_read_b64_tr_b8 v[152:153], v15 offset:16384\n" \
    "ds_read_b64_tr_b8 v[154:155], v15 offset:17408\n" \
    "ds_read_b64_tr_b8 v[156:157], v15 offset:24576\n" \
    "ds_read_b64_tr_b8 v[158:159], v15 offset:25600\n" \
    "ds_read_b128 v[24:27], v16 offset:6144\n" \
    "ds_read_b128 v[28:31], v17 offset:6144\n" \
    "ds_write_b32 v7, v22 offset:512\n" \
    "ds_read_b32 v22, v21 offset:0\n" \
    "ds_read_b32 v23, v19 offset:16\n" \
    "s_cmp_lg_u32 %[c], 0\n" \
    "s_cbranch_scc1 22f\n" \
    "s_cbranch_vccnz 22f\n" \
    "s_waitcnt vmcnt(8)\n" \
    "12:\n" \
    "s_waitcnt lgkmcnt(14)\n" \
    "v_mfma_f32_16x16x128_f8f6f4 v[160:163], v[96:103], v[80:87], v[160:163] cbsz:1 blgp:1\n" \
    "s_add_u32 %[thr], %[u], 4\n" \
    "s_cmp_lt_u32 %[thr], 64\n" \
    "s_cselect_b32 %[thr], %[thr], 0\n" \
    "s_add_u32 %[n1], %[u], 6\n" \
    "v_mfma_f32_16x16x128_f8f6f4 v[192:195], v[96:103], v[88:95], v[192:195] cbsz:1 blgp:1\n" \
    "v_add_u32 v8, %[dl], v8\n" \
    "s_lshl_b32 %[so], %[n1], 14\n" \
    "s_waitcnt lgkmcnt(3)\n" \
    "v_mfma_f32_16x16x128_f8f6f4 v[224:227], v[96:103], v[24:31], v[224:227] cbsz:1 blgp:1\n" \
    "ds_read_b64_tr_b8 v[96:97], v8 offset:0\n" \
    "ds_read_b64_tr_b8 v[98:99], v8 offset:1024\n" \
    "v_mfma_f32_16x16x128_f8f6f4 v[164:167], v[104:111], v[80:87], v[164:167] cbsz:1 blgp:1\n" \
    "ds_read_b64_tr_b8 v[100:101], v8 offset:8192\n" \
    "ds_read_b64_tr_b8 v[102:103], v8 offset:9216\n" \
    "s_waitcnt lgkmcnt(4)\n" \
    "v_mfma_f32_16x16x128_f8f6f4 v[196:199], v[104:111], v[88:95], v[196:199] cbsz:1 blgp:1\n" \
    "v_add_u32 v9, %[dl], v9\n" \
    "v_cmp_ge_u32 vcc, %[thr], v22\n" \
    "v_readfirstlane_b32 %[rv], v23\n" \
    "s_cmp_eq_u32 %[xi], %[n1]\n" \
    "s_cselect_b32 %[rv], %[rv], 0\n" \
    "v_mfma_f32_16x16x128_f8f6f4 v[228:231], v[104:111], v[24:31], v[228:231] cbsz:1 blgp:1\n" \
    "ds_read_b64_tr_b8 v[104:105], v9 offset:0\n" \
    "ds_read_b64_tr_b8 v[106:107], v9 offset:1024\n" \
    "s_waitcnt lgkmcnt(14)\n" \
    "v_mfma_f32_16x16x128_f8f6f4 v[168:171], v[112:119], v[80:87], v[168:171] cbsz:1 blgp:1\n" \
    "ds_read_b64_tr_b8 v[108:109], v9 offset:8192\n" \
    "ds_read_b64_tr_b8 v[110:111], v9 offset:9216\n" \
    "s_cmp_gt_u32 %[rv], %[n1]\n" \
    "s_cselect_b32 s62, %[sz], 0\n" \
    "s_cselect_b32 s72, %[vc], s72\n" \
    "s_addc_u32 %[xi], %[xi], 0\n" \
    "v_mfma_f32_16x16x128_f8f6f4 v[200:203], v[112:119], v[88:95], v[200:203] cbsz:1 blgp:1\n" \
    "v_add_u32 v10, %[dl], v10\n" \
    "buffer_load_dwordx4 v[64:67], v20, s[60:63], %[so] offen offset:0 sc0 sc1\n" \
    "v_mfma_f32_16x16x128_f8f6f4 v[232:235], v[112:119], v[24:31], v[232:235] cbsz:1 blgp:1\n" \
    "ds_read_b64_tr_b8 v[112:113], v10 offset:0\n" \
    "ds_read_b64_tr_b8 v[114:115], v10 offset:1024\n" \
    "buffer_load_dwordx4 v[68:71], v20, s[60:63], %[so] offen offset:1024 sc0 sc1\n" \
    "v_mfma_f32_16x16x128_f8f6f4 v[172:175], v[120:127], v[80:87], v[172:175] cbsz:1 blgp:1\n" \
    "ds_read_b64_tr_b8 v[116:117], v10 offset:8192\n" \
    "ds_read_b64_tr_b8 v[118:119], v10 offset:9216\n" \
    "buffer_load_dwordx4 v[72:75], v20, s[60:63], %[so] offen offset:2048 sc0 sc1\n" \
    "s_add_u32 %[vc], %[vc], 1\n" \
    "v_mfma_f32_16x16x128_f8f6f4 v[204:207], v[120:127], v[88:95], v[204:207] cbsz:1 blgp:1\n" \
    "v_add_u32 v11, %[dl], v11\n" \
    "buffer_load_dwordx4 v[76:79], v20, s[60:63], %[so] offen offset:3072 sc0 sc1\n" \
    "v_mfma_f32_16x16x128_f8f6f4 v[236:239], v[120:127], v[24:31], v[236:239] cbsz:1 blgp:1\n" \
    "ds_read_b64_tr_b8 v[120:121], v11 offset:0\n" \
    "ds_read_b64_tr_b8 v[122:123], v11 offset:1024\n" \
    "s_waitcnt lgkmcnt(14)\n" \
    "v_mfma_f32_16x16x128_f8f6f4 v[176:179], v[128:135], v[80:87], v[176:179] cbsz:1 blgp:1\n" \
    "ds_read_b32 v23, v19 offset:16\n" \
    "ds_read_b64_tr_b8 v[124:125], v11 offset:8192\n" \
    "ds_read_b64_tr_b8 v[126:127], v11 offset:9216\n" \
    "s_mov_b32 %[sa], %[sb]\n" \
    "s_add_u32 %[sb], %[sb], 1024\n" \
    "v_mfma_f32_16x16x128_f8f6f4 v[208:211], v[128:135], v[88:95], v[208:211] cbsz:1 blgp:1\n" \
    "v_add_u32 v12, %[dl], v12\n" \
    "s_cmp_eq_u32 %[sb], 3072\n" \
    "s_cselect_b32 %[sb], 0, %[sb]\n" \
    "v_mfma_f32_16x16x128_f8f6f4 v[240:243], v[128:135], v[24:31], v[240:243] cbsz:1 blgp:1\n" \
    "ds_read_b64_tr_b8 v[128:129], v12 offset:0\n" \
    "ds_read_b64_tr_b8 v[130:131], v12 offset:1024\n" \
    "v_mfma_f32_16x16x128_f8f6f4 v[180:183], v[136:143], v[80:87], v[180:183] cbsz:1 blgp:1\n" \
    "ds_read_b64_tr_b8 v[132:133], v12 offset:8192\n" \
    "ds_read_b64_tr_b8 v[134:135], v12 offset:9216\n" \
    "v_mfma_f32_16x16x128_f8f6f4 v[212:215], v[136:143], v[88:95], v[212:215] cbsz:1 blgp:1\n" \
    "v_add_u32 v13, %[dl], v13\n" \
    "s_cmp_le_u32 %[xi], %[n1]\n" \
    "s_cbranch_scc1 303f\n" \
    "313:\n" \
    "v_mfma_f32_16x16x128_f8f6f4 v[244:247], v[136:143], v[24:31], v[244:247] cbsz:1 blgp:1\n" \
    "ds_read_b64_tr_b8 v[136:137], v13 offset:0\n" \
    "ds_read_b64_tr_b8 v[138:139], v13 offset:1024\n" \
    "s_waitcnt lgkmcnt(14)\n" \
    "v_mfma_f32_16x16x128_f8f6f4 v[184:187], v[144:151], v[80:87], v[184:187] cbsz:1 blgp:1\n" \
    "ds_read_b64_tr_b8 v[140:141], v13 offset:8192\n" \
    "ds_read_b64_tr_b8 v[142:143], v13 offset:9216\n" \
    "v_add_u32 v7, %[sa], v18\n" \
    "v_mfma_f32_16x16x128_f8f6f4 v[216:219], v[144:151], v[88:95], v[216:219] cbsz:1 blgp:1\n" \
    "v_add_u32 v14, %[dl], v14\n" \
    "s_sub_u32 %[c], %[vc], s70\n" \
    "s_add_u32 %[n1], %[u], 5\n" \
    "s_sub_u32 %[c], %[c], 3\n" \
    "s_sub_u32 %[n1], %[xi], %[n1]\n" \
    "v_mfma_f32_16x16x128_f8f6f4 v[248:251], v[144:151], v[24:31], v[248:251] cbsz:1 blgp:1\n" \
    "ds_read_b64_tr_b8 v[144:145], v14 offset:0\n" \
    "ds_read_b64_tr_b8 v[146:147], v14 offset:1024\n" \
    "v_mfma_f32_16x16x128_f8f6f4 v[188:191], v[152:159], v[80:87], v[188:191] cbsz:1 blgp:1\n" \
    "ds_read_b64_tr_b8 v[148:149], v14 offset:8192\n" \
    "ds_read_b64_tr_b8 v[150:151], v14 offset:9216\n" \
    "s_andn2_b32 %[c], %[c], 3\n" \
    "s_lshr_b32 %[n1], %[n1], 31\n" \
    "v_mfma_f32_16x16x128_f8f6f4 v[220:223], v[152:159], v[88:95], v[220:223] cbsz:1 blgp:1\n" \
    "v_add_u32 v15, %[dl], v15\n" \
    "s_or_b32 %[c], %[c], %[n1]\n" \
    "v_mfma_f32_16x16x128_f8f6f4 v[252:255], v[152:159], v[24:31], v[252:255] cbsz:1 blgp:1\n" \
    "ds_read_b64_tr_b8 v[152:153], v15 offset:0\n" \
    "ds_read_b64_tr_b8 v[154:155], v15 offset:1024\n" \
    "ds_read_b64_tr_b8 v[156:157], v15 offset:8192\n" \
    "ds_read_b64_tr_b8 v[158:159], v15 offset:9216\n" \
    "ds_read_b128 v[24:27], v16 offset:0\n" \
    "ds_read_b128 v[28:31], v17 offset:0\n" \
    "ds_write_b32 v7, v22 offset:0\n" \
    "ds_read_b32 v22, v21 offset:512\n" \
    "ds_read_b32 v23, v19 offset:16\n" \
    "s_cmp_lg_u32 %[c], 0\n" \
    "s_cbranch_scc1 23f\n" \
    "s_cbranch_vccnz 23f\n" \
    "s_waitcnt vmcnt(8)\n" \
    "13:\n" \
    "s_add_u32 %[u], %[u], 4\n" \
    "s_cmp_lt_u32 %[u], %[uend]\n" \
    "s_cbranch_scc1 0b\n" \
    "s_branch 9f\n" \
    "20:\n" \
    "s_add_u32 %[c], %[u], 0\n" \
    "s_cbranch_vccz 30f\n" \
    "v_add_u32 v7, %[sa], v18\n" \
    "s_waitcnt lgkmcnt(0)\n" \
    "v_add_u32 v23, %[sa], v19\n" \
    "40:\n" \
    "ds_read_b32 v21, v23 offset:512\n" \
    "s_waitcnt lgkmcnt(0)\n" \
    "v_cmp_ge_u32 vcc, %[thr], v21\n" \
    "s_cbranch_vccnz 40b\n" \
    "ds_read_b64_tr_b8 v[96:97], v8 offset:16384\n" \
    "ds_read_b64_tr_b8 v[98:99], v8 offset:17408\n" \
    "ds_read_b64_tr_b8 v[100:101], v8 offset:24576\n" \
    "ds_read_b64_tr_b8 v[102:103], v8 offset:25600\n" \
    "ds_read_b64_tr_b8 v[104:105], v9 offset:16384\n" \
    "ds_read_b64_tr_b8 v[106:107], v9 offset:17408\n" \
    "ds_read_b64_tr_b8 v[108:109], v9 offset:24576\n" \
    "ds_read_b64_tr_b8 v[110:111], v9 offset:25600\n" \
    "ds_read_b64_tr_b8 v[112:113], v10 offset:16384\n" \
    "ds_read_b64_tr_b8 v[114:115], v10 offset:17408\n" \
    "ds_read_b64_tr_b8 v[116:117], v10 offset:24576\n" \
    "ds_read_b64_tr_b8 v[118:119], v10 offset:25600\n" \
    "ds_read_b64_tr_b8 v[120:121], v11 offset:16384\n" \
    "ds_read_b64_tr_b8 v[122:123], v11 offset:17408\n" \
    "ds_read_b64_tr_b8 v[124:125], v11 offset:24576\n" \
    "ds_read_b64_tr_b8 v[126:127], v11 offset:25600\n" \
    "ds_read_b64_tr_b8 v[128:129], v12 offset:16384\n" \
    "ds_read_b64_tr_b8 v[130:131], v12 offset:17408\n" \
    "ds_read_b64_tr_b8 v[132:133], v12 offset:24576\n" \
    "ds_read_b64_tr_b8 v[134:135], v12 offset:25600\n" \
    "ds_read_b64_tr_b8 v[136:137], v13 offset:16384\n" \
    "ds_read_b64_tr_b8 v[138:139], v13 offset:17408\n" \
    "ds_read_b64_tr_b8 v[140:141], v13 offset:24576\n" \
    "ds_read_b64_tr_b8 v[142:143], v13 offset:25600\n" \
    "ds_read_b64_tr_b8 v[144:145], v14 offset:16384\n" \
    "ds_read_b64_tr_b8 v[146:147], v14 offset:17408\n" \
    "ds_read_b64_tr_b8 v[148:149], v14 offset:24576\n" \
    "ds_read_b64_tr_b8 v[150:151], v14 offset:25600\n" \
    "ds_read_b64_tr_b8 v[152:153], v15 offset:16384\n" \
    "ds_read_b64_tr_b8 v[154:155], v15 offset:17408\n" \
    "ds_read_b64_tr_b8 v[156:157], v15 offset:24576\n" \
    "ds_read_b64_tr_b8 v[158:159], v15 offset:25600\n" \
    "ds_read_b128 v[24:27], v16 offset:2048\n" \
    "ds_read_b128 v[28:31], v17 offset:2048\n" \
    "ds_write_b32 v7, v21 offset:512\n" \
    "30:\n" \
    "s_mov_b32 s62, %[sz]\n" \
    "s_add_u32 %[n1], %[c], 1\n" \
    "s_cmp_ge_u32 %[n1], 64\n" \
    "s_cbranch_scc1 10b\n" \
    "s_cmp_eq_u32 %[xi], %[n1]\n" \
    "s_cselect_b32 %[n1], %[n1], 0\n" \
    "60:\n" \
    "ds_read_b32 v21, v19 offset:16\n" \
    "s_waitcnt lgkmcnt(0)\n" \
    "v_readfirstlane_b32 %[rv], v21\n" \
    "s_cmp_le_u32 %[rv], %[n1]\n" \
    "s_cbranch_scc1 60b\n" \
    "s_add_u32 %[n1], %[c], 1\n" \
    "s_cmp_lg_u32 %[xi], %[n1]\n" \
    "s_cbranch_scc1 3301f\n" \
    "s_cmp_le_u32 %[rv], %[xi]\n" \
    "s_cbranch_scc1 80f\n" \
    "s_lshl_b32 %[so], %[xi], 14\n" \
    "buffer_load_dwordx4 v[48:51], v20, s[60:63], %[so] offen offset:0 sc0 sc1\n" \
    "buffer_load_dwordx4 v[52:55], v20, s[60:63], %[so] offen offset:1024 sc0 sc1\n" \
    "buffer_load_dwordx4 v[56:59], v20, s[60:63], %[so] offen offset:2048 sc0 sc1\n" \
    "buffer_load_dwordx4 v[60:63], v20, s[60:63], %[so] offen offset:3072 sc0 sc1\n" \
    "s_mov_b32 s71, %[vc]\n" \
    "s_add_u32 %[vc], %[vc], 1\n" \
    "s_add_u32 %[xi], %[xi], 1\n" \
    "3301:\n" \
    "s_add_u32 %[n1], %[c], 2\n" \
    "s_cmp_lg_u32 %[xi], %[n1]\n" \
    "s_cbranch_scc1 3302f\n" \
    "s_cmp_le_u32 %[rv], %[xi]\n" \
    "s_cbranch_scc1 80f\n" \
    "s_lshl_b32 %[so], %[xi], 14\n" \
    "buffer_load_dwordx4 v[64:67], v20, s[60:63], %[so] offen offset:0 sc0 sc1\n" \
    "buffer_load_dwordx4 v[68:71], v20, s[60:63], %[so] offen offset:1024 sc0 sc1\n" \
    "buffer_load_dwordx4 v[72:75], v20, s[60:63], %[so] offen offset:2048 sc0 sc1\n" \
    "buffer_load_dwordx4 v[76:79], v20, s[60:63], %[so] offen offset:3072 sc0 sc1\n" \
    "s_mov_b32 s72, %[vc]\n" \
    "s_add_u32 %[vc], %[vc], 1\n" \
    "s_add_u32 %[xi], %[xi], 1\n" \
    "3302:\n" \
    "s_add_u32 %[n1], %[c], 3\n" \
    "s_cmp_lg_u32 %[xi], %[n1]\n" \
    "s_cbranch_scc1 3303f\n" \
    "s_cmp_le_u32 %[rv], %[xi]\n" \
    "s_cbranch_scc1 80f\n" \
    "s_lshl_b32 %[so], %[xi], 14\n" \
    "buffer_load_dwordx4 v[80:83], v20, s[60:63], %[so] offen offset:0 sc0 sc1\n" \
    "buffer_load_dwordx4 v[84:87], v20, s[60:63], %[so] offen offset:1024 sc0 sc1\n" \
    "buffer_load_dwordx4 v[88:91], v20, s[60:63], %[so] offen offset:2048 sc0 sc1\n" \
    "buffer_load_dwordx4 v[92:95], v20, s[60:63], %[so] offen offset:3072 sc0 sc1\n" \
    "s_mov_b32 s73, %[vc]\n" \
    "s_add_u32 %[vc], %[vc], 1\n" \
    "s_add_u32 %[xi], %[xi], 1\n" \
    "3303:\n" \
    "80:\n" \
    "v_add_u32 v21, %[sb], v19\n" \
    "s_sub_u32 %[c], %[vc], s71\n" \
    "ds_read_b32 v22, v21 offset:0\n" \
    "ds_read_b32 v23, v19 offset:16\n" \
    "s_waitcnt vmcnt(24)\n" \
    "s_cmp_ge_u32 %[c], 7\n" \
    "s_cbranch_scc1 10b\n" \
    "s_waitcnt vmcnt(12)\n" \
    "s_cmp_ge_u32 %[c], 4\n" \
    "s_cbranch_scc1 10b\n" \
    "s_waitcnt vmcnt(8)\n" \
    "s_cmp_ge_u32 %[c], 3\n" \
    "s_cbranch_scc1 10b\n" \
    "s_waitcnt vmcnt(4)\n" \
    "s_cmp_ge_u32 %[c], 2\n" \
    "s_cbranch_scc1 10b\n" \
    "s_waitcnt vmcnt(0)\n" \
    "s_branch 10b\n" \
    "300:\n" \
    "s_add_u32 %[c], %[u], 0\n" \
    "s_mov_b32 s62, %[sz]\n" \
    "s_waitcnt lgkmcnt(4)\n" \
    "v_readfirstlane_b32 %[rv], v23\n" \
    "s_add_u32 %[n1], %[c], 1\n" \
    "s_cmp_lg_u32 %[xi], %[n1]\n" \
    "s_cbranch_scc1 3201f\n" \
    "s_cmp_le_u32 %[rv], %[xi]\n" \
    "s_cbranch_scc1 310b\n" \
    "s_lshl_b32 %[so], %[xi], 14\n" \
    "buffer_load_dwordx4 v[48:51], v20, s[60:63], %[so] offen offset:0 sc0 sc1\n" \
    "buffer_load_dwordx4 v[52:55], v20, s[60:63], %[so] offen offset:1024 sc0 sc1\n" \
    "buffer_load_dwordx4 v[56:59], v20, s[60:63], %[so] offen offset:2048 sc0 sc1\n" \
    "buffer_load_dwordx4 v[60:63], v20, s[60:63], %[so] offen offset:3072 sc0 sc1\n" \
    "s_mov_b32 s71, %[vc]\n" \
    "s_add_u32 %[vc], %[vc], 1\n" \
    "s_add_u32 %[xi], %[xi], 1\n" \
    "3201:\n" \
    "s_add_u32 %[n1], %[c], 2\n" \
    "s_cmp_lg_u32 %[xi], %[n1]\n" \
    "s_cbranch_scc1 3202f\n" \
    "s_cmp_le_u32 %[rv], %[xi]\n" \
    "s_cbranch_scc1 310b\n" \
    "s_lshl_b32 %[so], %[xi], 14\n" \
    "buffer_load_dwordx4 v[64:67], v20, s[60:63], %[so] offen offset:0 sc0 sc1\n" \
    "buffer_load_dwordx4 v[68:71], v20, s[60:63], %[so] offen offset:1024 sc0 sc1\n" \
    "buffer_load_dwordx4 v[72:75], v20, s[60:63], %[so] offen offset:2048 sc0 sc1\n" \
    "buffer_load_dwordx4 v[76:79], v20, s[60:63], %[so] offen offset:3072 sc0 sc1\n" \
    "s_mov_b32 s72, %[vc]\n" \
    "s_add_u32 %[vc], %[vc], 1\n" \
    "s_add_u32 %[xi], %[xi], 1\n" \
    "3202:\n" \
    "s_add_u32 %[n1], %[c], 3\n" \
    "s_cmp_lg_u32 %[xi], %[n1]\n" \
    "s_cbranch_scc1 3203f\n" \
    "s_cmp_le_u32 %[rv], %[xi]\n" \
    "s_cbranch_scc1 310b\n" \
    "s_lshl_b32 %[so], %[xi], 14\n" \
    "buffer_load_dwordx4 v[80:83], v20, s[60:63], %[so] offen offset:0 sc0 sc1\n" \
    "buffer_load_dwordx4 v[84:87], v20, s[60:63], %[so] offen offset:1024 sc0 sc1\n" \
    "buffer_load_dwordx4 v[88:91], v20, s[60:63], %[so] offen offset:2048 sc0 sc1\n" \
    "buffer_load_dwordx4 v[92:95], v20, s[60:63], %[so] offen offset:3072 sc0 sc1\n" \
    "s_mov_b32 s73, %[vc]\n" \
    "s_add_u32 %[vc], %[vc], 1\n" \
    "s_add_u32 %[xi], %[xi], 1\n" \
    "3203:\n" \
    "s_branch 310b\n" \
    "21:\n" \
    "s_add_u32 %[c], %[u], 1\n" \
    "s_cbranch_vccz 31f\n" \
    "v_add_u32 v7, %[sa], v18\n" \
    "s_waitcnt lgkmcnt(0)\n" \
    "v_add_u32 v23, %[sa], v19\n" \
    "41:\n" \
    "ds_read_b32 v21, v23 offset:0\n" \
    "s_waitcnt lgkmcnt(0)\n" \
    "v_cmp_ge_u32 vcc, %[thr], v21\n" \
    "s_cbranch_vccnz 41b\n" \
    "ds_read_b64_tr_b8 v[96:97], v8 offset:0\n" \
    "ds_read_b64_tr_b8 v[98:99], v8 offset:1024\n" \
    "ds_read_b64_tr_b8 v[100:101], v8 offset:8192\n" \
    "ds_read_b64_tr_b8 v[102:103], v8 offset:9216\n" \
    "ds_read_b64_tr_b8 v[104:105], v9 offset:0\n" \
    "ds_read_b64_tr_b8 v[106:107], v9 offset:1024\n" \
    "ds_read_b64_tr_b8 v[108:109], v9 offset:8192\n" \
    "ds_read_b64_tr_b8 v[110:111], v9 offset:9216\n" \
    "ds_read_b64_tr_b8 v[112:113], v10 offset:0\n" \
    "ds_read_b64_tr_b8 v[114:115], v10 offset:1024\n" \
    "ds_read_b64_tr_b8 v[116:117], v10 offset:8192\n" \
    "ds_read_b64_tr_b8 v[118:119], v10 offset:9216\n" \
    "ds_read_b64_tr_b8 v[120:121], v11 offset:0\n" \
    "ds_read_b64_tr_b8 v[122:123], v11 offset:1024\n" \
    "ds_read_b64_tr_b8 v[124:125], v11 offset:8192\n" \
    "ds_read_b64_tr_b8 v[126:127], v11 offset:9216\n" \
    "ds_read_b64_tr_b8 v[128:129], v12 offset:0\n" \
    "ds_read_b64_tr_b8 v[130:131], v12 offset:1024\n" \
    "ds_read_b64_tr_b8 v[132:133], v12 offset:8192\n" \
    "ds_read_b64_tr_b8 v[134:135], v12 offset:9216\n" \
    "ds_read_b64_tr_b8 v[136:137], v13 offset:0\n" \
    "ds_read_b64_tr_b8 v[138:139], v13 offset:1024\n" \
    "ds_read_b64_tr_b8 v[140:141], v13 offset:8192\n" \
    "ds_read_b64_tr_b8 v[142:143], v13 offset:9216\n" \
    "ds_read_b64_tr_b8 v[144:145], v14 offset:0\n" \
    "ds_read_b64_tr_b8 v[146:147], v14 offset:1024\n" \
    "ds_read_b64_tr_b8 v[148:149], v14 offset:8192\n" \
    "ds_read_b64_tr_b8 v[150:151], v14 offset:9216\n" \
    "ds_read_b64_tr_b8 v[152:153], v15 offset:0\n" \
    "ds_read_b64_tr_b8 v[154:155], v15 offset:1024\n" \
    "ds_read_b64_tr_b8 v[156:157], v15 offset:8192\n" \
    "ds_read_b64_tr_b8 v[158:159], v15 offset:9216\n" \
    "ds_read_b128 v[24:27], v16 offset:4096\n" \
    "ds_read_b128 v[28:31], v17 offset:4096\n" \
    "ds_write_b32 v7, v21 offset:0\n" \
    "31:\n" \
    "s_mov_b32 s62, %[sz]\n" \
    "s_add_u32 %[n1], %[c], 1\n" \
    "s_cmp_ge_u32 %[n1], 64\n" \
    "s_cbranch_scc1 11b\n" \
    "s_cmp_eq_u32 %[xi], %[n1]\n" \
    "s_cselect_b32 %[n1], %[n1], 0\n" \
    "61:\n" \
    "ds_read_b32 v21, v19 offset:16\n" \
    "s_waitcnt lgkmcnt(0)\n" \
    "v_readfirstlane_b32 %[rv], v21\n" \
    "s_cmp_le_u32 %[rv], %[n1]\n" \
    "s_cbranch_scc1 61b\n" \
    "s_add_u32 %[n1], %[c], 1\n" \
    "s_cmp_lg_u32 %[xi], %[n1]\n" \
    "s_cbranch_scc1 3311f\n" \
    "s_cmp_le_u32 %[rv], %[xi]\n" \
    "s_cbranch_scc1 81f\n" \
    "s_lshl_b32 %[so], %[xi], 14\n" \
    "buffer_load_dwordx4 v[64:67], v20, s[60:63], %[so] offen offset:0 sc0 sc1\n" \
    "buffer_load_dwordx4 v[68:71], v20, s[60:63], %[so] offen offset:1024 sc0 sc1\n" \
    "buffer_load_dwordx4 v[72:75], v20, s[60:63], %[so] offen offset:2048 sc0 sc1\n" \
    "buffer_load_dwordx4 v[76:79], v20, s[60:63], %[so] offen offset:3072 sc0 sc1\n" \
    "s_mov_b32 s72, %[vc]\n" \
    "s_add_u32 %[vc], %[vc], 1\n" \
    "s_add_u32 %[xi], %[xi], 1\n" \
    "3311:\n" \
    "s_add_u32 %[n1], %[c], 2\n" \
    "s_cmp_lg_u32 %[xi], %[n1]\n" \
    "s_cbranch_scc1 3312f\n" \
    "s_cmp_le_u32 %[rv], %[xi]\n" \
    "s_cbranch_scc1 81f\n" \
    "s_lshl_b32 %[so], %[xi], 14\n" \
    "buffer_load_dwordx4 v[80:83], v20, s[60:63], %[so] offen offset:0 sc0 sc1\n" \
    "buffer_load_dwordx4 v[84:87], v20, s[60:63], %[so] offen offset:1024 sc0 sc1\n" \
    "buffer_load_dwordx4 v[88:91], v20, s[60:63], %[so] offen offset:2048 sc0 sc1\n" \
    "buffer_load_dwordx4 v[92:95], v20, s[60:63], %[so] offen offset:3072 sc0 sc1\n" \
    "s_mov_b32 s73, %[vc]\n" \
    "s_add_u32 %[vc], %[vc], 1\n" \
    "s_add_u32 %[xi], %[xi], 1\n" \
    "3312:\n" \
    "s_add_u32 %[n1], %[c], 3\n" \
    "s_cmp_lg_u32 %[xi], %[n1]\n" \
    "s_cbranch_scc1 3313f\n" \
    "s_cmp_le_u32 %[rv], %[xi]\n" \
    "s_cbranch_scc1 81f\n" \
    "s_lshl_b32 %[so], %[xi], 14\n" \
    "buffer_load_dwordx4 v[32:35], v20, s[60:63], %[so] offen offset:0 sc0 sc1\n" \
    "buffer_load_dwordx4 v[36:39], v20, s[60:63], %[so] offen offset:1024 sc0 sc1\n" \
    "buffer_load_dwordx4 v[40:43], v20, s[60:63], %[so] offen offset:2048 sc0 sc1\n" \
    "buffer_load_dwordx4 v[44:47], v20, s[60:63], %[so] offen offset:3072 sc0 sc1\n" \
    "s_mov_b32 s70, %[vc]\n" \
    "s_add_u32 %[vc], %[vc], 1\n" \
    "s_add_u32 %[xi], %[xi], 1\n" \
    "3313:\n" \
    "81:\n" \
    "v_add_u32 v21, %[sa], v19\n" \
    "s_sub_u32 %[c], %[vc], s72\n" \
    "ds_read_b32 v22, v21 offset:512\n" \
    "ds_read_b32 v23, v19 offset:16\n" \
    "s_waitcnt vmcnt(24)\n" \
    "s_cmp_ge_u32 %[c], 7\n" \
    "s_cbranch_scc1 11b\n" \
    "s_waitcnt vmcnt(12)\n" \
    "s_cmp_ge_u32 %[c], 4\n" \
    "s_cbranch_scc1 11b\n" \
    "s_waitcnt vmcnt(8)\n" \
    "s_cmp_ge_u32 %[c], 3\n" \
    "s_cbranch_scc1 11b\n" \
    "s_waitcnt vmcnt(4)\n" \
    "s_cmp_ge_u32 %[c], 2\n" \
    "s_cbranch_scc1 11b\n" \
    "s_waitcnt vmcnt(0)\n" \
    "s_branch 11b\n" \
    "301:\n" \
    "s_add_u32 %[c], %[u], 1\n" \
    "s_mov_b32 s62, %[sz]\n" \
    "s_waitcnt lgkmcnt(4)\n" \
    "v_readfirstlane_b32 %[rv], v23\n" \
    "s_add_u32 %[n1], %[c], 1\n" \
    "s_cmp_lg_u32 %[xi], %[n1]\n" \
    "s_cbranch_scc1 3211f\n" \
    "s_cmp_le_u32 %[rv], %[xi]\n" \
    "s_cbranch_scc1 311b\n" \
    "s_lshl_b32 %[so], %[xi], 14\n" \
    "buffer_load_dwordx4 v[64:67], v20, s[60:63], %[so] offen offset:0 sc0 sc1\n" \
    "buffer_load_dwordx4 v[68:71], v20, s[60:63], %[so] offen offset:1024 sc0 sc1\n" \
    "buffer_load_dwordx4 v[72:75], v20, s[60:63], %[so] offen offset:2048 sc0 sc1\n" \
    "buffer_load_dwordx4 v[76:79], v20, s[60:63], %[so] offen offset:3072 sc0 sc1\n" \
    "s_mov_b32 s72, %[vc]\n" \
    "s_add_u32 %[vc], %[vc], 1\n" \
    "s_add_u32 %[xi], %[xi], 1\n" \
    "3211:\n" \
    "s_add_u32 %[n1], %[c], 2\n" \
    "s_cmp_lg_u32 %[xi], %[n1]\n" \
    "s_cbranch_scc1 3212f\n" \
    "s_cmp_le_u32 %[rv], %[xi]\n" \
    "s_cbranch_scc1 311b\n" \
    "s_lshl_b32 %[so], %[xi], 14\n" \
    "buffer_load_dwordx4 v[80:83], v20, s[60:63], %[so] offen offset:0 sc0 sc1\n" \
    "buffer_load_dwordx4 v[84:87], v20, s[60:63], %[so] offen offset:1024 sc0 sc1\n" \
    "buffer_load_dwordx4 v[88:91], v20, s[60:63], %[so] offen offset:2048 sc0 sc1\n" \
    "buffer_load_dwordx4 v[92:95], v20, s[60:63], %[so] offen offset:3072 sc0 sc1\n" \
    "s_mov_b32 s73, %[vc]\n" \
    "s_add_u32 %[vc], %[vc], 1\n" \
    "s_add_u32 %[xi], %[xi], 1\n" \
    "3212:\n" \
    "s_add_u32 %[n1], %[c], 3\n" \
    "s_cmp_lg_u32 %[xi], %[n1]\n" \
    "s_cbranch_scc1 3213f\n" \
    "s_cmp_le_u32 %[rv], %[xi]\n" \
    "s_cbranch_scc1 311b\n" \
    "s_lshl_b32 %[so], %[xi], 14\n" \
    "buffer_load_dwordx4 v[32:35], v20, s[60:63], %[so] offen offset:0 sc0 sc1\n" \
    "buffer_load_dwordx4 v[36:39], v20, s[60:63], %[so] offen offset:1024 sc0 sc1\n" \
    "buffer_load_dwordx4 v[40:43], v20, s[60:63], %[so] offen offset:2048 sc0 sc1\n" \
    "buffer_load_dwordx4 v[44:47], v20, s[60:63], %[so] offen offset:3072 sc0 sc1\n" \
    "s_mov_b32 s70, %[vc]\n" \
    "s_add_u32 %[vc], %[vc], 1\n" \
    "s_add_u32 %[xi], %[xi], 1\n" \
    "3213:\n" \
    "s_branch 311b\n" \
    "22:\n" \
    "s_add_u32 %[c], %[u], 2\n" \
    "s_cbranch_vccz 32f\n" \
    "v_add_u32 v7, %[sa], v18\n" \
    "s_waitcnt lgkmcnt(0)\n" \
    "v_add_u32 v23, %[sa], v19\n" \
    "42:\n" \
    "ds_read_b32 v21, v23 offset:512\n" \
    "s_waitcnt lgkmcnt(0)\n" \
    "v_cmp_ge_u32 vcc, %[thr], v21\n" \
    "s_cbranch_vccnz 42b\n" \
    "ds_read_b64_tr_b8 v[96:97], v8 offset:16384\n" \
    "ds_read_b64_tr_b8 v[98:99], v8 offset:17408\n" \
    "ds_read_b64_tr_b8 v[100:101], v8 offset:24576\n" \
    "ds_read_b64_tr_b8 v[102:103], v8 offset:25600\n" \
    "ds_read_b64_tr_b8 v[104:105], v9 offset:16384\n" \
    "ds_read_b64_tr_b8 v[106:107], v9 offset:17408\n" \
    "ds_read_b64_tr_b8 v[108:109], v9 offset:24576\n" \
    "ds_read_b64_tr_b8 v[110:111], v9 offset:25600\n" \
    "ds_read_b64_tr_b8 v[112:113], v10 offset:16384\n" \
    "ds_read_b64_tr_b8 v[114:115], v10 offset:17408\n" \
    "ds_read_b64_tr_b8 v[116:117], v10 offset:24576\n" \
    "ds_read_b64_tr_b8 v[118:119], v10 offset:25600\n" \
    "ds_read_b64_tr_b8 v[120:121], v11 offset:16384\n" \
    "ds_read_b64_tr_b8 v[122:123], v11 offset:17408\n" \
    "ds_read_b64_tr_b8 v[124:125], v11 offset:24576\n" \
    "ds_read_b64_tr_b8 v[126:127], v11 offset:25600\n" \
    "ds_read_b64_tr_b8 v[128:129], v12 offset:16384\n" \
    "ds_read_b64_tr_b8 v[130:131], v12 offset:17408\n" \
    "ds_read_b64_tr_b8 v[132:133], v12 offset:24576\n" \
    "ds_read_b64_tr_b8 v[134:135], v12 offset:25600\n" \
    "ds_read_b64_tr_b8 v[136:137], v13 offset:16384\n" \
    "ds_read_b64_tr_b8 v[138:139], v13 offset:17408\n" \
    "ds_read_b64_tr_b8 v[140:141], v13 offset:24576\n" \
    "ds_read_b64_tr_b8 v[142:143], v13 offset:25600\n" \
    "ds_read_b64_tr_b8 v[144:145], v14 offset:16384\n" \
    "ds_read_b64_tr_b8 v[146:147], v14 offset:17408\n" \
    "ds_read_b64_tr_b8 v[148:149], v14 offset:24576\n" \
    "ds_read_b64_tr_b8 v[150:151], v14 offset:25600\n" \
    "ds_read_b64_tr_b8 v[152:153], v15 offset:16384\n" \
    "ds_read_b64_tr_b8 v[154:155], v15 offset:17408\n" \
    "ds_read_b64_tr_b8 v[156:157], v15 offset:24576\n" \
    "ds_read_b64_tr_b8 v[158:159], v15 offset:25600\n" \
    "ds_read_b128 v[24:27], v16 offset:6144\n" \
    "ds_read_b128 v[28:31], v17 offset:6144\n" \
    "ds_write_b32 v7, v21 offset:512\n" \
    "32:\n" \
    "s_mov_b32 s62, %[sz]\n" \
    "s_add_u32 %[n1], %[c], 1\n" \
    "s_cmp_ge_u32 %[n1], 64\n" \
    "s_cbranch_scc1 12b\n" \
    "s_cmp_eq_u32 %[xi], %[n1]\n" \
    "s_cselect_b32 %[n1], %[n1], 0\n" \
    "62:\n" \
    "ds_read_b32 v21, v19 offset:16\n" \
    "s_waitcnt lgkmcnt(0)\n" \
    "v_readfirstlane_b32 %[rv], v21\n" \
    "s_cmp_le_u32 %[rv], %[n1]\n" \
    "s_cbranch_scc1 62b\n" \
    "s_add_u32 %[n1], %[c], 1\n" \
    "s_cmp_lg_u32 %[xi], %[n1]\n" \
    "s_cbranch_scc1 3321f\n" \
    "s_cmp_le_u32 %[rv], %[xi]\n" \
    "s_cbranch_scc1 82f\n" \
    "s_lshl_b32 %[so], %[xi], 14\n" \
    "buffer_load_dwordx4 v[80:83], v20, s[60:63], %[so] offen offset:0 sc0 sc1\n" \
    "buffer_load_dwordx4 v[84:87], v20, s[60:63], %[so] offen offset:1024 sc0 sc1\n" \
    "buffer_load_dwordx4 v[88:91], v20, s[60:63], %[so] offen offset:2048 sc0 sc1\n" \
    "buffer_load_dwordx4 v[92:95], v20, s[60:63], %[so] offen offset:3072 sc0 sc1\n" \
    "s_mov_b32 s73, %[vc]\n" \
    "s_add_u32 %[vc], %[vc], 1\n" \
    "s_add_u32 %[xi], %[xi], 1\n" \
    "3321:\n" \
    "s_add_u32 %[n1], %[c], 2\n" \
    "s_cmp_lg_u32 %[xi], %[n1]\n" \
    "s_cbranch_scc1 3322f\n" \
    "s_cmp_le_u32 %[rv], %[xi]\n" \
    "s_cbranch_scc1 82f\n" \
    "s_lshl_b32 %[so], %[xi], 14\n" \
    "buffer_load_dwordx4 v[32:35], v20, s[60:63], %[so] offen offset:0 sc0 sc1\n" \
    "buffer_load_dwordx4 v[36:39], v20, s[60:63], %[so] offen offset:1024 sc0 sc1\n" \
    "buffer_load_dwordx4 v[40:43], v20, s[60:63], %[so] offen offset:2048 sc0 sc1\n" \
    "buffer_load_dwordx4 v[44:47], v20, s[60:63], %[so] offen offset:3072 sc0 sc1\n" \
    "s_mov_b32 s70, %[vc]\n" \
    "s_add_u32 %[vc], %[vc], 1\n" \
    "s_add_u32 %[xi], %[xi], 1\n" \
    "3322:\n" \
    "s_add_u32 %[n1], %[c], 3\n" \
    "s_cmp_lg_u32 %[xi], %[n1]\n" \
    "s_cbranch_scc1 3323f\n" \
    "s_cmp_le_u32 %[rv], %[xi]\n" \
    "s_cbranch_scc1 82f\n" \
    "s_lshl_b32 %[so], %[xi], 14\n" \
    "buffer_load_dwordx4 v[48:51], v20, s[60:63], %[so] offen offset:0 sc0 sc1\n" \
    "buffer_load_dwordx4 v[52:55], v20, s[60:63], %[so] offen offset:1024 sc0 sc1\n" \
    "buffer_load_dwordx4 v[56:59], v20, s[60:63], %[so] offen offset:2048 sc0 sc1\n" \
    "buffer_load_dwordx4 v[60:63], v20, s[60:63], %[so] offen offset:3072 sc0 sc1\n" \
    "s_mov_b32 s71, %[vc]\n" \
    "s_add_u32 %[vc], %[vc], 1\n" \
    "s_add_u32 %[xi], %[xi], 1\n" \
    "3323:\n" \
    "82:\n" \
    "v_add_u32 v21, %[sb], v19\n" \
    "s_sub_u32 %[c], %[vc], s73\n" \
    "ds_read_b32 v22, v21 offset:0\n" \
    "ds_read_b32 v23, v19 offset:16\n" \
    "s_waitcnt vmcnt(24)\n" \
    "s_cmp_ge_u32 %[c], 7\n" \
    "s_cbranch_scc1 12b\n" \
    "s_waitcnt vmcnt(12)\n" \
    "s_cmp_ge_u32 %[c], 4\n" \
    "s_cbranch_scc1 12b\n" \
    "s_waitcnt vmcnt(8)\n" \
    "s_cmp_ge_u32 %[c], 3\n" \
    "s_cbranch_scc1 12b\n" \
    "s_waitcnt vmcnt(4)\n" \
    "s_cmp_ge_u32 %[c], 2\n" \
    "s_cbranch_scc1 12b\n" \
    "s_waitcnt vmcnt(0)\n" \
    "s_branch 12b\n" \
    "302:\n" \
    "s_add_u32 %[c], %[u], 2\n" \
    "s_mov_b32 s62, %[sz]\n" \
    "s_waitcnt lgkmcnt(4)\n" \
    "v_readfirstlane_b32 %[rv], v23\n" \
    "s_add_u32 %[n1], %[c], 1\n" \
    "s_cmp_lg_u32 %[xi], %[n1]\n" \
    "s_cbranch_scc1 3221f\n" \
    "s_cmp_le_u32 %[rv], %[xi]\n" \
    "s_cbranch_scc1 312b\n" \
    "s_lshl_b32 %[so], %[xi], 14\n" \
    "buffer_load_dwordx4 v[80:83], v20, s[60:63], %[so] offen offset:0 sc0 sc1\n" \
    "buffer_load_dwordx4 v[84:87], v20, s[60:63], %[so] offen offset:1024 sc0 sc1\n" \
    "buffer_load_dwordx4 v[88:91], v20, s[60:63], %[so] offen offset:2048 sc0 sc1\n" \
    "buffer_load_dwordx4 v[92:95], v20, s[60:63], %[so] offen offset:3072 sc0 sc1\n" \
    "s_mov_b32 s73, %[vc]\n" \
    "s_add_u32 %[vc], %[vc], 1\n" \
    "s_add_u32 %[xi], %[xi], 1\n" \
    "3221:\n" \
    "s_add_u32 %[n1], %[c], 2\n" \
    "s_cmp_lg_u32 %[xi], %[n1]\n" \
    "s_cbranch_scc1 3222f\n" \
    "s_cmp_le_u32 %[rv], %[xi]\n" \
    "s_cbranch_scc1 312b\n" \
    "s_lshl_b32 %[so], %[xi], 14\n" \
    "buffer_load_dwordx4 v[32:35], v20, s[60:63], %[so] offen offset:0 sc0 sc1\n" \
    "buffer_load_dwordx4 v[36:39], v20, s[60:63], %[so] offen offset:1024 sc0 sc1\n" \
    "buffer_load_dwordx4 v[40:43], v20, s[60:63], %[so] offen offset:2048 sc0 sc1\n" \
    "buffer_load_dwordx4 v[44:47], v20, s[60:63], %[so] offen offset:3072 sc0 sc1\n" \
    "s_mov_b32 s70, %[vc]\n" \
    "s_add_u32 %[vc], %[vc], 1\n" \
    "s_add_u32 %[xi], %[xi], 1\n" \
    "3222:\n" \
    "s_add_u32 %[n1], %[c], 3\n" \
    "s_cmp_lg_u32 %[xi], %[n1]\n" \
    "s_cbranch_scc1 3223f\n" \
    "s_cmp_le_u32 %[rv], %[xi]\n" \
    "s_cbranch_scc1 312b\n" \
    "s_lshl_b32 %[so], %[xi], 14\n" \
    "buffer_load_dwordx4 v[48:51], v20, s[60:63], %[so] offen offset:0 sc0 sc1\n" \
    "buffer_load_dwordx4 v[52:55], v20, s[60:63], %[so] offen offset:1024 sc0 sc1\n" \
    "buffer_load_dwordx4 v[56:59], v20, s[60:63], %[so] offen offset:2048 sc0 sc1\n" \
    "buffer_load_dwordx4 v[60:63], v20, s[60:63], %[so] offen offset:3072 sc0 sc1\n" \
    "s_mov_b32 s71, %[vc]\n" \
    "s_add_u32 %[vc], %[vc], 1\n" \
    "s_add_u32 %[xi], %[xi], 1\n" \
    "3223:\n" \
    "s_branch 312b\n" \
    "23:\n" \
    "s_add_u32 %[c], %[u], 3\n" \
    "s_cbranch_vccz 33f\n" \
    "v_add_u32 v7, %[sa], v18\n" \
    "s_waitcnt lgkmcnt(0)\n" \
    "v_add_u32 v23, %[sa], v19\n" \
    "43:\n" \
    "ds_read_b32 v21, v23 offset:0\n" \
    "s_waitcnt lgkmcnt(0)\n" \
    "v_cmp_ge_u32 vcc, %[thr], v21\n" \
    "s_cbranch_vccnz 43b\n" \
    "ds_read_b64_tr_b8 v[96:97], v8 offset:0\n" \
    "ds_read_b64_tr_b8 v[98:99], v8 offset:1024\n" \
    "ds_read_b64_tr_b8 v[100:101], v8 offset:8192\n" \
    "ds_read_b64_tr_b8 v[102:103], v8 offset:9216\n" \
    "ds_read_b64_tr_b8 v[104:105], v9 offset:0\n" \
    "ds_read_b64_tr_b8 v[106:107], v9 offset:1024\n" \
    "ds_read_b64_tr_b8 v[108:109], v9 offset:8192\n" \
    "ds_read_b64_tr_b8 v[110:111], v9 offset:9216\n" \
    "ds_read_b64_tr_b8 v[112:113], v10 offset:0\n" \
    "ds_read_b64_tr_b8 v[114:115], v10 offset:1024\n" \
    "ds_read_b64_tr_b8 v[116:117], v10 offset:8192\n" \
    "ds_read_b64_tr_b8 v[118:119], v10 offset:9216\n" \
    "ds_read_b64_tr_b8 v[120:121], v11 offset:0\n" \
    "ds_read_b64_tr_b8 v[122:123], v11 offset:1024\n" \
    "ds_read_b64_tr_b8 v[124:125], v11 offset:8192\n" \
    "ds_read_b64_tr_b8 v[126:127], v11 offset:9216\n" \
    "ds_read_b64_tr_b8 v[128:129], v12 offset:0\n" \
    "ds_read_b64_tr_b8 v[130:131], v12 offset:1024\n" \
    "ds_read_b64_tr_b8 v[132:133], v12 offset:8192\n" \
    "ds_read_b64_tr_b8 v[134:135], v12 offset:9216\n" \
    "ds_read_b64_tr_b8 v[136:137], v13 offset:0\n" \
    "ds_read_b64_tr_b8 v[138:139], v13 offset:1024\n" \
    "ds_read_b64_tr_b8 v[140:141], v13 offset:8192\n" \
    "ds_read_b64_tr_b8 v[142:143], v13 offset:9216\n" \
    "ds_read_b64_tr_b8 v[144:145], v14 offset:0\n" \
    "ds_read_b64_tr_b8 v[146:147], v14 offset:1024\n" \
    "ds_read_b64_tr_b8 v[148:149], v14 offset:8192\n" \
    "ds_read_b64_tr_b8 v[150:151], v14 offset:9216\n" \
    "ds_read_b64_tr_b8 v[152:153], v15 offset:0\n" \
    "ds_read_b64_tr_b8 v[154:155], v15 offset:1024\n" \
    "ds_read_b64_tr_b8 v[156:157], v15 offset:8192\n" \
    "ds_read_b64_tr_b8 v[158:159], v15 offset:9216\n" \
    "ds_read_b128 v[24:27], v16 offset:0\n" \
    "ds_read_b128 v[28:31], v17 offset:0\n" \
    "ds_write_b32 v7, v21 offset:0\n" \
    "33:\n" \
    "s_mov_b32 s62, %[sz]\n" \
    "s_add_u32 %[n1], %[c], 1\n" \
    "s_cmp_ge_u32 %[n1], 64\n" \
    "s_cbranch_scc1 13b\n" \
    "s_cmp_eq_u32 %[xi], %[n1]\n" \
    "s_cselect_b32 %[n1], %[n1], 0\n" \
    "63:\n" \
    "ds_read_b32 v21, v19 offset:16\n" \
    "s_waitcnt lgkmcnt(0)\n" \
    "v_readfirstlane_b32 %[rv], v21\n" \
    "s_cmp_le_u32 %[rv], %[n1]\n" \
    "s_cbranch_scc1 63b\n" \
    "s_add_u32 %[n1], %[c], 1\n" \
    "s_cmp_lg_u32 %[xi], %[n1]\n" \
    "s_cbranch_scc1 3331f\n" \
    "s_cmp_le_u32 %[rv], %[xi]\n" \
    "s_cbranch_scc1 83f\n" \
    "s_lshl_b32 %[so], %[xi], 14\n" \
    "buffer_load_dwordx4 v[32:35], v20, s[60:63], %[so] offen offset:0 sc0 sc1\n" \
    "buffer_load_dwordx4 v[36:39], v20, s[60:63], %[so] offen offset:1024 sc0 sc1\n" \
    "buffer_load_dwordx4 v[40:43], v20, s[60:63], %[so] offen offset:2048 sc0 sc1\n" \
    "buffer_load_dwordx4 v[44:47], v20, s[60:63], %[so] offen offset:3072 sc0 sc1\n" \
    "s_mov_b32 s70, %[vc]\n" \
    "s_add_u32 %[vc], %[vc], 1\n" \
    "s_add_u32 %[xi], %[xi], 1\n" \
    "3331:\n" \
    "s_add_u32 %[n1], %[c], 2\n" \
    "s_cmp_lg_u32 %[xi], %[n1]\n" \
    "s_cbranch_scc1 3332f\n" \
    "s_cmp_le_u32 %[rv], %[xi]\n" \
    "s_cbranch_scc1 83f\n" \
    "s_lshl_b32 %[so], %[xi], 14\n" \
    "buffer_load_dwordx4 v[48:51], v20, s[60:63], %[so] offen offset:0 sc0 sc1\n" \
    "buffer_load_dwordx4 v[52:55], v20, s[60:63], %[so] offen offset:1024 sc0 sc1\n" \
    "buffer_load_dwordx4 v[56:59], v20, s[60:63], %[so] offen offset:2048 sc0 sc1\n" \
    "buffer_load_dwordx4 v[60:63], v20, s[60:63], %[so] offen offset:3072 sc0 sc1\n" \
    "s_mov_b32 s71, %[vc]\n" \
    "s_add_u32 %[vc], %[vc], 1\n" \
    "s_add_u32 %[xi], %[xi], 1\n" \
    "3332:\n" \
    "s_add_u32 %[n1], %[c], 3\n" \
    "s_cmp_lg_u32 %[xi], %[n1]\n" \
    "s_cbranch_scc1 3333f\n" \
    "s_cmp_le_u32 %[rv], %[xi]\n" \
    "s_cbranch_scc1 83f\n" \
    "s_lshl_b32 %[so], %[xi], 14\n" \
    "buffer_load_dwordx4 v[64:67], v20, s[60:63], %[so] offen offset:0 sc0 sc1\n" \
    "buffer_load_dwordx4 v[68:71], v20, s[60:63], %[so] offen offset:1024 sc0 sc1\n" \
    "buffer_load_dwordx4 v[72:75], v20, s[60:63], %[so] offen offset:2048 sc0 sc1\n" \
    "buffer_load_dwordx4 v[76:79], v20, s[60:63], %[so] offen offset:3072 sc0 sc1\n" \
    "s_mov_b32 s72, %[vc]\n" \
    "s_add_u32 %[vc], %[vc], 1\n" \
    "s_add_u32 %[xi], %[xi], 1\n" \
    "3333:\n" \
    "83:\n" \
    "v_add_u32 v21, %[sa], v19\n" \
    "s_sub_u32 %[c], %[vc], s70\n" \
    "ds_read_b32 v22, v21 offset:512\n" \
    "ds_read_b32 v23, v19 offset:16\n" \
    "s_waitcnt vmcnt(24)\n" \
    "s_cmp_ge_u32 %[c], 7\n" \
    "s_cbranch_scc1 13b\n" \
    "s_waitcnt vmcnt(12)\n" \
    "s_cmp_ge_u32 %[c], 4\n" \
    "s_cbranch_scc1 13b\n" \
    "s_waitcnt vmcnt(8)\n" \
    "s_cmp_ge_u32 %[c], 3\n" \
    "s_cbranch_scc1 13b\n" \
    "s_waitcnt vmcnt(4)\n" \
    "s_cmp_ge_u32 %[c], 2\n" \
    "s_cbranch_scc1 13b\n" \
    "s_waitcnt vmcnt(0)\n" \
    "s_branch 13b\n" \
    "303:\n" \
    "s_add_u32 %[c], %[u], 3\n" \
    "s_mov_b32 s62, %[sz]\n" \
    "s_waitcnt lgkmcnt(4)\n" \
    "v_readfirstlane_b32 %[rv], v23\n" \
    "s_add_u32 %[n1], %[c], 1\n" \
    "s_cmp_lg_u32 %[xi], %[n1]\n" \
    "s_cbranch_scc1 3231f\n" \
    "s_cmp_le_u32 %[rv], %[xi]\n" \
    "s_cbranch_scc1 313b\n" \
    "s_lshl_b32 %[so], %[xi], 14\n" \
    "buffer_load_dwordx4 v[32:35], v20, s[60:63], %[so] offen offset:0 sc0 sc1\n" \
    "buffer_load_dwordx4 v[36:39], v20, s[60:63], %[so] offen offset:1024 sc0 sc1\n" \
    "buffer_load_dwordx4 v[40:43], v20, s[60:63], %[so] offen offset:2048 sc0 sc1\n" \
    "buffer_load_dwordx4 v[44:47], v20, s[60:63], %[so] offen offset:3072 sc0 sc1\n" \
    "s_mov_b32 s70, %[vc]\n" \
    "s_add_u32 %[vc], %[vc], 1\n" \
    "s_add_u32 %[xi], %[xi], 1\n" \
    "3231:\n" \
    "s_add_u32 %[n1], %[c], 2\n" \
    "s_cmp_lg_u32 %[xi], %[n1]\n" \
    "s_cbranch_scc1 3232f\n" \
    "s_cmp_le_u32 %[rv], %[xi]\n" \
    "s_cbranch_scc1 313b\n" \
    "s_lshl_b32 %[so], %[xi], 14\n" \
    "buffer_load_dwordx4 v[48:51], v20, s[60:63], %[so] offen offset:0 sc0 sc1\n" \
    "buffer_load_dwordx4 v[52:55], v20, s[60:63], %[so] offen offset:1024 sc0 sc1\n" \
    "buffer_load_dwordx4 v[56:59], v20, s[60:63], %[so] offen offset:2048 sc0 sc1\n" \
    "buffer_load_dwordx4 v[60:63], v20, s[60:63], %[so] offen offset:3072 sc0 sc1\n" \
    "s_mov_b32 s71, %[vc]\n" \
    "s_add_u32 %[vc], %[vc], 1\n" \
    "s_add_u32 %[xi], %[xi], 1\n" \
    "3232:\n" \
    "s_add_u32 %[n1], %[c], 3\n" \
    "s_cmp_lg_u32 %[xi], %[n1]\n" \
    "s_cbranch_scc1 3233f\n" \
    "s_cmp_le_u32 %[rv], %[xi]\n" \
    "s_cbranch_scc1 313b\n" \
    "s_lshl_b32 %[so], %[xi], 14\n" \
    "buffer_load_dwordx4 v[64:67], v20, s[60:63], %[so] offen offset:0 sc0 sc1\n" \
    "buffer_load_dwordx4 v[68:71], v20, s[60:63], %[so] offen offset:1024 sc0 sc1\n" \
    "buffer_load_dwordx4 v[72:75], v20, s[60:63], %[so] offen offset:2048 sc0 sc1\n" \
    "buffer_load_dwordx4 v[76:79], v20, s[60:63], %[so] offen offset:3072 sc0 sc1\n" \
    "s_mov_b32 s72, %[vc]\n" \
    "s_add_u32 %[vc], %[vc], 1\n" \
    "s_add_u32 %[xi], %[xi], 1\n" \
    "3233:\n" \
    "s_branch 313b\n" \
    "9:\n" \
    "s_branch 99f\n" \
    "98:\n" \
    "s_cmp_lt_i32 %[u], 0\n" \
    "s_cbranch_scc0 0f\n" \
    "s_mov_b32 %[thr], 0\n" \
    "s_mov_b64 vcc, -1\n" \
    "s_branch 23f\n" \
    "600:\n" \
    "s_waitcnt lgkmcnt(14)\n" \
    "v_mfma_f32_16x16x128_f8f6f4 v[160:163], v[96:103], v[32:39], v[160:163] cbsz:1 blgp:1\n" \
    "s_add_u32 %[thr], %[u], 1\n" \
    "s_add_u32 %[n1], %[u], 3\n" \
    "v_mfma_f32_16x16x128_f8f6f4 v[192:195], v[96:103], v[40:47], v[192:195] cbsz:1 blgp:1\n" \
    "ds_read_b64_tr_b8 v[96:97], v8 offset:16384\n" \
    "ds_read_b64_tr_b8 v[98:99], v8 offset:17408\n" \
    "s_lshl_b32 %[so], %[n1], 14\n" \
    "v_mfma_f32_16x16x128_f8f6f4 v[164:167], v[104:111], v[32:39], v[164:167] cbsz:1 blgp:1\n" \
    "ds_read_b64_tr_b8 v[100:101], v8 offset:24576\n" \
    "ds_read_b64_tr_b8 v[102:103], v8 offset:25600\n" \
    "v_mfma_f32_16x16x128_f8f6f4 v[196:199], v[104:111], v[40:47], v[196:199] cbsz:1 blgp:1\n" \
    "ds_read_b64_tr_b8 v[104:105], v9 offset:16384\n" \
    "ds_read_b64_tr_b8 v[106:107], v9 offset:17408\n" \
    "s_waitcnt lgkmcnt(6)\n" \
    "v_mfma_f32_16x16x128_f8f6f4 v[168:171], v[112:119], v[32:39], v[168:171] cbsz:1 blgp:1\n" \
    "v_cmp_ge_u32 vcc, %[thr], v22\n" \
    "v_cmp_ge_u32 s[74:75], %[n1], v23\n" \
    "ds_read_b64_tr_b8 v[108:109], v9 offset:24576\n" \
    "ds_read_b64_tr_b8 v[110:111], v9 offset:25600\n" \
    "v_mfma_f32_16x16x128_f8f6f4 v[200:203], v[112:119], v[40:47], v[200:203] cbsz:1 blgp:1\n" \
    "ds_read_b64_tr_b8 v[112:113], v10 offset:16384\n" \
    "ds_read_b64_tr_b8 v[114:115], v10 offset:17408\n" \
    "s_or_b64 vcc, vcc, s[74:75]\n" \
    "s_cbranch_vccnz 500f\n" \
    "v_mfma_f32_16x16x128_f8f6f4 v[172:175], v[120:127], v[32:39], v[172:175] cbsz:1 blgp:1\n" \
    "ds_read_b64_tr_b8 v[116:117], v10 offset:24576\n" \
    "ds_read_b64_tr_b8 v[118:119], v10 offset:25600\n" \
    "buffer_load_dwordx4 v[80:83], v20, s[60:63], %[so] offen offset:0 sc0 sc1\n" \
    "v_mfma_f32_16x16x128_f8f6f4 v[204:207], v[120:127], v[40:47], v[204:207] cbsz:1 blgp:1\n" \
    "ds_read_b64_tr_b8 v[120:121], v11 offset:16384\n" \
    "ds_read_b64_tr_b8 v[122:123], v11 offset:17408\n" \
    "buffer_load_dwordx4 v[84:87], v20, s[60:63], %[so] offen offset:1024 sc0 sc1\n" \
    "s_waitcnt lgkmcnt(14)\n" \
    "v_mfma_f32_16x16x128_f8f6f4 v[176:179], v[128:135], v[32:39], v[176:179] cbsz:1 blgp:1\n" \
    "ds_read_b64_tr_b8 v[124:125], v11 offset:24576\n" \
    "ds_read_b64_tr_b8 v[126:127], v11 offset:25600\n" \
    "buffer_load_dwordx4 v[88:91], v20, s[60:63], %[so] offen offset:2048 sc0 sc1\n" \
    "v_mfma_f32_16x16x128_f8f6f4 v[208:211], v[128:135], v[40:47], v[208:211] cbsz:1 blgp:1\n" \
    "ds_read_b64_tr_b8 v[128:129], v12 offset:16384\n" \
    "ds_read_b64_tr_b8 v[130:131], v12 offset:17408\n" \
    "buffer_load_dwordx4 v[92:95], v20, s[60:63], %[so] offen offset:3072 sc0 sc1\n" \
    "v_mfma_f32_16x16x128_f8f6f4 v[180:183], v[136:143], v[32:39], v[180:183] cbsz:1 blgp:1\n" \
    "ds_read_b64_tr_b8 v[132:133], v12 offset:24576\n" \
    "ds_read_b64_tr_b8 v[134:135], v12 offset:25600\n" \
    "v_add_u32 v21, %[sb], v19\n" \
    "v_mfma_f32_16x16x128_f8f6f4 v[212:215], v[136:143], v[40:47], v[212:215] cbsz:1 blgp:1\n" \
    "ds_read_b64_tr_b8 v[136:137], v13 offset:16384\n" \
    "ds_read_b64_tr_b8 v[138:139], v13 offset:17408\n" \
    "s_waitcnt vmcnt(8)\n" \
    "v_max3_u32 v24, v48, v49, v50\n" \
    "s_waitcnt lgkmcnt(14)\n" \
    "v_mfma_f32_16x16x128_f8f6f4 v[184:187], v[144:151], v[32:39], v[184:187] cbsz:1 blgp:1\n" \
    "ds_read_b64_tr_b8 v[140:141], v13 offset:24576\n" \
    "ds_read_b64_tr_b8 v[142:143], v13 offset:25600\n" \
    "v_max3_u32 v25, v51, v52, v53\n" \
    "v_max3_u32 v26, v54, v55, v56\n" \
    "v_mfma_f32_16x16x128_f8f6f4 v[216:219], v[144:151], v[40:47], v[216:219] cbsz:1 blgp:1\n" \
    "ds_read_b64_tr_b8 v[144:145], v14 offset:16384\n" \
    "ds_read_b64_tr_b8 v[146:147], v14 offset:17408\n" \
    "v_max3_u32 v27, v57, v58, v59\n" \
    "v_max3_u32 v28, v60, v61, v62\n" \
    "s_sub_u32 %[dl], %[sb], %[sa]\n" \
    "s_lshl_b32 %[dl], %[dl], 5\n" \
    "v_mfma_f32_16x16x128_f8f6f4 v[188:191], v[152:159], v[32:39], v[188:191] cbsz:1 blgp:1\n" \
    "ds_read_b64_tr_b8 v[148:149], v14 offset:24576\n" \
    "ds_read_b64_tr_b8 v[150:151], v14 offset:25600\n" \
    "v_max3_u32 v24, v24, v25, v63\n" \
    "v_max3_u32 v26, v26, v27, v28\n" \
    "v_mfma_f32_16x16x128_f8f6f4 v[220:223], v[152:159], v[40:47], v[220:223] cbsz:1 blgp:1\n" \
    "v_max_u32 v24, v24, v26\n" \
    "ds_read_b64_tr_b8 v[152:153], v15 offset:16384\n" \
    "ds_read_b64_tr_b8 v[154:155], v15 offset:17408\n" \
    "ds_read_b64_tr_b8 v[156:157], v15 offset:24576\n" \
    "ds_read_b64_tr_b8 v[158:159], v15 offset:25600\n" \
    "ds_write_b32 v7, v22 offset:512\n" \
    "ds_read_b32 v22, v21 offset:0\n" \
    "ds_read_b32 v23, v19 offset:16\n" \
    "v_cmp_eq_u32 vcc, -1, v24\n" \
    "s_cbranch_vccnz 510f\n" \
    "601:\n" \
    "s_waitcnt lgkmcnt(14)\n" \
    "v_mfma_f32_16x16x128_f8f6f4 v[160:163], v[96:103], v[48:55], v[160:163] cbsz:1 blgp:1\n" \
    "s_add_u32 %[thr], %[u], 2\n" \
    "s_add_u32 %[n1], %[u], 4\n" \
    "v_add_u32 v8, %[dl], v8\n" \
    "v_mfma_f32_16x16x128_f8f6f4 v[192:195], v[96:103], v[56:63], v[192:195] cbsz:1 blgp:1\n" \
    "ds_read_b64_tr_b8 v[96:97], v8 offset:0\n" \
    "ds_read_b64_tr_b8 v[98:99], v8 offset:1024\n" \
    "s_lshl_b32 %[so], %[n1], 14\n" \
    "v_mfma_f32_16x16x128_f8f6f4 v[164:167], v[104:111], v[48:55], v[164:167] cbsz:1 blgp:1\n" \
    "ds_read_b64_tr_b8 v[100:101], v8 offset:8192\n" \
    "ds_read_b64_tr_b8 v[102:103], v8 offset:9216\n" \
    "v_add_u32 v9, %[dl], v9\n" \
    "s_mov_b32 %[sa], %[sb]\n" \
    "s_add_u32 %[sb], %[sb], 1024\n" \
    "v_mfma_f32_16x16x128_f8f6f4 v[196:199], v[104:111], v[56:63], v[196:199] cbsz:1 blgp:1\n" \
    "ds_read_b64_tr_b8 v[104:105], v9 offset:0\n" \
    "ds_read_b64_tr_b8 v[106:107], v9 offset:1024\n" \
    "s_waitcnt lgkmcnt(6)\n" \
    "v_mfma_f32_16x16x128_f8f6f4 v[168:171], v[112:119], v[48:55], v[168:171] cbsz:1 blgp:1\n" \
    "v_cmp_ge_u32 vcc, %[thr], v22\n" \
    "v_cmp_ge_u32 s[74:75], %[n1], v23\n" \
    "ds_read_b64_tr_b8 v[108:109], v9 offset:8192\n" \
    "ds_read_b64_tr_b8 v[110:111], v9 offset:9216\n" \
    "v_add_u32 v10, %[dl], v10\n" \
    "v_mfma_f32_16x16x128_f8f6f4 v[200:203], v[112:119], v[56:63], v[200:203] cbsz:1 blgp:1\n" \
    "ds_read_b64_tr_b8 v[112:113], v10 offset:0\n" \
    "ds_read_b64_tr_b8 v[114:115], v10 offset:1024\n" \
    "s_or_b64 vcc, vcc, s[74:75]\n" \
    "s_cbranch_vccnz 501f\n" \
    "v_mfma_f32_16x16x128_f8f6f4 v[172:175], v[120:127], v[48:55], v[172:175] cbsz:1 blgp:1\n" \
    "ds_read_b64_tr_b8 v[116:117], v10 offset:8192\n" \
    "ds_read_b64_tr_b8 v[118:119], v10 offset:9216\n" \
    "v_add_u32 v11, %[dl], v11\n" \
    "buffer_load_dwordx4 v[32:35], v20, s[60:63], %[so] offen offset:0 sc0 sc1\n" \
    "v_mfma_f32_16x16x128_f8f6f4 v[204:207], v[120:127], v[56:63], v[204:207] cbsz:1 blgp:1\n" \
    "ds_read_b64_tr_b8 v[120:121], v11 offset:0\n" \
    "ds_read_b64_tr_b8 v[122:123], v11 offset:1024\n" \
    "buffer_load_dwordx4 v[36:39], v20, s[60:63], %[so] offen offset:1024 sc0 sc1\n" \
    "s_waitcnt lgkmcnt(14)\n" \
    "v_mfma_f32_16x16x128_f8f6f4 v[176:179], v[128:135], v[48:55], v[176:179] cbsz:1 blgp:1\n" \
    "ds_read_b64_tr_b8 v[124:125], v11 offset:8192\n" \
    "ds_read_b64_tr_b8 v[126:127], v11 offset:9216\n" \
    "v_add_u32 v12, %[dl], v12\n" \
    "buffer_load_dwordx4 v[40:43], v20, s[60:63], %[so] offen offset:2048 sc0 sc1\n" \
    "s_cmp_eq_u32 %[sb], 3072\n" \
    "s_cselect_b32 %[sb], 0, %[sb]\n" \
    "v_mfma_f32_16x16x128_f8f6f4 v[208:211], v[128:135], v[56:63], v[208:211] cbsz:1 blgp:1\n" \
    "ds_read_b64_tr_b8 v[128:129], v12 offset:0\n" \
    "ds_read_b64_tr_b8 v[130:131], v12 offset:1024\n" \
    "buffer_load_dwordx4 v[44:47], v20, s[60:63], %[so] offen offset:3072 sc0 sc1\n" \
    "v_mfma_f32_16x16x128_f8f6f4 v[180:183], v[136:143], v[48:55], v[180:183] cbsz:1 blgp:1\n" \
    "ds_read_b64_tr_b8 v[132:133], v12 offset:8192\n" \
    "ds_read_b64_tr_b8 v[134:135], v12 offset:9216\n" \
    "v_add_u32 v13, %[dl], v13\n" \
    "v_add_u32 v7, %[sa], v18\n" \
    "v_mfma_f32_16x16x128_f8f6f4 v[212:215], v[136:143], v[56:63], v[212:215] cbsz:1 blgp:1\n" \
    "ds_read_b64_tr_b8 v[136:137], v13 offset:0\n" \
    "ds_read_b64_tr_b8 v[138:139], v13 offset:1024\n" \
    "s_waitcnt vmcnt(8)\n" \
    "v_max3_u32 v24, v64, v65, v66\n" \
    "s_waitcnt lgkmcnt(14)\n" \
    "v_mfma_f32_16x16x128_f8f6f4 v[184:187], v[144:151], v[48:55], v[184:187] cbsz:1 blgp:1\n" \
    "ds_read_b64_tr_b8 v[140:141], v13 offset:8192\n" \
    "ds_read_b64_tr_b8 v[142:143], v13 offset:9216\n" \
    "v_add_u32 v14, %[dl], v14\n" \
    "v_max3_u32 v25, v67, v68, v69\n" \
    "v_max3_u32 v26, v70, v71, v72\n" \
    "v_mfma_f32_16x16x128_f8f6f4 v[216:219], v[144:151], v[56:63], v[216:219] cbsz:1 blgp:1\n" \
    "ds_read_b64_tr_b8 v[144:145], v14 offset:0\n" \
    "ds_read_b64_tr_b8 v[146:147], v14 offset:1024\n" \
    "v_max3_u32 v27, v73, v74, v75\n" \
    "v_max3_u32 v28, v76, v77, v78\n" \
    "v_mfma_f32_16x16x128_f8f6f4 v[188:191], v[152:159], v[48:55], v[188:191] cbsz:1 blgp:1\n" \
    "ds_read_b64_tr_b8 v[148:149], v14 offset:8192\n" \
    "ds_read_b64_tr_b8 v[150:151], v14 offset:9216\n" \
    "v_add_u32 v15, %[dl], v15\n" \
    "v_max3_u32 v24, v24, v25, v79\n" \
    "v_max3_u32 v26, v26, v27, v28\n" \
    "v_mfma_f32_16x16x128_f8f6f4 v[220:223], v[152:159], v[56:63], v[220:223] cbsz:1 blgp:1\n" \
    "v_max_u32 v24, v24, v26\n" \
    "ds_read_b64_tr_b8 v[152:153], v15 offset:0\n" \
    "ds_read_b64_tr_b8 v[154:155], v15 offset:1024\n" \
    "ds_read_b64_tr_b8 v[156:157], v15 offset:8192\n" \
    "ds_read_b64_tr_b8 v[158:159], v15 offset:9216\n" \
    "ds_write_b32 v7, v22 offset:0\n" \
    "ds_read_b32 v22, v21 offset:512\n" \
    "ds_read_b32 v23, v19 offset:16\n" \
    "v_cmp_eq_u32 vcc, -1, v24\n" \
    "s_cbranch_vccnz 511f\n" \
    "602:\n" \
    "s_waitcnt lgkmcnt(14)\n" \
    "v_mfma_f32_16x16x128_f8f6f4 v[160:163], v[96:103], v[64:71], v[160:163] cbsz:1 blgp:1\n" \
    "s_add_u32 %[thr], %[u], 3\n" \
    "s_add_u32 %[n1], %[u], 5\n" \
    "v_mfma_f32_16x16x128_f8f6f4 v[192:195], v[96:103], v[72:79], v[192:195] cbsz:1 blgp:1\n" \
    "ds_read_b64_tr_b8 v[96:97], v8 offset:16384\n" \
    "ds_read_b64_tr_b8 v[98:99], v8 offset:17408\n" \
    "s_lshl_b32 %[so], %[n1], 14\n" \
    "v_mfma_f32_16x16x128_f8f6f4 v[164:167], v[104:111], v[64:71], v[164:167] cbsz:1 blgp:1\n" \
    "ds_read_b64_tr_b8 v[100:101], v8 offset:24576\n" \
    "ds_read_b64_tr_b8 v[102:103], v8 offset:25600\n" \
    "v_mfma_f32_16x16x128_f8f6f4 v[196:199], v[104:111], v[72:79], v[196:199] cbsz:1 blgp:1\n" \
    "ds_read_b64_tr_b8 v[104:105], v9 offset:16384\n" \
    "ds_read_b64_tr_b8 v[106:107], v9 offset:17408\n" \
    "s_waitcnt lgkmcnt(6)\n" \
    "v_mfma_f32_16x16x128_f8f6f4 v[168:171], v[112:119], v[64:71], v[168:171] cbsz:1 blgp:1\n" \
    "v_cmp_ge_u32 vcc, %[thr], v22\n" \
    "v_cmp_ge_u32 s[74:75], %[n1], v23\n" \
    "ds_read_b64_tr_b8 v[108:109], v9 offset:24576\n" \
    "ds_read_b64_tr_b8 v[110:111], v9 offset:25600\n" \
    "v_mfma_f32_16x16x128_f8f6f4 v[200:203], v[112:119], v[72:79], v[200:203] cbsz:1 blgp:1\n" \
    "ds_read_b64_tr_b8 v[112:113], v10 offset:16384\n" \
    "ds_read_b64_tr_b8 v[114:115], v10 offset:17408\n" \
    "s_or_b64 vcc, vcc, s[74:75]\n" \
    "s_cbranch_vccnz 502f\n" \
    "v_mfma_f32_16x16x128_f8f6f4 v[172:175], v[120:127], v[64:71], v[172:175] cbsz:1 blgp:1\n" \
    "ds_read_b64_tr_b8 v[116:117], v10 offset:24576\n" \
    "ds_read_b64_tr_b8 v[118:119], v10 offset:25600\n" \
    "buffer_load_dwordx4 v[48:51], v20, s[60:63], %[so] offen offset:0 sc0 sc1\n" \
    "v_mfma_f32_16x16x128_f8f6f4 v[204:207], v[120:127], v[72:79], v[204:207] cbsz:1 blgp:1\n" \
    "ds_read_b64_tr_b8 v[120:121], v11 offset:16384\n" \
    "ds_read_b64_tr_b8 v[122:123], v11 offset:17408\n" \
    "buffer_load_dwordx4 v[52:55], v20, s[60:63], %[so] offen offset:1024 sc0 sc1\n" \
    "s_waitcnt lgkmcnt(14)\n" \
    "v_mfma_f32_16x16x128_f8f6f4 v[176:179], v[128:135], v[64:71], v[176:179] cbsz:1 blgp:1\n" \
    "ds_read_b64_tr_b8 v[124:125], v11 offset:24576\n" \
    "ds_read_b64_tr_b8 v[126:127], v11 offset:25600\n" \
    "buffer_load_dwordx4 v[56:59], v20, s[60:63], %[so] offen offset:2048 sc0 sc1\n" \
    "v_mfma_f32_16x16x128_f8f6f4 v[208:211], v[128:135], v[72:79], v[208:211] cbsz:1 blgp:1\n" \
    "ds_read_b64_tr_b8 v[128:129], v12 offset:16384\n" \
    "ds_read_b64_tr_b8 v[130:131], v12 offset:17408\n" \
    "buffer_load_dwordx4 v[60:63], v20, s[60:63], %[so] offen offset:3072 sc0 sc1\n" \
    "v_mfma_f32_16x16x128_f8f6f4 v[180:183], v[136:143], v[64:71], v[180:183] cbsz:1 blgp:1\n" \
    "ds_read_b64_tr_b8 v[132:133], v12 offset:24576\n" \
    "ds_read_b64_tr_b8 v[134:135], v12 offset:25600\n" \
    "v_add_u32 v21, %[sb], v19\n" \
    "v_mfma_f32_16x16x128_f8f6f4 v[212:215], v[136:143], v[72:79], v[212:215] cbsz:1 blgp:1\n" \
    "ds_read_b64_tr_b8 v[136:137], v13 offset:16384\n" \
    "ds_read_b64_tr_b8 v[138:139], v13 offset:17408\n" \
    "s_waitcnt vmcnt(8)\n" \
    "v_max3_u32 v24, v80, v81, v82\n" \
    "s_waitcnt lgkmcnt(14)\n" \
    "v_mfma_f32_16x16x128_f8f6f4 v[184:187], v[144:151], v[64:71], v[184:187] cbsz:1 blgp:1\n" \
    "ds_read_b64_tr_b8 v[140:141], v13 offset:24576\n" \
    "ds_read_b64_tr_b8 v[142:143], v13 offset:25600\n" \
    "v_max3_u32 v25, v83, v84, v85\n" \
    "v_max3_u32 v26, v86, v87, v88\n" \
    "v_mfma_f32_16x16x128_f8f6f4 v[216:219], v[144:151], v[72:79], v[216:219] cbsz:1 blgp:1\n" \
    "ds_read_b64_tr_b8 v[144:145], v14 offset:16384\n" \
    "ds_read_b64_tr_b8 v[146:147], v14 offset:17408\n" \
    "v_max3_u32 v27, v89, v90, v91\n" \
    "v_max3_u32 v28, v92, v93, v94\n" \
    "s_sub_u32 %[dl], %[sb], %[sa]\n" \
    "s_lshl_b32 %[dl], %[dl], 5\n" \
    "v_mfma_f32_16x16x128_f8f6f4 v[188:191], v[152:159], v[64:71], v[188:191] cbsz:1 blgp:1\n" \
    "ds_read_b64_tr_b8 v[148:149], v14 offset:24576\n" \
    "ds_read_b64_tr_b8 v[150:151], v14 offset:25600\n" \
    "v_max3_u32 v24, v24, v25, v95\n" \
    "v_max3_u32 v26, v26, v27, v28\n" \
    "v_mfma_f32_16x16x128_f8f6f4 v[220:223], v[152:159], v[72:79], v[220:223] cbsz:1 blgp:1\n" \
    "v_max_u32 v24, v24, v26\n" \
    "ds_read_b64_tr_b8 v[152:153], v15 offset:16384\n" \
    "ds_read_b64_tr_b8 v[154:155], v15 offset:17408\n" \
    "ds_read_b64_tr_b8 v[156:157], v15 offset:24576\n" \
    "ds_read_b64_tr_b8 v[158:159], v15 offset:25600\n" \
    "ds_write_b32 v7, v22 offset:512\n" \
    "ds_read_b32 v22, v21 offset:0\n" \
    "ds_read_b32 v23, v19 offset:16\n" \
    "v_cmp_eq_u32 vcc, -1, v24\n" \
    "s_cbranch_vccnz 512f\n" \
    "603:\n" \
    "s_waitcnt lgkmcnt(14)\n" \
    "v_mfma_f32_16x16x128_f8f6f4 v[160:163], v[96:103], v[80:87], v[160:163] cbsz:1 blgp:1\n" \
    "s_add_u32 %[thr], %[u], 4\n" \
    "s_cmp_lt_u32 %[thr], 64\n" \
    "s_cselect_b32 %[thr], %[thr], 0\n" \
    "s_add_u32 %[n1], %[u], 6\n" \
    "v_add_u32 v8, %[dl], v8\n" \
    "v_mfma_f32_16x16x128_f8f6f4 v[192:195], v[96:103], v[88:95], v[192:195] cbsz:1 blgp:1\n" \
    "ds_read_b64_tr_b8 v[96:97], v8 offset:0\n" \
    "ds_read_b64_tr_b8 v[98:99], v8 offset:1024\n" \
    "s_lshl_b32 %[so], %[n1], 14\n" \
    "v_mfma_f32_16x16x128_f8f6f4 v[164:167], v[104:111], v[80:87], v[164:167] cbsz:1 blgp:1\n" \
    "ds_read_b64_tr_b8 v[100:101], v8 offset:8192\n" \
    "ds_read_b64_tr_b8 v[102:103], v8 offset:9216\n" \
    "v_add_u32 v9, %[dl], v9\n" \
    "s_mov_b32 %[sa], %[sb]\n" \
    "s_add_u32 %[sb], %[sb], 1024\n" \
    "v_mfma_f32_16x16x128_f8f6f4 v[196:199], v[104:111], v[88:95], v[196:199] cbsz:1 blgp:1\n" \
    "ds_read_b64_tr_b8 v[104:105], v9 offset:0\n" \
    "ds_read_b64_tr_b8 v[106:107], v9 offset:1024\n" \
    "s_waitcnt lgkmcnt(6)\n" \
    "v_mfma_f32_16x16x128_f8f6f4 v[168:171], v[112:119], v[80:87], v[168:171] cbsz:1 blgp:1\n" \
    "v_cmp_ge_u32 vcc, %[thr], v22\n" \
    "v_cmp_ge_u32 s[74:75], %[n1], v23\n" \
    "ds_read_b64_tr_b8 v[108:109], v9 offset:8192\n" \
    "ds_read_b64_tr_b8 v[110:111], v9 offset:9216\n" \
    "v_add_u32 v10, %[dl], v10\n" \
    "v_mfma_f32_16x16x128_f8f6f4 v[200:203], v[112:119], v[88:95], v[200:203] cbsz:1 blgp:1\n" \
    "ds_read_b64_tr_b8 v[112:113], v10 offset:0\n" \
    "ds_read_b64_tr_b8 v[114:115], v10 offset:1024\n" \
    "s_or_b64 vcc, vcc, s[74:75]\n" \
    "s_cbranch_vccnz 503f\n" \
    "v_mfma_f32_16x16x128_f8f6f4 v[172:175], v[120:127], v[80:87], v[172:175] cbsz:1 blgp:1\n" \
    "ds_read_b64_tr_b8 v[116:117], v10 offset:8192\n" \
    "ds_read_b64_tr_b8 v[118:119], v10 offset:9216\n" \
    "v_add_u32 v11, %[dl], v11\n" \
    "buffer_load_dwordx4 v[64:67], v20, s[60:63], %[so] offen offset:0 sc0 sc1\n" \
    "v_mfma_f32_16x16x128_f8f6f4 v[204:207], v[120:127], v[88:95], v[204:207] cbsz:1 blgp:1\n" \
    "ds_read_b64_tr_b8 v[120:121], v11 offset:0\n" \
    "ds_read_b64_tr_b8 v[122:123], v11 offset:1024\n" \
    "buffer_load_dwordx4 v[68:71], v20, s[60:63], %[so] offen offset:1024 sc0 sc1\n" \
    "s_waitcnt lgkmcnt(14)\n" \
    "v_mfma_f32_16x16x128_f8f6f4 v[176:179], v[128:135], v[80:87], v[176:179] cbsz:1 blgp:1\n" \
    "ds_read_b64_tr_b8 v[124:125], v11 offset:8192\n" \
    "ds_read_b64_tr_b8 v[126:127], v11 offset:9216\n" \
    "v_add_u32 v12, %[dl], v12\n" \
    "buffer_load_dwordx4 v[72:75], v20, s[60:63], %[so] offen offset:2048 sc0 sc1\n" \
    "s_cmp_eq_u32 %[sb], 3072\n" \
    "s_cselect_b32 %[sb], 0, %[sb]\n" \
    "v_mfma_f32_16x16x128_f8f6f4 v[208:211], v[128:135], v[88:95], v[208:211] cbsz:1 blgp:1\n" \
    "ds_read_b64_tr_b8 v[128:129], v12 offset:0\n" \
    "ds_read_b64_tr_b8 v[130:131], v12 offset:1024\n" \
    "buffer_load_dwordx4 v[76:79], v20, s[60:63], %[so] offen offset:3072 sc0 sc1\n" \
    "v_mfma_f32_16x16x128_f8f6f4 v[180:183], v[136:143], v[80:87], v[180:183] cbsz:1 blgp:1\n" \
    "ds_read_b64_tr_b8 v[132:133], v12 offset:8192\n" \
    "ds_read_b64_tr_b8 v[134:135], v12 offset:9216\n" \
    "v_add_u32 v13, %[dl], v13\n" \
    "v_add_u32 v7, %[sa], v18\n" \
    "v_mfma_f32_16x16x128_f8f6f4 v[212:215], v[136:143], v[88:95], v[212:215] cbsz:1 blgp:1\n" \
    "ds_read_b64_tr_b8 v[136:137], v13 offset:0\n" \
    "ds_read_b64_tr_b8 v[138:139], v13 offset:1024\n" \
    "s_waitcnt vmcnt(8)\n" \
    "v_max3_u32 v24, v32, v33, v34\n" \
    "s_waitcnt lgkmcnt(14)\n" \
    "v_mfma_f32_16x16x128_f8f6f4 v[184:187], v[144:151], v[80:87], v[184:187] cbsz:1 blgp:1\n" \
    "ds_read_b64_tr_b8 v[140:141], v13 offset:8192\n" \
    "ds_read_b64_tr_b8 v[142:143], v13 offset:9216\n" \
    "v_add_u32 v14, %[dl], v14\n" \
    "v_max3_u32 v25, v35, v36, v37\n" \
    "v_max3_u32 v26, v38, v39, v40\n" \
    "v_mfma_f32_16x16x128_f8f6f4 v[216:219], v[144:151], v[88:95], v[216:219] cbsz:1 blgp:1\n" \
    "ds_read_b64_tr_b8 v[144:145], v14 offset:0\n" \
    "ds_read_b64_tr_b8 v[146:147], v14 offset:1024\n" \
    "v_max3_u32 v27, v41, v42, v43\n" \
    "v_max3_u32 v28, v44, v45, v46\n" \
    "v_mfma_f32_16x16x128_f8f6f4 v[188:191], v[152:159], v[80:87], v[188:191] cbsz:1 blgp:1\n" \
    "ds_read_b64_tr_b8 v[148:149], v14 offset:8192\n" \
    "ds_read_b64_tr_b8 v[150:151], v14 offset:9216\n" \
    "v_add_u32 v15, %[dl], v15\n" \
    "v_max3_u32 v24, v24, v25, v47\n" \
    "v_max3_u32 v26, v26, v27, v28\n" \
    "v_mfma_f32_16x16x128_f8f6f4 v[220:223], v[152:159], v[88:95], v[220:223] cbsz:1 blgp:1\n" \
    "v_max_u32 v24, v24, v26\n" \
    "ds_read_b64_tr_b8 v[152:153], v15 offset:0\n" \
    "ds_read_b64_tr_b8 v[154:155], v15 offset:1024\n" \
    "ds_read_b64_tr_b8 v[156:157], v15 offset:8192\n" \
    "ds_read_b64_tr_b8 v[158:159], v15 offset:9216\n" \
    "ds_write_b32 v7, v22 offset:0\n" \
    "ds_read_b32 v22, v21 offset:512\n" \
    "ds_read_b32 v23, v19 offset:16\n" \
    "v_cmp_eq_u32 vcc, -1, v24\n" \
    "s_cbranch_vccnz 513f\n" \
    "69:\n" \
    "s_add_u32 %[u], %[u], 4\n" \
    "s_cmp_lt_u32 %[u], %[uend]\n" \
    "s_cbranch_scc1 600b\n" \
    "s_add_u32 %[xi], %[u], 3\n" \
    "s_sub_u32 s71, %[vc], 2\n" \
    "s_sub_u32 s72, %[vc], 1\n" \
    "s_branch 9f\n" \
    "500:\n" \
    "v_cmp_ge_u32 vcc, %[thr], v22\n" \
    "v_readfirstlane_b32 %[rv], v23\n" \
    "s_mov_b32 %[xi], %[n1]\n" \
    "s_sub_u32 s71, %[vc], 2\n" \
    "s_sub_u32 s72, %[vc], 1\n" \
    "s_branch 520f\n" \
    "510:\n" \
    "s_cmp_ge_u32 %[thr], 48\n" \
    "s_cbranch_scc1 450f\n" \
    "s_branch 601b\n" \
    "501:\n" \
    "v_cmp_ge_u32 vcc, %[thr], v22\n" \
    "v_readfirstlane_b32 %[rv], v23\n" \
    "s_mov_b32 %[xi], %[n1]\n" \
    "s_sub_u32 s72, %[vc], 2\n" \
    "s_sub_u32 s73, %[vc], 1\n" \
    "s_branch 521f\n" \
    "511:\n" \
    "s_cmp_ge_u32 %[thr], 48\n" \
    "s_cbranch_scc1 451f\n" \
    "s_branch 602b\n" \
    "502:\n" \
    "v_cmp_ge_u32 vcc, %[thr], v22\n" \
    "v_readfirstlane_b32 %[rv], v23\n" \
    "s_mov_b32 %[xi], %[n1]\n" \
    "s_sub_u32 s73, %[vc], 2\n" \
    "s_sub_u32 s70, %[vc], 1\n" \
    "s_branch 522f\n" \
    "512:\n" \
    "s_cmp_ge_u32 %[thr], 48\n" \
    "s_cbranch_scc1 452f\n" \
    "s_branch 603b\n" \
    "503:\n" \
    "v_cmp_ge_u32 vcc, %[thr], v22\n" \
    "v_readfirstlane_b32 %[rv], v23\n" \
    "s_mov_b32 %[xi], %[n1]\n" \
    "s_sub_u32 s70, %[vc], 2\n" \
    "s_sub_u32 s71, %[vc], 1\n" \
    "s_branch 523f\n" \
    "513:\n" \
    "s_cmp_ge_u32 %[thr], 48\n" \
    "s_cbranch_scc1 453f\n" \
    "s_branch 69b\n" \
    "0:\n" \
    "s_waitcnt lgkmcnt(14)\n" \
    "v_mfma_f32_16x16x128_f8f6f4 v[160:163], v[96:103], v[32:39], v[160:163] cbsz:1 blgp:1\n" \
    "s_add_u32 %[thr], %[u], 1\n" \
    "s_add_u32 %[n1], %[u], 3\n" \
    "v_mfma_f32_16x16x128_f8f6f4 v[192:195], v[96:103], v[40:47], v[192:195] cbsz:1 blgp:1\n" \
    "ds_read_b64_tr_b8 v[96:97], v8 offset:16384\n" \
    "ds_read_b64_tr_b8 v[98:99], v8 offset:17408\n" \
    "s_lshl_b32 %[so], %[n1], 14\n" \
    "v_mfma_f32_16x16x128_f8f6f4 v[164:167], v[104:111], v[32:39], v[164:167] cbsz:1 blgp:1\n" \
    "ds_read_b64_tr_b8 v[100:101], v8 offset:24576\n" \
    "ds_read_b64_tr_b8 v[102:103], v8 offset:25600\n" \
    "v_mfma_f32_16x16x128_f8f6f4 v[196:199], v[104:111], v[40:47], v[196:199] cbsz:1 blgp:1\n" \
    "ds_read_b64_tr_b8 v[104:105], v9 offset:16384\n" \
    "ds_read_b64_tr_b8 v[106:107], v9 offset:17408\n" \
    "s_waitcnt lgkmcnt(6)\n" \
    "v_mfma_f32_16x16x128_f8f6f4 v[168:171], v[112:119], v[32:39], v[168:171] cbsz:1 blgp:1\n" \
    "v_cmp_ge_u32 vcc, %[thr], v22\n" \
    "v_readfirstlane_b32 %[rv], v23\n" \
    "ds_read_b64_tr_b8 v[108:109], v9 offset:24576\n" \
    "ds_read_b64_tr_b8 v[110:111], v9 offset:25600\n" \
    "v_mfma_f32_16x16x128_f8f6f4 v[200:203], v[112:119], v[40:47], v[200:203] cbsz:1 blgp:1\n" \
    "ds_read_b64_tr_b8 v[112:113], v10 offset:16384\n" \
    "ds_read_b64_tr_b8 v[114:115], v10 offset:17408\n" \
    "520:\n" \
    "s_cmp_eq_u32 %[xi], %[n1]\n" \
    "s_cselect_b32 %[rv], %[rv], 0\n" \
    "v_mfma_f32_16x16x128_f8f6f4 v[172:175], v[120:127], v[32:39], v[172:175] cbsz:1 blgp:1\n" \
    "ds_read_b64_tr_b8 v[116:117], v10 offset:24576\n" \
    "ds_read_b64_tr_b8 v[118:119], v10 offset:25600\n" \
    "s_cmp_gt_u32 %[rv], %[n1]\n" \
    "s_cselect_b32 s62, %[sz], 0\n" \
    "v_mfma_f32_16x16x128_f8f6f4 v[204:207], v[120:127], v[40:47], v[204:207] cbsz:1 blgp:1\n" \
    "s_cselect_b32 s73, %[vc], s73\n" \
    "s_addc_u32 %[xi], %[xi], 0\n" \
    "ds_read_b64_tr_b8 v[120:121], v11 offset:16384\n" \
    "ds_read_b64_tr_b8 v[122:123], v11 offset:17408\n" \
    "buffer_load_dwordx4 v[80:83], v20, s[60:63], %[so] offen offset:0 sc0 sc1\n" \
    "s_waitcnt lgkmcnt(14)\n" \
    "v_mfma_f32_16x16x128_f8f6f4 v[176:179], v[128:135], v[32:39], v[176:179] cbsz:1 blgp:1\n" \
    "ds_read_b32 v23, v19 offset:16\n" \
    "ds_read_b64_tr_b8 v[124:125], v11 offset:24576\n" \
    "ds_read_b64_tr_b8 v[126:127], v11 offset:25600\n" \
    "buffer_load_dwordx4 v[84:87], v20, s[60:63], %[so] offen offset:1024 sc0 sc1\n" \
    "s_add_u32 %[vc], %[vc], 1\n" \
    "v_mfma_f32_16x16x128_f8f6f4 v[208:211], v[128:135], v[40:47], v[208:211] cbsz:1 blgp:1\n" \
    "ds_read_b64_tr_b8 v[128:129], v12 offset:16384\n" \
    "ds_read_b64_tr_b8 v[130:131], v12 offset:17408\n" \
    "buffer_load_dwordx4 v[88:91], v20, s[60:63], %[so] offen offset:2048 sc0 sc1\n" \
    "v_mfma_f32_16x16x128_f8f6f4 v[180:183], v[136:143], v[32:39], v[180:183] cbsz:1 blgp:1\n" \
    "ds_read_b64_tr_b8 v[132:133], v12 offset:24576\n" \
    "ds_read_b64_tr_b8 v[134:135], v12 offset:25600\n" \
    "buffer_load_dwordx4 v[92:95], v20, s[60:63], %[so] offen offset:3072 sc0 sc1\n" \
    "v_add_u32 v21, %[sb], v19\n" \
    "v_mfma_f32_16x16x128_f8f6f4 v[212:215], v[136:143], v[40:47], v[212:215] cbsz:1 blgp:1\n" \
    "ds_read_b64_tr_b8 v[136:137], v13 offset:16384\n" \
    "ds_read_b64_tr_b8 v[138:139], v13 offset:17408\n" \
    "s_cmp_le_u32 %[xi], %[n1]\n" \
    "s_cbranch_scc1 300f\n" \
    "310:\n" \
    "s_waitcnt lgkmcnt(14)\n" \
    "v_mfma_f32_16x16x128_f8f6f4 v[184:187], v[144:151], v[32:39], v[184:187] cbsz:1 blgp:1\n" \
    "ds_read_b64_tr_b8 v[140:141], v13 offset:24576\n" \
    "ds_read_b64_tr_b8 v[142:143], v13 offset:25600\n" \
    "s_sub_u32 %[c], %[vc], s71\n" \
    "s_add_u32 %[n1], %[u], 2\n" \
    "v_mfma_f32_16x16x128_f8f6f4 v[216:219], v[144:151], v[40:47], v[216:219] cbsz:1 blgp:1\n" \
    "ds_read_b64_tr_b8 v[144:145], v14 offset:16384\n" \
    "ds_read_b64_tr_b8 v[146:147], v14 offset:17408\n" \
    "s_sub_u32 %[c], %[c], 3\n" \
    "s_sub_u32 %[n1], %[xi], %[n1]\n" \
    "s_sub_u32 %[dl], %[sb], %[sa]\n" \
    "s_lshl_b32 %[dl], %[dl], 5\n" \
    "v_mfma_f32_16x16x128_f8f6f4 v[188:191], v[152:159], v[32:39], v[188:191] cbsz:1 blgp:1\n" \
    "ds_read_b64_tr_b8 v[148:149], v14 offset:24576\n" \
    "ds_read_b64_tr_b8 v[150:151], v14 offset:25600\n" \
    "s_andn2_b32 %[c], %[c], 3\n" \
    "s_lshr_b32 %[n1], %[n1], 31\n" \
    "v_mfma_f32_16x16x128_f8f6f4 v[220:223], v[152:159], v[40:47], v[220:223] cbsz:1 blgp:1\n" \
    "s_or_b32 %[c], %[c], %[n1]\n" \
    "ds_read_b64_tr_b8 v[152:153], v15 offset:16384\n" \
    "ds_read_b64_tr_b8 v[154:155], v15 offset:17408\n" \
    "ds_read_b64_tr_b8 v[156:157], v15 offset:24576\n" \
    "ds_read_b64_tr_b8 v[158:159], v15 offset:25600\n" \
    "ds_write_b32 v7, v22 offset:512\n" \
    "ds_read_b32 v22, v21 offset:0\n" \
    "ds_read_b32 v23, v19 offset:16\n" \
    "s_cmp_lg_u32 %[c], 0\n" \
    "s_cbranch_scc1 20f\n" \
    "s_cbranch_vccnz 20f\n" \
    "s_waitcnt vmcnt(8)\n" \
    "10:\n" \
    "s_cmp_ge_u32 %[thr], 48\n" \
    "s_cbranch_scc1 400f\n" \
    "410:\n" \
    "s_sub_u32 %[c], %[vc], s72\n" \
    "s_sub_u32 %[n1], %[xi], %[u]\n" \
    "s_sub_u32 %[c], %[c], 2\n" \
    "s_sub_u32 %[n1], %[n1], 4\n" \
    "s_mov_b32 s62, %[sz]\n" \
    "s_or_b32 %[c], %[c], %[n1]\n" \
    "s_cbranch_scc0 601b\n" \
    "s_waitcnt lgkmcnt(14)\n" \
    "v_mfma_f32_16x16x128_f8f6f4 v[160:163], v[96:103], v[48:55], v[160:163] cbsz:1 blgp:1\n" \
    "s_add_u32 %[thr], %[u], 2\n" \
    "s_add_u32 %[n1], %[u], 4\n" \
    "v_add_u32 v8, %[dl], v8\n" \
    "v_mfma_f32_16x16x128_f8f6f4 v[192:195], v[96:103], v[56:63], v[192:195] cbsz:1 blgp:1\n" \
    "ds_read_b64_tr_b8 v[96:97], v8 offset:0\n" \
    "ds_read_b64_tr_b8 v[98:99], v8 offset:1024\n" \
    "s_lshl_b32 %[so], %[n1], 14\n" \
    "v_mfma_f32_16x16x128_f8f6f4 v[164:167], v[104:111], v[48:55], v[164:167] cbsz:1 blgp:1\n" \
    "ds_read_b64_tr_b8 v[100:101], v8 offset:8192\n" \
    "ds_read_b64_tr_b8 v[102:103], v8 offset:9216\n" \
    "v_add_u32 v9, %[dl], v9\n" \
    "s_mov_b32 %[sa], %[sb]\n" \
    "s_add_u32 %[sb], %[sb], 1024\n" \
    "v_mfma_f32_16x16x128_f8f6f4 v[196:199], v[104:111], v[56:63], v[196:199] cbsz:1 blgp:1\n" \
    "ds_read_b64_tr_b8 v[104:105], v9 offset:0\n" \
    "ds_read_b64_tr_b8 v[106:107], v9 offset:1024\n" \
    "s_waitcnt lgkmcnt(6)\n" \
    "v_mfma_f32_16x16x128_f8f6f4 v[168:171], v[112:119], v[48:55], v[168:171] cbsz:1 blgp:1\n" \
    "v_cmp_ge_u32 vcc, %[thr], v22\n" \
    "v_readfirstlane_b32 %[rv], v23\n" \
    "ds_read_b64_tr_b8 v[108:109], v9 offset:8192\n" \
    "ds_read_b64_tr_b8 v[110:111], v9 offset:9216\n" \
    "v_add_u32 v10, %[dl], v10\n" \
    "v_mfma_f32_16x16x128_f8f6f4 v[200:203], v[112:119], v[56:63], v[200:203] cbsz:1 blgp:1\n" \
    "ds_read_b64_tr_b8 v[112:113], v10 offset:0\n" \
    "ds_read_b64_tr_b8 v[114:115], v10 offset:1024\n" \
    "521:\n" \
    "s_cmp_eq_u32 %[xi], %[n1]\n" \
    "s_cselect_b32 %[rv], %[rv], 0\n" \
    "v_mfma_f32_16x16x128_f8f6f4 v[172:175], v[120:127], v[48:55], v[172:175] cbsz:1 blgp:1\n" \
    "ds_read_b64_tr_b8 v[116:117], v10 offset:8192\n" \
    "ds_read_b64_tr_b8 v[118:119], v10 offset:9216\n" \
    "v_add_u32 v11, %[dl], v11\n" \
    "s_cmp_gt_u32 %[rv], %[n1]\n" \
    "s_cselect_b32 s62, %[sz], 0\n" \
    "v_mfma_f32_16x16x128_f8f6f4 v[204:207], v[120:127], v[56:63], v[204:207] cbsz:1 blgp:1\n" \
    "s_cselect_b32 s70, %[vc], s70\n" \
    "s_addc_u32 %[xi], %[xi], 0\n" \
    "ds_read_b64_tr_b8 v[120:121], v11 offset:0\n" \
    "ds_read_b64_tr_b8 v[122:123], v11 offset:1024\n" \
    "buffer_load_dwordx4 v[32:35], v20, s[60:63], %[so] offen offset:0 sc0 sc1\n" \
    "s_waitcnt lgkmcnt(14)\n" \
    "v_mfma_f32_16x16x128_f8f6f4 v[176:179], v[128:135], v[48:55], v[176:179] cbsz:1 blgp:1\n" \
    "ds_read_b32 v23, v19 offset:16\n" \
    "ds_read_b64_tr_b8 v[124:125], v11 offset:8192\n" \
    "ds_read_b64_tr_b8 v[126:127], v11 offset:9216\n" \
    "v_add_u32 v12, %[dl], v12\n" \
    "buffer_load_dwordx4 v[36:39], v20, s[60:63], %[so] offen offset:1024 sc0 sc1\n" \
    "s_add_u32 %[vc], %[vc], 1\n" \
    "v_mfma_f32_16x16x128_f8f6f4 v[208:211], v[128:135], v[56:63], v[208:211] cbsz:1 blgp:1\n" \
    "ds_read_b64_tr_b8 v[128:129], v12 offset:0\n" \
    "ds_read_b64_tr_b8 v[130:131], v12 offset:1024\n" \
    "buffer_load_dwordx4 v[40:43], v20, s[60:63], %[so] offen offset:2048 sc0 sc1\n" \
    "s_cmp_eq_u32 %[sb], 3072\n" \
    "s_cselect_b32 %[sb], 0, %[sb]\n" \
    "v_mfma_f32_16x16x128_f8f6f4 v[180:183], v[136:143], v[48:55], v[180:183] cbsz:1 blgp:1\n" \
    "ds_read_b64_tr_b8 v[132:133], v12 offset:8192\n" \
    "ds_read_b64_tr_b8 v[134:135], v12 offset:9216\n" \
    "v_add_u32 v13, %[dl], v13\n" \
    "buffer_load_dwordx4 v[44:47], v20, s[60:63], %[so] offen offset:3072 sc0 sc1\n" \
    "v_add_u32 v7, %[sa], v18\n" \
    "v_mfma_f32_16x16x128_f8f6f4 v[212:215], v[136:143], v[56:63], v[212:215] cbsz:1 blgp:1\n" \
    "ds_read_b64_tr_b8 v[136:137], v13 offset:0\n" \
    "ds_read_b64_tr_b8 v[138:139], v13 offset:1024\n" \
    "s_cmp_le_u32 %[xi], %[n1]\n" \
    "s_cbranch_scc1 301f\n" \
    "311:\n" \
    "s_waitcnt lgkmcnt(14)\n" \
    "v_mfma_f32_16x16x128_f8f6f4 v[184:187], v[144:151], v[48:55], v[184:187] cbsz:1 blgp:1\n" \
    "ds_read_b64_tr_b8 v[140:141], v13 offset:8192\n" \
    "ds_read_b64_tr_b8 v[142:143], v13 offset:9216\n" \
    "v_add_u32 v14, %[dl], v14\n" \
    "s_sub_u32 %[c], %[vc], s72\n" \
    "s_add_u32 %[n1], %[u], 3\n" \
    "v_mfma_f32_16x16x128_f8f6f4 v[216:219], v[144:151], v[56:63], v[216:219] cbsz:1 blgp:1\n" \
    "ds_read_b64_tr_b8 v[144:145], v14 offset:0\n" \
    "ds_read_b64_tr_b8 v[146:147], v14 offset:1024\n" \
    "s_sub_u32 %[c], %[c], 3\n" \
    "s_sub_u32 %[n1], %[xi], %[n1]\n" \
    "v_mfma_f32_16x16x128_f8f6f4 v[188:191], v[152:159], v[48:55], v[188:191] cbsz:1 blgp:1\n" \
    "ds_read_b64_tr_b8 v[148:149], v14 offset:8192\n" \
    "ds_read_b64_tr_b8 v[150:151], v14 offset:9216\n" \
    "v_add_u32 v15, %[dl], v15\n" \
    "s_andn2_b32 %[c], %[c], 3\n" \
    "s_lshr_b32 %[n1], %[n1], 31\n" \
    "v_mfma_f32_16x16x128_f8f6f4 v[220:223], v[152:159], v[56:63], v[220:223] cbsz:1 blgp:1\n" \
    "s_or_b32 %[c], %[c], %[n1]\n" \
    "ds_read_b64_tr_b8 v[152:153], v15 offset:0\n" \
    "ds_read_b64_tr_b8 v[154:155], v15 offset:1024\n" \
    "ds_read_b64_tr_b8 v[156:157], v15 offset:8192\n" \
    "ds_read_b64_tr_b8 v[158:159], v15 offset:9216\n" \
    "ds_write_b32 v7, v22 offset:0\n" \
    "ds_read_b32 v22, v21 offset:512\n" \
    "ds_read_b32 v23, v19 offset:16\n" \
    "s_cmp_lg_u32 %[c], 0\n" \
    "s_cbranch_scc1 21f\n" \
    "s_cbranch_vccnz 21f\n" \
    "s_waitcnt vmcnt(8)\n" \
    "11:\n" \
    "s_cmp_ge_u32 %[thr], 48\n" \
    "s_cbranch_scc1 401f\n" \
    "411:\n" \
    "s_sub_u32 %[c], %[vc], s73\n" \
    "s_sub_u32 %[n1], %[xi], %[u]\n" \
    "s_sub_u32 %[c], %[c], 2\n" \
    "s_sub_u32 %[n1], %[n1], 5\n" \
    "s_mov_b32 s62, %[sz]\n" \
    "s_or_b32 %[c], %[c], %[n1]\n" \
    "s_cbranch_scc0 602b\n" \
    "s_waitcnt lgkmcnt(14)\n" \
    "v_mfma_f32_16x16x128_f8f6f4 v[160:163], v[96:103], v[64:71], v[160:163] cbsz:1 blgp:1\n" \
    "s_add_u32 %[thr], %[u], 3\n" \
    "s_add_u32 %[n1], %[u], 5\n" \
    "v_mfma_f32_16x16x128_f8f6f4 v[192:195], v[96:103], v[72:79], v[192:195] cbsz:1 blgp:1\n" \
    "ds_read_b64_tr_b8 v[96:97], v8 offset:16384\n" \
    "ds_read_b64_tr_b8 v[98:99], v8 offset:17408\n" \
    "s_lshl_b32 %[so], %[n1], 14\n" \
    "v_mfma_f32_16x16x128_f8f6f4 v[164:167], v[104:111], v[64:71], v[164:167] cbsz:1 blgp:1\n" \
    "ds_read_b64_tr_b8 v[100:101], v8 offset:24576\n" \
    "ds_read_b64_tr_b8 v[102:103], v8 offset:25600\n" \
    "v_mfma_f32_16x16x128_f8f6f4 v[196:199], v[104:111], v[72:79], v[196:199] cbsz:1 blgp:1\n" \
    "ds_read_b64_tr_b8 v[104:105], v9 offset:16384\n" \
    "ds_read_b64_tr_b8 v[106:107], v9 offset:17408\n" \
    "s_waitcnt lgkmcnt(6)\n" \
    "v_mfma_f32_16x16x128_f8f6f4 v[168:171], v[112:119], v[64:71], v[168:171] cbsz:1 blgp:1\n" \
    "v_cmp_ge_u32 vcc, %[thr], v22\n" \
    "v_readfirstlane_b32 %[rv], v23\n" \
    "ds_read_b64_tr_b8 v[108:109], v9 offset:24576\n" \
    "ds_read_b64_tr_b8 v[110:111], v9 offset:25600\n" \
    "v_mfma_f32_16x16x128_f8f6f4 v[200:203], v[112:119], v[72:79], v[200:203] cbsz:1 blgp:1\n" \
    "ds_read_b64_tr_b8 v[112:113], v10 offset:16384\n" \
    "ds_read_b64_tr_b8 v[114:115], v10 offset:17408\n" \
    "522:\n" \
    "s_cmp_eq_u32 %[xi], %[n1]\n" \
    "s_cselect_b32 %[rv], %[rv], 0\n" \
    "v_mfma_f32_16x16x128_f8f6f4 v[172:175], v[120:127], v[64:71], v[172:175] cbsz:1 blgp:1\n" \
    "ds_read_b64_tr_b8 v[116:117], v10 offset:24576\n" \
    "ds_read_b64_tr_b8 v[118:119], v10 offset:25600\n" \
    "s_cmp_gt_u32 %[rv], %[n1]\n" \
    "s_cselect_b32 s62, %[sz], 0\n" \
    "v_mfma_f32_16x16x128_f8f6f4 v[204:207], v[120:127], v[72:79], v[204:207] cbsz:1 blgp:1\n" \
    "s_cselect_b32 s71, %[vc], s71\n" \
    "s_addc_u32 %[xi], %[xi], 0\n" \
    "ds_read_b64_tr_b8 v[120:121], v11 offset:16384\n" \
    "ds_read_b64_tr_b8 v[122:123], v11 offset:17408\n" \
    "buffer_load_dwordx4 v[48:51], v20, s[60:63], %[so] offen offset:0 sc0 sc1\n" \
    "s_waitcnt lgkmcnt(14)\n" \
    "v_mfma_f32_16x16x128_f8f6f4 v[176:179], v[128:135], v[64:71], v[176:179] cbsz:1 blgp:1\n" \
    "ds_read_b32 v23, v19 offset:16\n" \
    "ds_read_b64_tr_b8 v[124:125], v11 offset:24576\n" \
    "ds_read_b64_tr_b8 v[126:127], v11 offset:25600\n" \
    "buffer_load_dwordx4 v[52:55], v20, s[60:63], %[so] offen offset:1024 sc0 sc1\n" \
    "s_add_u32 %[vc], %[vc], 1\n" \
    "v_mfma_f32_16x16x128_f8f6f4 v[208:211], v[128:135], v[72:79], v[208:211] cbsz:1 blgp:1\n" \
    "ds_read_b64_tr_b8 v[128:129], v12 offset:16384\n" \
    "ds_read_b64_tr_b8 v[130:131], v12 offset:17408\n" \
    "buffer_load_dwordx4 v[56:59], v20, s[60:63], %[so] offen offset:2048 sc0 sc1\n" \
    "v_mfma_f32_16x16x128_f8f6f4 v[180:183], v[136:143], v[64:71], v[180:183] cbsz:1 blgp:1\n" \
    "ds_read_b64_tr_b8 v[132:133], v12 offset:24576\n" \
    "ds_read_b64_tr_b8 v[134:135], v12 offset:25600\n" \
    "buffer_load_dwordx4 v[60:63], v20, s[60:63], %[so] offen offset:3072 sc0 sc1\n" \
    "v_add_u32 v21, %[sb], v19\n" \
    "v_mfma_f32_16x16x128_f8f6f4 v[212:215], v[136:143], v[72:79], v[212:215] cbsz:1 blgp:1\n" \
    "ds_read_b64_tr_b8 v[136:137], v13 offset:16384\n" \
    "ds_read_b64_tr_b8 v[138:139], v13 offset:17408\n" \
    "s_cmp_le_u32 %[xi], %[n1]\n" \
    "s_cbranch_scc1 302f\n" \
    "312:\n" \
    "s_waitcnt lgkmcnt(14)\n" \
    "v_mfma_f32_16x16x128_f8f6f4 v[184:187], v[144:151], v[64:71], v[184:187] cbsz:1 blgp:1\n" \
    "ds_read_b64_tr_b8 v[140:141], v13 offset:24576\n" \
    "ds_read_b64_tr_b8 v[142:143], v13 offset:25600\n" \
    "s_sub_u32 %[c], %[vc], s73\n" \
    "s_add_u32 %[n1], %[u], 4\n" \
    "v_mfma_f32_16x16x128_f8f6f4 v[216:219], v[144:151], v[72:79], v[216:219] cbsz:1 blgp:1\n" \
    "ds_read_b64_tr_b8 v[144:145], v14 offset:16384\n" \
    "ds_read_b64_tr_b8 v[146:147], v14 offset:17408\n" \
    "s_sub_u32 %[c], %[c], 3\n" \
    "s_sub_u32 %[n1], %[xi], %[n1]\n" \
    "s_sub_u32 %[dl], %[sb], %[sa]\n" \
    "s_lshl_b32 %[dl], %[dl], 5\n" \
    "v_mfma_f32_16x16x128_f8f6f4 v[188:191], v[152:159], v[64:71], v[188:191] cbsz:1 blgp:1\n" \
    "ds_read_b64_tr_b8 v[148:149], v14 offset:24576\n" \
    "ds_read_b64_tr_b8 v[150:151], v14 offset:25600\n" \
    "s_andn2_b32 %[c], %[c], 3\n" \
    "s_lshr_b32 %[n1], %[n1], 31\n" \
    "v_mfma_f32_16x16x128_f8f6f4 v[220:223], v[152:159], v[72:79], v[220:223] cbsz:1 blgp:1\n" \
    "s_or_b32 %[c], %[c], %[n1]\n" \
    "ds_read_b64_tr_b8 v[152:153], v15 offset:16384\n" \
    "ds_read_b64_tr_b8 v[154:155], v15 offset:17408\n" \
    "ds_read_b64_tr_b8 v[156:157], v15 offset:24576\n" \
    "ds_read_b64_tr_b8 v[158:159], v15 offset:25600\n" \
    "ds_write_b32 v7, v22 offset:512\n" \
    "ds_read_b32 v22, v21 offset:0\n" \
    "ds_read_b32 v23, v19 offset:16\n" \
    "s_cmp_lg_u32 %[c], 0\n" \
    "s_cbranch_scc1 22f\n" \
    "s_cbranch_vccnz 22f\n" \
    "s_waitcnt vmcnt(8)\n" \
    "12:\n" \
    "s_cmp_ge_u32 %[thr], 48\n" \
    "s_cbranch_scc1 402f\n" \
    "412:\n" \
    "s_sub_u32 %[c], %[vc], s70\n" \
    "s_sub_u32 %[n1], %[xi], %[u]\n" \
    "s_sub_u32 %[c], %[c], 2\n" \
    "s_sub_u32 %[n1], %[n1], 6\n" \
    "s_mov_b32 s62, %[sz]\n" \
    "s_or_b32 %[c], %[c], %[n1]\n" \
    "s_cbranch_scc0 603b\n" \
    "s_waitcnt lgkmcnt(14)\n" \
    "v_mfma_f32_16x16x128_f8f6f4 v[160:163], v[96:103], v[80:87], v[160:163] cbsz:1 blgp:1\n" \
    "s_add_u32 %[thr], %[u], 4\n" \
    "s_cmp_lt_u32 %[thr], 64\n" \
    "s_cselect_b32 %[thr], %[thr], 0\n" \
    "s_add_u32 %[n1], %[u], 6\n" \
    "v_add_u32 v8, %[dl], v8\n" \
    "v_mfma_f32_16x16x128_f8f6f4 v[192:195], v[96:103], v[88:95], v[192:195] cbsz:1 blgp:1\n" \
    "ds_read_b64_tr_b8 v[96:97], v8 offset:0\n" \
    "ds_read_b64_tr_b8 v[98:99], v8 offset:1024\n" \
    "s_lshl_b32 %[so], %[n1], 14\n" \
    "v_mfma_f32_16x16x128_f8f6f4 v[164:167], v[104:111], v[80:87], v[164:167] cbsz:1 blgp:1\n" \
    "ds_read_b64_tr_b8 v[100:101], v8 offset:8192\n" \
    "ds_read_b64_tr_b8 v[102:103], v8 offset:9216\n" \
    "v_add_u32 v9, %[dl], v9\n" \
    "s_mov_b32 %[sa], %[sb]\n" \
    "s_add_u32 %[sb], %[sb], 1024\n" \
    "v_mfma_f32_16x16x128_f8f6f4 v[196:199], v[104:111], v[88:95], v[196:199] cbsz:1 blgp:1\n" \
    "ds_read_b64_tr_b8 v[104:105], v9 offset:0\n" \
    "ds_read_b64_tr_b8 v[106:107], v9 offset:1024\n" \
    "s_waitcnt lgkmcnt(6)\n" \
    "v_mfma_f32_16x16x128_f8f6f4 v[168:171], v[112:119], v[80:87], v[168:171] cbsz:1 blgp:1\n" \
    "v_cmp_ge_u32 vcc, %[thr], v22\n" \
    "v_readfirstlane_b32 %[rv], v23\n" \
    "ds_read_b64_tr_b8 v[108:109], v9 offset:8192\n" \
    "ds_read_b64_tr_b8 v[110:111], v9 offset:9216\n" \
    "v_add_u32 v10, %[dl], v10\n" \
    "v_mfma_f32_16x16x128_f8f6f4 v[200:203], v[112:119], v[88:95], v[200:203] cbsz:1 blgp:1\n" \
    "ds_read_b64_tr_b8 v[112:113], v10 offset:0\n" \
    "ds_read_b64_tr_b8 v[114:115], v10 offset:1024\n" \
    "523:\n" \
    "s_cmp_eq_u32 %[xi], %[n1]\n" \
    "s_cselect_b32 %[rv], %[rv], 0\n" \
    "v_mfma_f32_16x16x128_f8f6f4 v[172:175], v[120:127], v[80:87], v[172:175] cbsz:1 blgp:1\n" \
    "ds_read_b64_tr_b8 v[116:117], v10 offset:8192\n" \
    "ds_read_b64_tr_b8 v[118:119], v10 offset:9216\n" \
    "v_add_u32 v11, %[dl], v11\n" \
    "s_cmp_gt_u32 %[rv], %[n1]\n" \
    "s_cselect_b32 s62, %[sz], 0\n" \
    "v_mfma_f32_16x16x128_f8f6f4 v[204:207], v[120:127], v[88:95], v[204:207] cbsz:1 blgp:1\n" \
    "s_cselect_b32 s72, %[vc], s72\n" \
    "s_addc_u32 %[xi], %[xi], 0\n" \
    "ds_read_b64_tr_b8 v[120:121], v11 offset:0\n" \
    "ds_read_b64_tr_b8 v[122:123], v11 offset:1024\n" \
    "buffer_load_dwordx4 v[64:67], v20, s[60:63], %[so] offen offset:0 sc0 sc1\n" \
    "s_waitcnt lgkmcnt(14)\n" \
    "v_mfma_f32_16x16x128_f8f6f4 v[176:179], v[128:135], v[80:87], v[176:179] cbsz:1 blgp:1\n" \
    "ds_read_b32 v23, v19 offset:16\n" \
    "ds_read_b64_tr_b8 v[124:125], v11 offset:8192\n" \
    "ds_read_b64_tr_b8 v[126:127], v11 offset:9216\n" \
    "v_add_u32 v12, %[dl], v12\n" \
    "buffer_load_dwordx4 v[68:71], v20, s[60:63], %[so] offen offset:1024 sc0 sc1\n" \
    "s_add_u32 %[vc], %[vc], 1\n" \
    "v_mfma_f32_16x16x128_f8f6f4 v[208:211], v[128:135], v[88:95], v[208:211] cbsz:1 blgp:1\n" \
    "ds_read_b64_tr_b8 v[128:129], v12 offset:0\n" \
    "ds_read_b64_tr_b8 v[130:131], v12 offset:1024\n" \
    "buffer_load_dwordx4 v[72:75], v20, s[60:63], %[so] offen offset:2048 sc0 sc1\n" \
    "s_cmp_eq_u32 %[sb], 3072\n" \
    "s_cselect_b32 %[sb], 0, %[sb]\n" \
    "v_mfma_f32_16x16x128_f8f6f4 v[180:183], v[136:143], v[80:87], v[180:183] cbsz:1 blgp:1\n" \
    "ds_read_b64_tr_b8 v[132:133], v12 offset:8192\n" \
    "ds_read_b64_tr_b8 v[134:135], v12 offset:9216\n" \
    "v_add_u32 v13, %[dl], v13\n" \
    "buffer_load_dwordx4 v[76:79], v20, s[60:63], %[so] offen offset:3072 sc0 sc1\n" \
    "v_add_u32 v7, %[sa], v18\n" \
    "v_mfma_f32_16x16x128_f8f6f4 v[212:215], v[136:143], v[88:95], v[212:215] cbsz:1 blgp:1\n" \
    "ds_read_b64_tr_b8 v[136:137], v13 offset:0\n" \
    "ds_read_b64_tr_b8 v[138:139], v13 offset:1024\n" \
    "s_cmp_le_u32 %[xi], %[n1]\n" \
    "s_cbranch_scc1 303f\n" \
    "313:\n" \
    "s_waitcnt lgkmcnt(14)\n" \
    "v_mfma_f32_16x16x128_f8f6f4 v[184:187], v[144:151], v[80:87], v[184:187] cbsz:1 blgp:1\n" \
    "ds_read_b64_tr_b8 v[140:141], v13 offset:8192\n" \
    "ds_read_b64_tr_b8 v[142:143], v13 offset:9216\n" \
    "v_add_u32 v14, %[dl], v14\n" \
    "s_sub_u32 %[c], %[vc], s70\n" \
    "s_add_u32 %[n1], %[u], 5\n" \
    "v_mfma_f32_16x16x128_f8f6f4 v[216:219], v[144:151], v[88:95], v[216:219] cbsz:1 blgp:1\n" \
    "ds_read_b64_tr_b8 v[144:145], v14 offset:0\n" \
    "ds_read_b64_tr_b8 v[146:147], v14 offset:1024\n" \
    "s_sub_u32 %[c], %[c], 3\n" \
    "s_sub_u32 %[n1], %[xi], %[n1]\n" \
    "v_mfma_f32_16x16x128_f8f6f4 v[188:191], v[152:159], v[80:87], v[188:191] cbsz:1 blgp:1\n" \
    "ds_read_b64_tr_b8 v[148:149], v14 offset:8192\n" \
    "ds_read_b64_tr_b8 v[150:151], v14 offset:9216\n" \
    "v_add_u32 v15, %[dl], v15\n" \
    "s_andn2_b32 %[c], %[c], 3\n" \
    "s_lshr_b32 %[n1], %[n1], 31\n" \
    "v_mfma_f32_16x16x128_f8f6f4 v[220:223], v[152:159], v[88:95], v[220:223] cbsz:1 blgp:1\n" \
    "s_or_b32 %[c], %[c], %[n1]\n" \
    "ds_read_b64_tr_b8 v[152:153], v15 offset:0\n" \
    "ds_read_b64_tr_b8 v[154:155], v15 offset:1024\n" \
    "ds_read_b64_tr_b8 v[156:157], v15 offset:8192\n" \
    "ds_read_b64_tr_b8 v[158:159], v15 offset:9216\n" \
    "ds_write_b32 v7, v22 offset:0\n" \
    "ds_read_b32 v22, v21 offset:512\n" \
    "ds_read_b32 v23, v19 offset:16\n" \
    "s_cmp_lg_u32 %[c], 0\n" \
    "s_cbranch_scc1 23f\n" \
    "s_cbranch_vccnz 23f\n" \
    "s_waitcnt vmcnt(8)\n" \
    "13:\n" \
    "s_cmp_ge_u32 %[thr], 48\n" \
    "s_cbranch_scc1 403f\n" \
    "413:\n" \
    "s_sub_u32 %[c], %[vc], s71\n" \
    "s_sub_u32 %[n1], %[xi], %[u]\n" \
    "s_sub_u32 %[c], %[c], 2\n" \
    "s_sub_u32 %[n1], %[n1], 7\n" \
    "s_mov_b32 s62, %[sz]\n" \
    "s_or_b32 %[c], %[c], %[n1]\n" \
    "s_cbranch_scc0 69b\n" \
    "s_add_u32 %[u], %[u], 4\n" \
    "s_cmp_lt_u32 %[u], %[uend]\n" \
    "s_cbranch_scc1 0b\n" \
    "s_branch 9f\n" \
    "20:\n" \
    "s_add_u32 %[c], %[u], 0\n" \
    "s_cbranch_vccz 30f\n" \
    "v_add_u32 v7, %[sa], v18\n" \
    "s_waitcnt lgkmcnt(0)\n" \
    "v_add_u32 v23, %[sa], v19\n" \
    "40:\n" \
    "ds_read_b32 v21, v23 offset:512\n" \
    "s_waitcnt lgkmcnt(0)\n" \
    "v_cmp_ge_u32 vcc, %[thr], v21\n" \
    "s_cbranch_vccnz 40b\n" \
    "ds_read_b64_tr_b8 v[96:97], v8 offset:16384\n" \
    "ds_read_b64_tr_b8 v[98:99], v8 offset:17408\n" \
    "ds_read_b64_tr_b8 v[100:101], v8 offset:24576\n" \
    "ds_read_b64_tr_b8 v[102:103], v8 offset:25600\n" \
    "ds_read_b64_tr_b8 v[104:105], v9 offset:16384\n" \
    "ds_read_b64_tr_b8 v[106:107], v9 offset:17408\n" \
    "ds_read_b64_tr_b8 v[108:109], v9 offset:24576\n" \
    "ds_read_b64_tr_b8 v[110:111], v9 offset:25600\n" \
    "ds_read_b64_tr_b8 v[112:113], v10 offset:16384\n" \
    "ds_read_b64_tr_b8 v[114:115], v10 offset:17408\n" \
    "ds_read_b64_tr_b8 v[116:117], v10 offset:24576\n" \
    "ds_read_b64_tr_b8 v[118:119], v10 offset:25600\n" \
    "ds_read_b64_tr_b8 v[120:121], v11 offset:16384\n" \
    "ds_read_b64_tr_b8 v[122:123], v11 offset:17408\n" \
    "ds_read_b64_tr_b8 v[124:125], v11 offset:24576\n" \
    "ds_read_b64_tr_b8 v[126:127], v11 offset:25600\n" \
    "ds_read_b64_tr_b8 v[128:129], v12 offset:16384\n" \
    "ds_read_b64_tr_b8 v[130:131], v12 offset:17408\n" \
    "ds_read_b64_tr_b8 v[132:133], v12 offset:24576\n" \
    "ds_read_b64_tr_b8 v[134:135], v12 offset:25600\n" \
    "ds_read_b64_tr_b8 v[136:137], v13 offset:16384\n" \
    "ds_read_b64_tr_b8 v[138:139], v13 offset:17408\n" \
    "ds_read_b64_tr_b8 v[140:141], v13 offset:24576\n" \
    "ds_read_b64_tr_b8 v[142:143], v13 offset:25600\n" \
    "ds_read_b64_tr_b8 v[144:145], v14 offset:16384\n" \
    "ds_read_b64_tr_b8 v[146:147], v14 offset:17408\n" \
    "ds_read_b64_tr_b8 v[148:149], v14 offset:24576\n" \
    "ds_read_b64_tr_b8 v[150:151], v14 offset:25600\n" \
    "ds_read_b64_tr_b8 v[152:153], v15 offset:16384\n" \
    "ds_read_b64_tr_b8 v[154:155], v15 offset:17408\n" \
    "ds_read_b64_tr_b8 v[156:157], v15 offset:24576\n" \
    "ds_read_b64_tr_b8 v[158:159], v15 offset:25600\n" \
    "ds_write_b32 v7, v21 offset:512\n" \
    "30:\n" \
    "s_mov_b32 s62, %[sz]\n" \
    "s_add_u32 %[n1], %[c], 1\n" \
    "s_cmp_ge_u32 %[n1], 64\n" \
    "s_cbranch_scc1 10b\n" \
    "s_cmp_eq_u32 %[xi], %[n1]\n" \
    "s_cselect_b32 %[n1], %[n1], 0\n" \
    "60:\n" \
    "ds_read_b32 v21, v19 offset:16\n" \
    "s_waitcnt lgkmcnt(0)\n" \
    "v_readfirstlane_b32 %[rv], v21\n" \
    "s_cmp_le_u32 %[rv], %[n1]\n" \
    "s_cbranch_scc1 60b\n" \
    "s_add_u32 %[n1], %[c], 1\n" \
    "s_cmp_lg_u32 %[xi], %[n1]\n" \
    "s_cbranch_scc1 3301f\n" \
    "s_cmp_le_u32 %[rv], %[xi]\n" \
    "s_cbranch_scc1 80f\n" \
    "s_lshl_b32 %[so], %[xi], 14\n" \
    "buffer_load_dwordx4 v[48:51], v20, s[60:63], %[so] offen offset:0 sc0 sc1\n" \
    "buffer_load_dwordx4 v[52:55], v20, s[60:63], %[so] offen offset:1024 sc0 sc1\n" \
    "buffer_load_dwordx4 v[56:59], v20, s[60:63], %[so] offen offset:2048 sc0 sc1\n" \
    "buffer_load_dwordx4 v[60:63], v20, s[60:63], %[so] offen offset:3072 sc0 sc1\n" \
    "s_mov_b32 s71, %[vc]\n" \
    "s_add_u32 %[vc], %[vc], 1\n" \
    "s_add_u32 %[xi], %[xi], 1\n" \
    "3301:\n" \
    "s_add_u32 %[n1], %[c], 2\n" \
    "s_cmp_lg_u32 %[xi], %[n1]\n" \
    "s_cbranch_scc1 3302f\n" \
    "s_cmp_le_u32 %[rv], %[xi]\n" \
    "s_cbranch_scc1 80f\n" \
    "s_lshl_b32 %[so], %[xi], 14\n" \
    "buffer_load_dwordx4 v[64:67], v20, s[60:63], %[so] offen offset:0 sc0 sc1\n" \
    "buffer_load_dwordx4 v[68:71], v20, s[60:63], %[so] offen offset:1024 sc0 sc1\n" \
    "buffer_load_dwordx4 v[72:75], v20, s[60:63], %[so] offen offset:2048 sc0 sc1\n" \
    "buffer_load_dwordx4 v[76:79], v20, s[60:63], %[so] offen offset:3072 sc0 sc1\n" \
    "s_mov_b32 s72, %[vc]\n" \
    "s_add_u32 %[vc], %[vc], 1\n" \
    "s_add_u32 %[xi], %[xi], 1\n" \
    "3302:\n" \
    "s_add_u32 %[n1], %[c], 3\n" \
    "s_cmp_lg_u32 %[xi], %[n1]\n" \
    "s_cbranch_scc1 3303f\n" \
    "s_cmp_le_u32 %[rv], %[xi]\n" \
    "s_cbranch_scc1 80f\n" \
    "s_lshl_b32 %[so], %[xi], 14\n" \
    "buffer_load_dwordx4 v[80:83], v20, s[60:63], %[so] offen offset:0 sc0 sc1\n" \
    "buffer_load_dwordx4 v[84:87], v20, s[60:63], %[so] offen offset:1024 sc0 sc1\n" \
    "buffer_load_dwordx4 v[88:91], v20, s[60:63], %[so] offen offset:2048 sc0 sc1\n" \
    "buffer_load_dwordx4 v[92:95], v20, s[60:63], %[so] offen offset:3072 sc0 sc1\n" \
    "s_mov_b32 s73, %[vc]\n" \
    "s_add_u32 %[vc], %[vc], 1\n" \
    "s_add_u32 %[xi], %[xi], 1\n" \
    "3303:\n" \
    "80:\n" \
    "v_add_u32 v21, %[sb], v19\n" \
    "s_sub_u32 %[c], %[vc], s71\n" \
    "ds_read_b32 v22, v21 offset:0\n" \
    "ds_read_b32 v23, v19 offset:16\n" \
    "s_waitcnt vmcnt(24)\n" \
    "s_cmp_ge_u32 %[c], 7\n" \
    "s_cbranch_scc1 10b\n" \
    "s_waitcnt vmcnt(12)\n" \
    "s_cmp_ge_u32 %[c], 4\n" \
    "s_cbranch_scc1 10b\n" \
    "s_waitcnt vmcnt(8)\n" \
    "s_cmp_ge_u32 %[c], 3\n" \
    "s_cbranch_scc1 10b\n" \
    "s_waitcnt vmcnt(4)\n" \
    "s_cmp_ge_u32 %[c], 2\n" \
    "s_cbranch_scc1 10b\n" \
    "s_waitcnt vmcnt(0)\n" \
    "s_branch 10b\n" \
    "300:\n" \
    "s_add_u32 %[c], %[u], 0\n" \
    "s_mov_b32 s62, %[sz]\n" \
    "s_waitcnt lgkmcnt(6)\n" \
    "v_readfirstlane_b32 %[rv], v23\n" \
    "s_add_u32 %[n1], %[c], 1\n" \
    "s_cmp_lg_u32 %[xi], %[n1]\n" \
    "s_cbranch_scc1 3201f\n" \
    "s_cmp_le_u32 %[rv], %[xi]\n" \
    "s_cbranch_scc1 310b\n" \
    "s_lshl_b32 %[so], %[xi], 14\n" \
    "buffer_load_dwordx4 v[48:51], v20, s[60:63], %[so] offen offset:0 sc0 sc1\n" \
    "buffer_load_dwordx4 v[52:55], v20, s[60:63], %[so] offen offset:1024 sc0 sc1\n" \
    "buffer_load_dwordx4 v[56:59], v20, s[60:63], %[so] offen offset:2048 sc0 sc1\n" \
    "buffer_load_dwordx4 v[60:63], v20, s[60:63], %[so] offen offset:3072 sc0 sc1\n" \
    "s_mov_b32 s71, %[vc]\n" \
    "s_add_u32 %[vc], %[vc], 1\n" \
    "s_add_u32 %[xi], %[xi], 1\n" \
    "3201:\n" \
    "s_add_u32 %[n1], %[c], 2\n" \
    "s_cmp_lg_u32 %[xi], %[n1]\n" \
    "s_cbranch_scc1 3202f\n" \
    "s_cmp_le_u32 %[rv], %[xi]\n" \
    "s_cbranch_scc1 310b\n" \
    "s_lshl_b32 %[so], %[xi], 14\n" \
    "buffer_load_dwordx4 v[64:67], v20, s[60:63], %[so] offen offset:0 sc0 sc1\n" \
    "buffer_load_dwordx4 v[68:71], v20, s[60:63], %[so] offen offset:1024 sc0 sc1\n" \
    "buffer_load_dwordx4 v[72:75], v20, s[60:63], %[so] offen offset:2048 sc0 sc1\n" \
    "buffer_load_dwordx4 v[76:79], v20, s[60:63], %[so] offen offset:3072 sc0 sc1\n" \
    "s_mov_b32 s72, %[vc]\n" \
    "s_add_u32 %[vc], %[vc], 1\n" \
    "s_add_u32 %[xi], %[xi], 1\n" \
    "3202:\n" \
    "s_add_u32 %[n1], %[c], 3\n" \
    "s_cmp_lg_u32 %[xi], %[n1]\n" \
    "s_cbranch_scc1 3203f\n" \
    "s_cmp_le_u32 %[rv], %[xi]\n" \
    "s_cbranch_scc1 310b\n" \
    "s_lshl_b32 %[so], %[xi], 14\n" \
    "buffer_load_dwordx4 v[80:83], v20, s[60:63], %[so] offen offset:0 sc0 sc1\n" \
    "buffer_load_dwordx4 v[84:87], v20, s[60:63], %[so] offen offset:1024 sc0 sc1\n" \
    "buffer_load_dwordx4 v[88:91], v20, s[60:63], %[so] offen offset:2048 sc0 sc1\n" \
    "buffer_load_dwordx4 v[92:95], v20, s[60:63], %[so] offen offset:3072 sc0 sc1\n" \
    "s_mov_b32 s73, %[vc]\n" \
    "s_add_u32 %[vc], %[vc], 1\n" \
    "s_add_u32 %[xi], %[xi], 1\n" \
    "3203:\n" \
    "s_branch 310b\n" \
    "400:\n" \
    "v_max3_u32 v24, v48, v49, v50\n" \
    "v_max3_u32 v25, v51, v52, v53\n" \
    "v_max3_u32 v26, v54, v55, v56\n" \
    "v_max3_u32 v27, v57, v58, v59\n" \
    "v_max3_u32 v28, v60, v61, v62\n" \
    "v_max3_u32 v24, v24, v25, v63\n" \
    "v_max3_u32 v26, v26, v27, v28\n" \
    "v_max_u32 v24, v24, v26\n" \
    "v_cmp_eq_u32 vcc, -1, v24\n" \
    "s_cbranch_vccz 410b\n" \
    "450:\n" \
    "ds_read_b32 v24, v19 offset:24\n" \
    "s_mov_b32 s62, %[sz]\n" \
    "s_mov_b32 %[xi], %[thr]\n" \
    "s_sub_u32 %[c], %[thr], 1\n" \
    "s_waitcnt lgkmcnt(0)\n" \
    "v_readfirstlane_b32 s76, v24\n" \
    "v_readfirstlane_b32 %[rv], v23\n" \
    "s_add_u32 %[n1], %[c], 1\n" \
    "s_cmp_lg_u32 %[xi], %[n1]\n" \
    "s_cbranch_scc1 4201f\n" \
    "s_cmp_le_u32 %[rv], %[xi]\n" \
    "s_cbranch_scc1 430f\n" \
    "s_lshl_b32 %[so], %[xi], 14\n" \
    "buffer_load_dwordx4 v[48:51], v20, s[60:63], %[so] offen offset:0 sc0 sc1\n" \
    "buffer_load_dwordx4 v[52:55], v20, s[60:63], %[so] offen offset:1024 sc0 sc1\n" \
    "buffer_load_dwordx4 v[56:59], v20, s[60:63], %[so] offen offset:2048 sc0 sc1\n" \
    "buffer_load_dwordx4 v[60:63], v20, s[60:63], %[so] offen offset:3072 sc0 sc1\n" \
    "s_mov_b32 s71, %[vc]\n" \
    "s_add_u32 %[vc], %[vc], 1\n" \
    "s_add_u32 %[xi], %[xi], 1\n" \
    "4201:\n" \
    "s_add_u32 %[n1], %[c], 2\n" \
    "s_cmp_lg_u32 %[xi], %[n1]\n" \
    "s_cbranch_scc1 4202f\n" \
    "s_cmp_le_u32 %[rv], %[xi]\n" \
    "s_cbranch_scc1 430f\n" \
    "s_lshl_b32 %[so], %[xi], 14\n" \
    "buffer_load_dwordx4 v[64:67], v20, s[60:63], %[so] offen offset:0 sc0 sc1\n" \
    "buffer_load_dwordx4 v[68:71], v20, s[60:63], %[so] offen offset:1024 sc0 sc1\n" \
    "buffer_load_dwordx4 v[72:75], v20, s[60:63], %[so] offen offset:2048 sc0 sc1\n" \
    "buffer_load_dwordx4 v[76:79], v20, s[60:63], %[so] offen offset:3072 sc0 sc1\n" \
    "s_mov_b32 s72, %[vc]\n" \
    "s_add_u32 %[vc], %[vc], 1\n" \
    "s_add_u32 %[xi], %[xi], 1\n" \
    "4202:\n" \
    "s_add_u32 %[n1], %[c], 3\n" \
    "s_cmp_lg_u32 %[xi], %[n1]\n" \
    "s_cbranch_scc1 4203f\n" \
    "s_cmp_le_u32 %[rv], %[xi]\n" \
    "s_cbranch_scc1 430f\n" \
    "s_lshl_b32 %[so], %[xi], 14\n" \
    "buffer_load_dwordx4 v[80:83], v20, s[60:63], %[so] offen offset:0 sc0 sc1\n" \
    "buffer_load_dwordx4 v[84:87], v20, s[60:63], %[so] offen offset:1024 sc0 sc1\n" \
    "buffer_load_dwordx4 v[88:91], v20, s[60:63], %[so] offen offset:2048 sc0 sc1\n" \
    "buffer_load_dwordx4 v[92:95], v20, s[60:63], %[so] offen offset:3072 sc0 sc1\n" \
    "s_mov_b32 s73, %[vc]\n" \
    "s_add_u32 %[vc], %[vc], 1\n" \
    "s_add_u32 %[xi], %[xi], 1\n" \
    "4203:\n" \
    "430:\n" \
    "s_sub_u32 %[c], %[vc], s71\n" \
    "s_waitcnt vmcnt(8)\n" \
    "s_cmp_ge_u32 %[c], 3\n" \
    "s_cbranch_scc1 440f\n" \
    "s_waitcnt vmcnt(4)\n" \
    "s_cmp_ge_u32 %[c], 2\n" \
    "s_cbranch_scc1 440f\n" \
    "s_waitcnt vmcnt(0)\n" \
    "440:\n" \
    "s_cmp_gt_u32 s76, %[thr]\n" \
    "s_cbranch_scc1 410b\n" \
    "s_branch 400b\n" \
    "21:\n" \
    "s_add_u32 %[c], %[u], 1\n" \
    "s_cbranch_vccz 31f\n" \
    "v_add_u32 v7, %[sa], v18\n" \
    "s_waitcnt lgkmcnt(0)\n" \
    "v_add_u32 v23, %[sa], v19\n" \
    "41:\n" \
    "ds_read_b32 v21, v23 offset:0\n" \
    "s_waitcnt lgkmcnt(0)\n" \
    "v_cmp_ge_u32 vcc, %[thr], v21\n" \
    "s_cbranch_vccnz 41b\n" \
    "ds_read_b64_tr_b8 v[96:97], v8 offset:0\n" \
    "ds_read_b64_tr_b8 v[98:99], v8 offset:1024\n" \
    "ds_read_b64_tr_b8 v[100:101], v8 offset:8192\n" \
    "ds_read_b64_tr_b8 v[102:103], v8 offset:9216\n" \
    "ds_read_b64_tr_b8 v[104:105], v9 offset:0\n" \
    "ds_read_b64_tr_b8 v[106:107], v9 offset:1024\n" \
    "ds_read_b64_tr_b8 v[108:109], v9 offset:8192\n" \
    "ds_read_b64_tr_b8 v[110:111], v9 offset:9216\n" \
    "ds_read_b64_tr_b8 v[112:113], v10 offset:0\n" \
    "ds_read_b64_tr_b8 v[114:115], v10 offset:1024\n" \
    "ds_read_b64_tr_b8 v[116:117], v10 offset:8192\n" \
    "ds_read_b64_tr_b8 v[118:119], v10 offset:9216\n" \
    "ds_read_b64_tr_b8 v[120:121], v11 offset:0\n" \
    "ds_read_b64_tr_b8 v[122:123], v11 offset:1024\n" \
    "ds_read_b64_tr_b8 v[124:125], v11 offset:8192\n" \
    "ds_read_b64_tr_b8 v[126:127], v11 offset:9216\n" \
    "ds_read_b64_tr_b8 v[128:129], v12 offset:0\n" \
    "ds_read_b64_tr_b8 v[130:131], v12 offset:1024\n" \
    "ds_read_b64_tr_b8 v[132:133], v12 offset:8192\n" \
    "ds_read_b64_tr_b8 v[134:135], v12 offset:9216\n" \
    "ds_read_b64_tr_b8 v[136:137], v13 offset:0\n" \
    "ds_read_b64_tr_b8 v[138:139], v13 offset:1024\n" \
    "ds_read_b64_tr_b8 v[140:141], v13 offset:8192\n" \
    "ds_read_b64_tr_b8 v[142:143], v13 offset:9216\n" \
    "ds_read_b64_tr_b8 v[144:145], v14 offset:0\n" \
    "ds_read_b64_tr_b8 v[146:147], v14 offset:1024\n" \
    "ds_read_b64_tr_b8 v[148:149], v14 offset:8192\n" \
    "ds_read_b64_tr_b8 v[150:151], v14 offset:9216\n" \
    "ds_read_b64_tr_b8 v[152:153], v15 offset:0\n" \
    "ds_read_b64_tr_b8 v[154:155], v15 offset:1024\n" \
    "ds_read_b64_tr_b8 v[156:157], v15 offset:8192\n" \
    "ds_read_b64_tr_b8 v[158:159], v15 offset:9216\n" \
    "ds_write_b32 v7, v21 offset:0\n" \
    "31:\n" \
    "s_mov_b32 s62, %[sz]\n" \
    "s_add_u32 %[n1], %[c], 1\n" \
    "s_cmp_ge_u32 %[n1], 64\n" \
    "s_cbranch_scc1 11b\n" \
    "s_cmp_eq_u32 %[xi], %[n1]\n" \
    "s_cselect_b32 %[n1], %[n1], 0\n" \
    "61:\n" \
    "ds_read_b32 v21, v19 offset:16\n" \
    "s_waitcnt lgkmcnt(0)\n" \
    "v_readfirstlane_b32 %[rv], v21\n" \
    "s_cmp_le_u32 %[rv], %[n1]\n" \
    "s_cbranch_scc1 61b\n" \
    "s_add_u32 %[n1], %[c], 1\n" \
    "s_cmp_lg_u32 %[xi], %[n1]\n" \
    "s_cbranch_scc1 3311f\n" \
    "s_cmp_le_u32 %[rv], %[xi]\n" \
    "s_cbranch_scc1 81f\n" \
    "s_lshl_b32 %[so], %[xi], 14\n" \
    "buffer_load_dwordx4 v[64:67], v20, s[60:63], %[so] offen offset:0 sc0 sc1\n" \
    "buffer_load_dwordx4 v[68:71], v20, s[60:63], %[so] offen offset:1024 sc0 sc1\n" \
    "buffer_load_dwordx4 v[72:75], v20, s[60:63], %[so] offen offset:2048 sc0 sc1\n" \
    "buffer_load_dwordx4 v[76:79], v20, s[60:63], %[so] offen offset:3072 sc0 sc1\n" \
    "s_mov_b32 s72, %[vc]\n" \
    "s_add_u32 %[vc], %[vc], 1\n" \
    "s_add_u32 %[xi], %[xi], 1\n" \
    "3311:\n" \
    "s_add_u32 %[n1], %[c], 2\n" \
    "s_cmp_lg_u32 %[xi], %[n1]\n" \
    "s_cbranch_scc1 3312f\n" \
    "s_cmp_le_u32 %[rv], %[xi]\n" \
    "s_cbranch_scc1 81f\n" \
    "s_lshl_b32 %[so], %[xi], 14\n" \
    "buffer_load_dwordx4 v[80:83], v20, s[60:63], %[so] offen offset:0 sc0 sc1\n" \
    "buffer_load_dwordx4 v[84:87], v20, s[60:63], %[so] offen offset:1024 sc0 sc1\n" \
    "buffer_load_dwordx4 v[88:91], v20, s[60:63], %[so] offen offset:2048 sc0 sc1\n" \
    "buffer_load_dwordx4 v[92:95], v20, s[60:63], %[so] offen offset:3072 sc0 sc1\n" \
    "s_mov_b32 s73, %[vc]\n" \
    "s_add_u32 %[vc], %[vc], 1\n" \
    "s_add_u32 %[xi], %[xi], 1\n" \
    "3312:\n" \
    "s_add_u32 %[n1], %[c], 3\n" \
    "s_cmp_lg_u32 %[xi], %[n1]\n" \
    "s_cbranch_scc1 3313f\n" \
    "s_cmp_le_u32 %[rv], %[xi]\n" \
    "s_cbranch_scc1 81f\n" \
    "s_lshl_b32 %[so], %[xi], 14\n" \
    "buffer_load_dwordx4 v[32:35], v20, s[60:63], %[so] offen offset:0 sc0 sc1\n" \
    "buffer_load_dwordx4 v[36:39], v20, s[60:63], %[so] offen offset:1024 sc0 sc1\n" \
    "buffer_load_dwordx4 v[40:43], v20, s[60:63], %[so] offen offset:2048 sc0 sc1\n" \
    "buffer_load_dwordx4 v[44:47], v20, s[60:63], %[so] offen offset:3072 sc0 sc1\n" \
    "s_mov_b32 s70, %[vc]\n" \
    "s_add_u32 %[vc], %[vc], 1\n" \
    "s_add_u32 %[xi], %[xi], 1\n" \
    "3313:\n" \
    "81:\n" \
    "v_add_u32 v21, %[sa], v19\n" \
    "s_sub_u32 %[c], %[vc], s72\n" \
    "ds_read_b32 v22, v21 offset:512\n" \
    "ds_read_b32 v23, v19 offset:16\n" \
    "s_waitcnt vmcnt(24)\n" \
    "s_cmp_ge_u32 %[c], 7\n" \
    "s_cbranch_scc1 11b\n" \
    "s_waitcnt vmcnt(12)\n" \
    "s_cmp_ge_u32 %[c], 4\n" \
    "s_cbranch_scc1 11b\n" \
    "s_waitcnt vmcnt(8)\n" \
    "s_cmp_ge_u32 %[c], 3\n" \
    "s_cbranch_scc1 11b\n" \
    "s_waitcnt vmcnt(4)\n" \
    "s_cmp_ge_u32 %[c], 2\n" \
    "s_cbranch_scc1 11b\n" \
    "s_waitcnt vmcnt(0)\n" \
    "s_branch 11b\n" \
    "301:\n" \
    "s_add_u32 %[c], %[u], 1\n" \
    "s_mov_b32 s62, %[sz]\n" \
    "s_waitcnt lgkmcnt(6)\n" \
    "v_readfirstlane_b32 %[rv], v23\n" \
    "s_add_u32 %[n1], %[c], 1\n" \
    "s_cmp_lg_u32 %[xi], %[n1]\n" \
    "s_cbranch_scc1 3211f\n" \
    "s_cmp_le_u32 %[rv], %[xi]\n" \
    "s_cbranch_scc1 311b\n" \
    "s_lshl_b32 %[so], %[xi], 14\n" \
    "buffer_load_dwordx4 v[64:67], v20, s[60:63], %[so] offen offset:0 sc0 sc1\n" \
    "buffer_load_dwordx4 v[68:71], v20, s[60:63], %[so] offen offset:1024 sc0 sc1\n" \
    "buffer_load_dwordx4 v[72:75], v20, s[60:63], %[so] offen offset:2048 sc0 sc1\n" \
    "buffer_load_dwordx4 v[76:79], v20, s[60:63], %[so] offen offset:3072 sc0 sc1\n" \
    "s_mov_b32 s72, %[vc]\n" \
    "s_add_u32 %[vc], %[vc], 1\n" \
    "s_add_u32 %[xi], %[xi], 1\n" \
    "3211:\n" \
    "s_add_u32 %[n1], %[c], 2\n" \
    "s_cmp_lg_u32 %[xi], %[n1]\n" \
    "s_cbranch_scc1 3212f\n" \
    "s_cmp_le_u32 %[rv], %[xi]\n" \
    "s_cbranch_scc1 311b\n" \
    "s_lshl_b32 %[so], %[xi], 14\n" \
    "buffer_load_dwordx4 v[80:83], v20, s[60:63], %[so] offen offset:0 sc0 sc1\n" \
    "buffer_load_dwordx4 v[84:87], v20, s[60:63], %[so] offen offset:1024 sc0 sc1\n" \
    "buffer_load_dwordx4 v[88:91], v20, s[60:63], %[so] offen offset:2048 sc0 sc1\n" \
    "buffer_load_dwordx4 v[92:95], v20, s[60:63], %[so] offen offset:3072 sc0 sc1\n" \
    "s_mov_b32 s73, %[vc]\n" \
    "s_add_u32 %[vc], %[vc], 1\n" \
    "s_add_u32 %[xi], %[xi], 1\n" \
    "3212:\n" \
    "s_add_u32 %[n1], %[c], 3\n" \
    "s_cmp_lg_u32 %[xi], %[n1]\n" \
    "s_cbranch_scc1 3213f\n" \
    "s_cmp_le_u32 %[rv], %[xi]\n" \
    "s_cbranch_scc1 311b\n" \
    "s_lshl_b32 %[so], %[xi], 14\n" \
    "buffer_load_dwordx4 v[32:35], v20, s[60:63], %[so] offen offset:0 sc0 sc1\n" \
    "buffer_load_dwordx4 v[36:39], v20, s[60:63], %[so] offen offset:1024 sc0 sc1\n" \
    "buffer_load_dwordx4 v[40:43], v20, s[60:63], %[so] offen offset:2048 sc0 sc1\n" \
    "buffer_load_dwordx4 v[44:47], v20, s[60:63], %[so] offen offset:3072 sc0 sc1\n" \
    "s_mov_b32 s70, %[vc]\n" \
    "s_add_u32 %[vc], %[vc], 1\n" \
    "s_add_u32 %[xi], %[xi], 1\n" \
    "3213:\n" \
    "s_branch 311b\n" \
    "401:\n" \
    "v_max3_u32 v24, v64, v65, v66\n" \
    "v_max3_u32 v25, v67, v68, v69\n" \
    "v_max3_u32 v26, v70, v71, v72\n" \
    "v_max3_u32 v27, v73, v74, v75\n" \
    "v_max3_u32 v28, v76, v77, v78\n" \
    "v_max3_u32 v24, v24, v25, v79\n" \
    "v_max3_u32 v26, v26, v27, v28\n" \
    "v_max_u32 v24, v24, v26\n" \
    "v_cmp_eq_u32 vcc, -1, v24\n" \
    "s_cbranch_vccz 411b\n" \
    "451:\n" \
    "ds_read_b32 v24, v19 offset:24\n" \
    "s_mov_b32 s62, %[sz]\n" \
    "s_mov_b32 %[xi], %[thr]\n" \
    "s_sub_u32 %[c], %[thr], 1\n" \
    "s_waitcnt lgkmcnt(0)\n" \
    "v_readfirstlane_b32 s76, v24\n" \
    "v_readfirstlane_b32 %[rv], v23\n" \
    "s_add_u32 %[n1], %[c], 1\n" \
    "s_cmp_lg_u32 %[xi], %[n1]\n" \
    "s_cbranch_scc1 4211f\n" \
    "s_cmp_le_u32 %[rv], %[xi]\n" \
    "s_cbranch_scc1 431f\n" \
    "s_lshl_b32 %[so], %[xi], 14\n" \
    "buffer_load_dwordx4 v[64:67], v20, s[60:63], %[so] offen offset:0 sc0 sc1\n" \
    "buffer_load_dwordx4 v[68:71], v20, s[60:63], %[so] offen offset:1024 sc0 sc1\n" \
    "buffer_load_dwordx4 v[72:75], v20, s[60:63], %[so] offen offset:2048 sc0 sc1\n" \
    "buffer_load_dwordx4 v[76:79], v20, s[60:63], %[so] offen offset:3072 sc0 sc1\n" \
    "s_mov_b32 s72, %[vc]\n" \
    "s_add_u32 %[vc], %[vc], 1\n" \
    "s_add_u32 %[xi], %[xi], 1\n" \
    "4211:\n" \
    "s_add_u32 %[n1], %[c], 2\n" \
    "s_cmp_lg_u32 %[xi], %[n1]\n" \
    "s_cbranch_scc1 4212f\n" \
    "s_cmp_le_u32 %[rv], %[xi]\n" \
    "s_cbranch_scc1 431f\n" \
    "s_lshl_b32 %[so], %[xi], 14\n" \
    "buffer_load_dwordx4 v[80:83], v20, s[60:63], %[so] offen offset:0 sc0 sc1\n" \
    "buffer_load_dwordx4 v[84:87], v20, s[60:63], %[so] offen offset:1024 sc0 sc1\n" \
    "buffer_load_dwordx4 v[88:91], v20, s[60:63], %[so] offen offset:2048 sc0 sc1\n" \
    "buffer_load_dwordx4 v[92:95], v20, s[60:63], %[so] offen offset:3072 sc0 sc1\n" \
    "s_mov_b32 s73, %[vc]\n" \
    "s_add_u32 %[vc], %[vc], 1\n" \
    "s_add_u32 %[xi], %[xi], 1\n" \
    "4212:\n" \
    "s_add_u32 %[n1], %[c], 3\n" \
    "s_cmp_lg_u32 %[xi], %[n1]\n" \
    "s_cbranch_scc1 4213f\n" \
    "s_cmp_le_u32 %[rv], %[xi]\n" \
    "s_cbranch_scc1 431f\n" \
    "s_lshl_b32 %[so], %[xi], 14\n" \
    "buffer_load_dwordx4 v[32:35], v20, s[60:63], %[so] offen offset:0 sc0 sc1\n" \
    "buffer_load_dwordx4 v[36:39], v20, s[60:63], %[so] offen offset:1024 sc0 sc1\n" \
    "buffer_load_dwordx4 v[40:43], v20, s[60:63], %[so] offen offset:2048 sc0 sc1\n" \
    "buffer_load_dwordx4 v[44:47], v20, s[60:63], %[so] offen offset:3072 sc0 sc1\n" \
    "s_mov_b32 s70, %[vc]\n" \
    "s_add_u32 %[vc], %[vc], 1\n" \
    "s_add_u32 %[xi], %[xi], 1\n" \
    "4213:\n" \
    "431:\n" \
    "s_sub_u32 %[c], %[vc], s72\n" \
    "s_waitcnt vmcnt(8)\n" \
    "s_cmp_ge_u32 %[c], 3\n" \
    "s_cbranch_scc1 441f\n" \
    "s_waitcnt vmcnt(4)\n" \
    "s_cmp_ge_u32 %[c], 2\n" \
    "s_cbranch_scc1 441f\n" \
    "s_waitcnt vmcnt(0)\n" \
    "441:\n" \
    "s_cmp_gt_u32 s76, %[thr]\n" \
    "s_cbranch_scc1 411b\n" \
    "s_branch 401b\n" \
    "22:\n" \
    "s_add_u32 %[c], %[u], 2\n" \
    "s_cbranch_vccz 32f\n" \
    "v_add_u32 v7, %[sa], v18\n" \
    "s_waitcnt lgkmcnt(0)\n" \
    "v_add_u32 v23, %[sa], v19\n" \
    "42:\n" \
    "ds_read_b32 v21, v23 offset:512\n" \
    "s_waitcnt lgkmcnt(0)\n" \
    "v_cmp_ge_u32 vcc, %[thr], v21\n" \
    "s_cbranch_vccnz 42b\n" \
    "ds_read_b64_tr_b8 v[96:97], v8 offset:16384\n" \
    "ds_read_b64_tr_b8 v[98:99], v8 offset:17408\n" \
    "ds_read_b64_tr_b8 v[100:101], v8 offset:24576\n" \
    "ds_read_b64_tr_b8 v[102:103], v8 offset:25600\n" \
    "ds_read_b64_tr_b8 v[104:105], v9 offset:16384\n" \
    "ds_read_b64_tr_b8 v[106:107], v9 offset:17408\n" \
    "ds_read_b64_tr_b8 v[108:109], v9 offset:24576\n" \
    "ds_read_b64_tr_b8 v[110:111], v9 offset:25600\n" \
    "ds_read_b64_tr_b8 v[112:113], v10 offset:16384\n" \
    "ds_read_b64_tr_b8 v[114:115], v10 offset:17408\n" \
    "ds_read_b64_tr_b8 v[116:117], v10 offset:24576\n" \
    "ds_read_b64_tr_b8 v[118:119], v10 offset:25600\n" \
    "ds_read_b64_tr_b8 v[120:121], v11 offset:16384\n" \
    "ds_read_b64_tr_b8 v[122:123], v11 offset:17408\n" \
    "ds_read_b64_tr_b8 v[124:125], v11 offset:24576\n" \
    "ds_read_b64_tr_b8 v[126:127], v11 offset:25600\n" \
    "ds_read_b64_tr_b8 v[128:129], v12 offset:16384\n" \
    "ds_read_b64_tr_b8 v[130:131], v12 offset:17408\n" \
    "ds_read_b64_tr_b8 v[132:133], v12 offset:24576\n" \
    "ds_read_b64_tr_b8 v[134:135], v12 offset:25600\n" \
    "ds_read_b64_tr_b8 v[136:137], v13 offset:16384\n" \
    "ds_read_b64_tr_b8 v[138:139], v13 offset:17408\n" \
    "ds_read_b64_tr_b8 v[140:141], v13 offset:24576\n" \
    "ds_read_b64_tr_b8 v[142:143], v13 offset:25600\n" \
    "ds_read_b64_tr_b8 v[144:145], v14 offset:16384\n" \
    "ds_read_b64_tr_b8 v[146:147], v14 offset:17408\n" \
    "ds_read_b64_tr_b8 v[148:149], v14 offset:24576\n" \
    "ds_read_b64_tr_b8 v[150:151], v14 offset:25600\n" \
    "ds_read_b64_tr_b8 v[152:153], v15 offset:16384\n" \
    "ds_read_b64_tr_b8 v[154:155], v15 offset:17408\n" \
    "ds_read_b64_tr_b8 v[156:157], v15 offset:24576\n" \
    "ds_read_b64_tr_b8 v[158:159], v15 offset:25600\n" \
    "ds_write_b32 v7, v21 offset:512\n" \
    "32:\n" \
    "s_mov_b32 s62, %[sz]\n" \
    "s_add_u32 %[n1], %[c], 1\n" \
    "s_cmp_ge_u32 %[n1], 64\n" \
    "s_cbranch_scc1 12b\n" \
    "s_cmp_eq_u32 %[xi], %[n1]\n" \
    "s_cselect_b32 %[n1], %[n1], 0\n" \
    "62:\n" \
    "ds_read_b32 v21, v19 offset:16\n" \
    "s_waitcnt lgkmcnt(0)\n" \
    "v_readfirstlane_b32 %[rv], v21\n" \
    "s_cmp_le_u32 %[rv], %[n1]\n" \
    "s_cbranch_scc1 62b\n" \
    "s_add_u32 %[n1], %[c], 1\n" \
    "s_cmp_lg_u32 %[xi], %[n1]\n" \
    "s_cbranch_scc1 3321f\n" \
    "s_cmp_le_u32 %[rv], %[xi]\n" \
    "s_cbranch_scc1 82f\n" \
    "s_lshl_b32 %[so], %[xi], 14\n" \
    "buffer_load_dwordx4 v[80:83], v20, s[60:63], %[so] offen offset:0 sc0 sc1\n" \
    "buffer_load_dwordx4 v[84:87], v20, s[60:63], %[so] offen offset:1024 sc0 sc1\n" \
    "buffer_load_dwordx4 v[88:91], v20, s[60:63], %[so] offen offset:2048 sc0 sc1\n" \
    "buffer_load_dwordx4 v[92:95], v20, s[60:63], %[so] offen offset:3072 sc0 sc1\n" \
    "s_mov_b32 s73, %[vc]\n" \
    "s_add_u32 %[vc], %[vc], 1\n" \
    "s_add_u32 %[xi], %[xi], 1\n" \
    "3321:\n" \
    "s_add_u32 %[n1], %[c], 2\n" \
    "s_cmp_lg_u32 %[xi], %[n1]\n" \
    "s_cbranch_scc1 3322f\n" \
    "s_cmp_le_u32 %[rv], %[xi]\n" \
    "s_cbranch_scc1 82f\n" \
    "s_lshl_b32 %[so], %[xi], 14\n" \
    "buffer_load_dwordx4 v[32:35], v20, s[60:63], %[so] offen offset:0 sc0 sc1\n" \
    "buffer_load_dwordx4 v[36:39], v20, s[60:63], %[so] offen offset:1024 sc0 sc1\n" \
    "buffer_load_dwordx4 v[40:43], v20, s[60:63], %[so] offen offset:2048 sc0 sc1\n" \
    "buffer_load_dwordx4 v[44:47], v20, s[60:63], %[so] offen offset:3072 sc0 sc1\n" \
    "s_mov_b32 s70, %[vc]\n" \
    "s_add_u32 %[vc], %[vc], 1\n" \
    "s_add_u32 %[xi], %[xi], 1\n" \
    "3322:\n" \
    "s_add_u32 %[n1], %[c], 3\n" \
    "s_cmp_lg_u32 %[xi], %[n1]\n" \
    "s_cbranch_scc1 3323f\n" \
    "s_cmp_le_u32 %[rv], %[xi]\n" \
    "s_cbranch_scc1 82f\n" \
    "s_lshl_b32 %[so], %[xi], 14\n" \
    "buffer_load_dwordx4 v[48:51], v20, s[60:63], %[so] offen offset:0 sc0 sc1\n" \
    "buffer_load_dwordx4 v[52:55], v20, s[60:63], %[so] offen offset:1024 sc0 sc1\n" \
    "buffer_load_dwordx4 v[56:59], v20, s[60:63], %[so] offen offset:2048 sc0 sc1\n" \
    "buffer_load_dwordx4 v[60:63], v20, s[60:63], %[so] offen offset:3072 sc0 sc1\n" \
    "s_mov_b32 s71, %[vc]\n" \
    "s_add_u32 %[vc], %[vc], 1\n" \
    "s_add_u32 %[xi], %[xi], 1\n" \
    "3323:\n" \
    "82:\n" \
    "v_add_u32 v21, %[sb], v19\n" \
    "s_sub_u32 %[c], %[vc], s73\n" \
    "ds_read_b32 v22, v21 offset:0\n" \
    "ds_read_b32 v23, v19 offset:16\n" \
    "s_waitcnt vmcnt(24)\n" \
    "s_cmp_ge_u32 %[c], 7\n" \
    "s_cbranch_scc1 12b\n" \
    "s_waitcnt vmcnt(12)\n" \
    "s_cmp_ge_u32 %[c], 4\n" \
    "s_cbranch_scc1 12b\n" \
    "s_waitcnt vmcnt(8)\n" \
    "s_cmp_ge_u32 %[c], 3\n" \
    "s_cbranch_scc1 12b\n" \
    "s_waitcnt vmcnt(4)\n" \
    "s_cmp_ge_u32 %[c], 2\n" \
    "s_cbranch_scc1 12b\n" \
    "s_waitcnt vmcnt(0)\n" \
    "s_branch 12b\n" \
    "302:\n" \
    "s_add_u32 %[c], %[u], 2\n" \
    "s_mov_b32 s62, %[sz]\n" \
    "s_waitcnt lgkmcnt(6)\n" \
    "v_readfirstlane_b32 %[rv], v23\n" \
    "s_add_u32 %[n1], %[c], 1\n" \
    "s_cmp_lg_u32 %[xi], %[n1]\n" \
    "s_cbranch_scc1 3221f\n" \
    "s_cmp_le_u32 %[rv], %[xi]\n" \
    "s_cbranch_scc1 312b\n" \
    "s_lshl_b32 %[so], %[xi], 14\n" \
    "buffer_load_dwordx4 v[80:83], v20, s[60:63], %[so] offen offset:0 sc0 sc1\n" \
    "buffer_load_dwordx4 v[84:87], v20, s[60:63], %[so] offen offset:1024 sc0 sc1\n" \
    "buffer_load_dwordx4 v[88:91], v20, s[60:63], %[so] offen offset:2048 sc0 sc1\n" \
    "buffer_load_dwordx4 v[92:95], v20, s[60:63], %[so] offen offset:3072 sc0 sc1\n" \
    "s_mov_b32 s73, %[vc]\n" \
    "s_add_u32 %[vc], %[vc], 1\n" \
    "s_add_u32 %[xi], %[xi], 1\n" \
    "3221:\n" \
    "s_add_u32 %[n1], %[c], 2\n" \
    "s_cmp_lg_u32 %[xi], %[n1]\n" \
    "s_cbranch_scc1 3222f\n" \
    "s_cmp_le_u32 %[rv], %[xi]\n" \
    "s_cbranch_scc1 312b\n" \
    "s_lshl_b32 %[so], %[xi], 14\n" \
    "buffer_load_dwordx4 v[32:35], v20, s[60:63], %[so] offen offset:0 sc0 sc1\n" \
    "buffer_load_dwordx4 v[36:39], v20, s[60:63], %[so] offen offset:1024 sc0 sc1\n" \
    "buffer_load_dwordx4 v[40:43], v20, s[60:63], %[so] offen offset:2048 sc0 sc1\n" \
    "buffer_load_dwordx4 v[44:47], v20, s[60:63], %[so] offen offset:3072 sc0 sc1\n" \
    "s_mov_b32 s70, %[vc]\n" \
    "s_add_u32 %[vc], %[vc], 1\n" \
    "s_add_u32 %[xi], %[xi], 1\n" \
    "3222:\n" \
    "s_add_u32 %[n1], %[c], 3\n" \
    "s_cmp_lg_u32 %[xi], %[n1]\n" \
    "s_cbranch_scc1 3223f\n" \
    "s_cmp_le_u32 %[rv], %[xi]\n" \
    "s_cbranch_scc1 312b\n" \
    "s_lshl_b32 %[so], %[xi], 14\n" \
    "buffer_load_dwordx4 v[48:51], v20, s[60:63], %[so] offen offset:0 sc0 sc1\n" \
    "buffer_load_dwordx4 v[52:55], v20, s[60:63], %[so] offen offset:1024 sc0 sc1\n" \
    "buffer_load_dwordx4 v[56:59], v20, s[60:63], %[so] offen offset:2048 sc0 sc1\n" \
    "buffer_load_dwordx4 v[60:63], v20, s[60:63], %[so] offen offset:3072 sc0 sc1\n" \
    "s_mov_b32 s71, %[vc]\n" \
    "s_add_u32 %[vc], %[vc], 1\n" \
    "s_add_u32 %[xi], %[xi], 1\n" \
    "3223:\n" \
    "s_branch 312b\n" \
    "402:\n" \
    "v_max3_u32 v24, v80, v81, v82\n" \
    "v_max3_u32 v25, v83, v84, v85\n" \
    "v_max3_u32 v26, v86, v87, v88\n" \
    "v_max3_u32 v27, v89, v90, v91\n" \
    "v_max3_u32 v28, v92, v93, v94\n" \
    "v_max3_u32 v24, v24, v25, v95\n" \
    "v_max3_u32 v26, v26, v27, v28\n" \
    "v_max_u32 v24, v24, v26\n" \
    "v_cmp_eq_u32 vcc, -1, v24\n" \
    "s_cbranch_vccz 412b\n" \
    "452:\n" \
    "ds_read_b32 v24, v19 offset:24\n" \
    "s_mov_b32 s62, %[sz]\n" \
    "s_mov_b32 %[xi], %[thr]\n" \
    "s_sub_u32 %[c], %[thr], 1\n" \
    "s_waitcnt lgkmcnt(0)\n" \
    "v_readfirstlane_b32 s76, v24\n" \
    "v_readfirstlane_b32 %[rv], v23\n" \
    "s_add_u32 %[n1], %[c], 1\n" \
    "s_cmp_lg_u32 %[xi], %[n1]\n" \
    "s_cbranch_scc1 4221f\n" \
    "s_cmp_le_u32 %[rv], %[xi]\n" \
    "s_cbranch_scc1 432f\n" \
    "s_lshl_b32 %[so], %[xi], 14\n" \
    "buffer_load_dwordx4 v[80:83], v20, s[60:63], %[so] offen offset:0 sc0 sc1\n" \
    "buffer_load_dwordx4 v[84:87], v20, s[60:63], %[so] offen offset:1024 sc0 sc1\n" \
    "buffer_load_dwordx4 v[88:91], v20, s[60:63], %[so] offen offset:2048 sc0 sc1\n" \
    "buffer_load_dwordx4 v[92:95], v20, s[60:63], %[so] offen offset:3072 sc0 sc1\n" \
    "s_mov_b32 s73, %[vc]\n" \
    "s_add_u32 %[vc], %[vc], 1\n" \
    "s_add_u32 %[xi], %[xi], 1\n" \
    "4221:\n" \
    "s_add_u32 %[n1], %[c], 2\n" \
    "s_cmp_lg_u32 %[xi], %[n1]\n" \
    "s_cbranch_scc1 4222f\n" \
    "s_cmp_le_u32 %[rv], %[xi]\n" \
    "s_cbranch_scc1 432f\n" \
    "s_lshl_b32 %[so], %[xi], 14\n" \
    "buffer_load_dwordx4 v[32:35], v20, s[60:63], %[so] offen offset:0 sc0 sc1\n" \
    "buffer_load_dwordx4 v[36:39], v20, s[60:63], %[so] offen offset:1024 sc0 sc1\n" \
    "buffer_load_dwordx4 v[40:43], v20, s[60:63], %[so] offen offset:2048 sc0 sc1\n" \
    "buffer_load_dwordx4 v[44:47], v20, s[60:63], %[so] offen offset:3072 sc0 sc1\n" \
    "s_mov_b32 s70, %[vc]\n" \
    "s_add_u32 %[vc], %[vc], 1\n" \
    "s_add_u32 %[xi], %[xi], 1\n" \
    "4222:\n" \
    "s_add_u32 %[n1], %[c], 3\n" \
    "s_cmp_lg_u32 %[xi], %[n1]\n" \
    "s_cbranch_scc1 4223f\n" \
    "s_cmp_le_u32 %[rv], %[xi]\n" \
    "s_cbranch_scc1 432f\n" \
    "s_lshl_b32 %[so], %[xi], 14\n" \
    "buffer_load_dwordx4 v[48:51], v20, s[60:63], %[so] offen offset:0 sc0 sc1\n" \
    "buffer_load_dwordx4 v[52:55], v20, s[60:63], %[so] offen offset:1024 sc0 sc1\n" \
    "buffer_load_dwordx4 v[56:59], v20, s[60:63], %[so] offen offset:2048 sc0 sc1\n" \
    "buffer_load_dwordx4 v[60:63], v20, s[60:63], %[so] offen offset:3072 sc0 sc1\n" \
    "s_mov_b32 s71, %[vc]\n" \
    "s_add_u32 %[vc], %[vc], 1\n" \
    "s_add_u32 %[xi], %[xi], 1\n" \
    "4223:\n" \
    "432:\n" \
    "s_sub_u32 %[c], %[vc], s73\n" \
    "s_waitcnt vmcnt(8)\n" \
    "s_cmp_ge_u32 %[c], 3\n" \
    "s_cbranch_scc1 442f\n" \
    "s_waitcnt vmcnt(4)\n" \
    "s_cmp_ge_u32 %[c], 2\n" \
    "s_cbranch_scc1 442f\n" \
    "s_waitcnt vmcnt(0)\n" \
    "442:\n" \
    "s_cmp_gt_u32 s76, %[thr]\n" \
    "s_cbranch_scc1 412b\n" \
    "s_branch 402b\n" \
    "23:\n" \
    "s_add_u32 %[c], %[u], 3\n" \
    "s_cbranch_vccz 33f\n" \
    "v_add_u32 v7, %[sa], v18\n" \
    "s_waitcnt lgkmcnt(0)\n" \
    "v_add_u32 v23, %[sa], v19\n" \
    "43:\n" \
    "ds_read_b32 v21, v23 offset:0\n" \
    "s_waitcnt lgkmcnt(0)\n" \
    "v_cmp_ge_u32 vcc, %[thr], v21\n" \
    "s_cbranch_vccnz 43b\n" \
    "ds_read_b64_tr_b8 v[96:97], v8 offset:0\n" \
    "ds_read_b64_tr_b8 v[98:99], v8 offset:1024\n" \
    "ds_read_b64_tr_b8 v[100:101], v8 offset:8192\n" \
    "ds_read_b64_tr_b8 v[102:103], v8 offset:9216\n" \
    "ds_read_b64_tr_b8 v[104:105], v9 offset:0\n" \
    "ds_read_b64_tr_b8 v[106:107], v9 offset:1024\n" \
    "ds_read_b64_tr_b8 v[108:109], v9 offset:8192\n" \
    "ds_read_b64_tr_b8 v[110:111], v9 offset:9216\n" \
    "ds_read_b64_tr_b8 v[112:113], v10 offset:0\n" \
    "ds_read_b64_tr_b8 v[114:115], v10 offset:1024\n" \
    "ds_read_b64_tr_b8 v[116:117], v10 offset:8192\n" \
    "ds_read_b64_tr_b8 v[118:119], v10 offset:9216\n" \
    "ds_read_b64_tr_b8 v[120:121], v11 offset:0\n" \
    "ds_read_b64_tr_b8 v[122:123], v11 offset:1024\n" \
    "ds_read_b64_tr_b8 v[124:125], v11 offset:8192\n" \
    "ds_read_b64_tr_b8 v[126:127], v11 offset:9216\n" \
    "ds_read_b64_tr_b8 v[128:129], v12 offset:0\n" \
    "ds_read_b64_tr_b8 v[130:131], v12 offset:1024\n" \
    "ds_read_b64_tr_b8 v[132:133], v12 offset:8192\n" \
    "ds_read_b64_tr_b8 v[134:135], v12 offset:9216\n" \
    "ds_read_b64_tr_b8 v[136:137], v13 offset:0\n" \
    "ds_read_b64_tr_b8 v[138:139], v13 offset:1024\n" \
    "ds_read_b64_tr_b8 v[140:141], v13 offset:8192\n" \
    "ds_read_b64_tr_b8 v[142:143], v13 offset:9216\n" \
    "ds_read_b64_tr_b8 v[144:145], v14 offset:0\n" \
    "ds_read_b64_tr_b8 v[146:147], v14 offset:1024\n" \
    "ds_read_b64_tr_b8 v[148:149], v14 offset:8192\n" \
    "ds_read_b64_tr_b8 v[150:151], v14 offset:9216\n" \
    "ds_read_b64_tr_b8 v[152:153], v15 offset:0\n" \
    "ds_read_b64_tr_b8 v[154:155], v15 offset:1024\n" \
    "ds_read_b64_tr_b8 v[156:157], v15 offset:8192\n" \
    "ds_read_b64_tr_b8 v[158:159], v15 offset:9216\n" \
    "ds_write_b32 v7, v21 offset:0\n" \
    "33:\n" \
    "s_mov_b32 s62, %[sz]\n" \
    "s_add_u32 %[n1], %[c], 1\n" \
    "s_cmp_ge_u32 %[n1], 64\n" \
    "s_cbranch_scc1 13b\n" \
    "s_cmp_eq_u32 %[xi], %[n1]\n" \
    "s_cselect_b32 %[n1], %[n1], 0\n" \
    "63:\n" \
    "ds_read_b32 v21, v19 offset:16\n" \
    "s_waitcnt lgkmcnt(0)\n" \
    "v_readfirstlane_b32 %[rv], v21\n" \
    "s_cmp_le_u32 %[rv], %[n1]\n" \
    "s_cbranch_scc1 63b\n" \
    "s_add_u32 %[n1], %[c], 1\n" \
    "s_cmp_lg_u32 %[xi], %[n1]\n" \
    "s_cbranch_scc1 3331f\n" \
    "s_cmp_le_u32 %[rv], %[xi]\n" \
    "s_cbranch_scc1 83f\n" \
    "s_lshl_b32 %[so], %[xi], 14\n" \
    "buffer_load_dwordx4 v[32:35], v20, s[60:63], %[so] offen offset:0 sc0 sc1\n" \
    "buffer_load_dwordx4 v[36:39], v20, s[60:63], %[so] offen offset:1024 sc0 sc1\n" \
    "buffer_load_dwordx4 v[40:43], v20, s[60:63], %[so] offen offset:2048 sc0 sc1\n" \
    "buffer_load_dwordx4 v[44:47], v20, s[60:63], %[so] offen offset:3072 sc0 sc1\n" \
    "s_mov_b32 s70, %[vc]\n" \
    "s_add_u32 %[vc], %[vc], 1\n" \
    "s_add_u32 %[xi], %[xi], 1\n" \
    "3331:\n" \
    "s_add_u32 %[n1], %[c], 2\n" \
    "s_cmp_lg_u32 %[xi], %[n1]\n" \
    "s_cbranch_scc1 3332f\n" \
    "s_cmp_le_u32 %[rv], %[xi]\n" \
    "s_cbranch_scc1 83f\n" \
    "s_lshl_b32 %[so], %[xi], 14\n" \
    "buffer_load_dwordx4 v[48:51], v20, s[60:63], %[so] offen offset:0 sc0 sc1\n" \
    "buffer_load_dwordx4 v[52:55], v20, s[60:63], %[so] offen offset:1024 sc0 sc1\n" \
    "buffer_load_dwordx4 v[56:59], v20, s[60:63], %[so] offen offset:2048 sc0 sc1\n" \
    "buffer_load_dwordx4 v[60:63], v20, s[60:63], %[so] offen offset:3072 sc0 sc1\n" \
    "s_mov_b32 s71, %[vc]\n" \
    "s_add_u32 %[vc], %[vc], 1\n" \
    "s_add_u32 %[xi], %[xi], 1\n" \
    "3332:\n" \
    "s_add_u32 %[n1], %[c], 3\n" \
    "s_cmp_lg_u32 %[xi], %[n1]\n" \
    "s_cbranch_scc1 3333f\n" \
    "s_cmp_le_u32 %[rv], %[xi]\n" \
    "s_cbranch_scc1 83f\n" \
    "s_lshl_b32 %[so], %[xi], 14\n" \
    "buffer_load_dwordx4 v[64:67], v20, s[60:63], %[so] offen offset:0 sc0 sc1\n" \
    "buffer_load_dwordx4 v[68:71], v20, s[60:63], %[so] offen offset:1024 sc0 sc1\n" \
    "buffer_load_dwordx4 v[72:75], v20, s[60:63], %[so] offen offset:2048 sc0 sc1\n" \
    "buffer_load_dwordx4 v[76:79], v20, s[60:63], %[so] offen offset:3072 sc0 sc1\n" \
    "s_mov_b32 s72, %[vc]\n" \
    "s_add_u32 %[vc], %[vc], 1\n" \
    "s_add_u32 %[xi], %[xi], 1\n" \
    "3333:\n" \
    "83:\n" \
    "v_add_u32 v21, %[sa], v19\n" \
    "s_sub_u32 %[c], %[vc], s70\n" \
    "ds_read_b32 v22, v21 offset:512\n" \
    "ds_read_b32 v23, v19 offset:16\n" \
    "s_waitcnt vmcnt(24)\n" \
    "s_cmp_ge_u32 %[c], 7\n" \
    "s_cbranch_scc1 13b\n" \
    "s_waitcnt vmcnt(12)\n" \
    "s_cmp_ge_u32 %[c], 4\n" \
    "s_cbranch_scc1 13b\n" \
    "s_waitcnt vmcnt(8)\n" \
    "s_cmp_ge_u32 %[c], 3\n" \
    "s_cbranch_scc1 13b\n" \
    "s_waitcnt vmcnt(4)\n" \
    "s_cmp_ge_u32 %[c], 2\n" \
    "s_cbranch_scc1 13b\n" \
    "s_waitcnt vmcnt(0)\n" \
    "s_branch 13b\n" \
    "303:\n" \
    "s_add_u32 %[c], %[u], 3\n" \
    "s_mov_b32 s62, %[sz]\n" \
    "s_waitcnt lgkmcnt(6)\n" \
    "v_readfirstlane_b32 %[rv], v23\n" \
    "s_add_u32 %[n1], %[c], 1\n" \
    "s_cmp_lg_u32 %[xi], %[n1]\n" \
    "s_cbranch_scc1 3231f\n" \
    "s_cmp_le_u32 %[rv], %[xi]\n" \
    "s_cbranch_scc1 313b\n" \
    "s_lshl_b32 %[so], %[xi], 14\n" \
    "buffer_load_dwordx4 v[32:35], v20, s[60:63], %[so] offen offset:0 sc0 sc1\n" \
    "buffer_load_dwordx4 v[36:39], v20, s[60:63], %[so] offen offset:1024 sc0 sc1\n" \
    "buffer_load_dwordx4 v[40:43], v20, s[60:63], %[so] offen offset:2048 sc0 sc1\n" \
    "buffer_load_dwordx4 v[44:47], v20, s[60:63], %[so] offen offset:3072 sc0 sc1\n" \
    "s_mov_b32 s70, %[vc]\n" \
    "s_add_u32 %[vc], %[vc], 1\n" \
    "s_add_u32 %[xi], %[xi], 1\n" \
    "3231:\n" \
    "s_add_u32 %[n1], %[c], 2\n" \
    "s_cmp_lg_u32 %[xi], %[n1]\n" \
    "s_cbranch_scc1 3232f\n" \
    "s_cmp_le_u32 %[rv], %[xi]\n" \
    "s_cbranch_scc1 313b\n" \
    "s_lshl_b32 %[so], %[xi], 14\n" \
    "buffer_load_dwordx4 v[48:51], v20, s[60:63], %[so] offen offset:0 sc0 sc1\n" \
    "buffer_load_dwordx4 v[52:55], v20, s[60:63], %[so] offen offset:1024 sc0 sc1\n" \
    "buffer_load_dwordx4 v[56:59], v20, s[60:63], %[so] offen offset:2048 sc0 sc1\n" \
    "buffer_load_dwordx4 v[60:63], v20, s[60:63], %[so] offen offset:3072 sc0 sc1\n" \
    "s_mov_b32 s71, %[vc]\n" \
    "s_add_u32 %[vc], %[vc], 1\n" \
    "s_add_u32 %[xi], %[xi], 1\n" \
    "3232:\n" \
    "s_add_u32 %[n1], %[c], 3\n" \
    "s_cmp_lg_u32 %[xi], %[n1]\n" \
    "s_cbranch_scc1 3233f\n" \
    "s_cmp_le_u32 %[rv], %[xi]\n" \
    "s_cbranch_scc1 313b\n" \
    "s_lshl_b32 %[so], %[xi], 14\n" \
    "buffer_load_dwordx4 v[64:67], v20, s[60:63], %[so] offen offset:0 sc0 sc1\n" \
    "buffer_load_dwordx4 v[68:71], v20, s[60:63], %[so] offen offset:1024 sc0 sc1\n" \
    "buffer_load_dwordx4 v[72:75], v20, s[60:63], %[so] offen offset:2048 sc0 sc1\n" \
    "buffer_load_dwordx4 v[76:79], v20, s[60:63], %[so] offen offset:3072 sc0 sc1\n" \
    "s_mov_b32 s72, %[vc]\n" \
    "s_add_u32 %[vc], %[vc], 1\n" \
    "s_add_u32 %[xi], %[xi], 1\n" \
    "3233:\n" \
    "s_branch 313b\n" \
    "403:\n" \
    "v_max3_u32 v24, v32, v33, v34\n" \
    "v_max3_u32 v25, v35, v36, v37\n" \
    "v_max3_u32 v26, v38, v39, v40\n" \
    "v_max3_u32 v27, v41, v42, v43\n" \
    "v_max3_u32 v28, v44, v45, v46\n" \
    "v_max3_u32 v24, v24, v25, v47\n" \
    "v_max3_u32 v26, v26, v27, v28\n" \
    "v_max_u32 v24, v24, v26\n" \
    "v_cmp_eq_u32 vcc, -1, v24\n" \
    "s_cbranch_vccz 413b\n" \
    "453:\n" \
    "ds_read_b32 v24, v19 offset:24\n" \
    "s_mov_b32 s62, %[sz]\n" \
    "s_mov_b32 %[xi], %[thr]\n" \
    "s_sub_u32 %[c], %[thr], 1\n" \
    "s_waitcnt lgkmcnt(0)\n" \
    "v_readfirstlane_b32 s76, v24\n" \
    "v_readfirstlane_b32 %[rv], v23\n" \
    "s_add_u32 %[n1], %[c], 1\n" \
    "s_cmp_lg_u32 %[xi], %[n1]\n" \
    "s_cbranch_scc1 4231f\n" \
    "s_cmp_le_u32 %[rv], %[xi]\n" \
    "s_cbranch_scc1 433f\n" \
    "s_lshl_b32 %[so], %[xi], 14\n" \
    "buffer_load_dwordx4 v[32:35], v20, s[60:63], %[so] offen offset:0 sc0 sc1\n" \
    "buffer_load_dwordx4 v[36:39], v20, s[60:63], %[so] offen offset:1024 sc0 sc1\n" \
    "buffer_load_dwordx4 v[40:43], v20, s[60:63], %[so] offen offset:2048 sc0 sc1\n" \
    "buffer_load_dwordx4 v[44:47], v20, s[60:63], %[so] offen offset:3072 sc0 sc1\n" \
    "s_mov_b32 s70, %[vc]\n" \
    "s_add_u32 %[vc], %[vc], 1\n" \
    "s_add_u32 %[xi], %[xi], 1\n" \
    "4231:\n" \
    "s_add_u32 %[n1], %[c], 2\n" \
    "s_cmp_lg_u32 %[xi], %[n1]\n" \
    "s_cbranch_scc1 4232f\n" \
    "s_cmp_le_u32 %[rv], %[xi]\n" \
    "s_cbranch_scc1 433f\n" \
    "s_lshl_b32 %[so], %[xi], 14\n" \
    "buffer_load_dwordx4 v[48:51], v20, s[60:63], %[so] offen offset:0 sc0 sc1\n" \
    "buffer_load_dwordx4 v[52:55], v20, s[60:63], %[so] offen offset:1024 sc0 sc1\n" \
    "buffer_load_dwordx4 v[56:59], v20, s[60:63], %[so] offen offset:2048 sc0 sc1\n" \
    "buffer_load_dwordx4 v[60:63], v20, s[60:63], %[so] offen offset:3072 sc0 sc1\n" \
    "s_mov_b32 s71, %[vc]\n" \
    "s_add_u32 %[vc], %[vc], 1\n" \
    "s_add_u32 %[xi], %[xi], 1\n" \
    "4232:\n" \
    "s_add_u32 %[n1], %[c], 3\n" \
    "s_cmp_lg_u32 %[xi], %[n1]\n" \
    "s_cbranch_scc1 4233f\n" \
    "s_cmp_le_u32 %[rv], %[xi]\n" \
    "s_cbranch_scc1 433f\n" \
    "s_lshl_b32 %[so], %[xi], 14\n" \
    "buffer_load_dwordx4 v[64:67], v20, s[60:63], %[so] offen offset:0 sc0 sc1\n" \
    "buffer_load_dwordx4 v[68:71], v20, s[60:63], %[so] offen offset:1024 sc0 sc1\n" \
    "buffer_load_dwordx4 v[72:75], v20, s[60:63], %[so] offen offset:2048 sc0 sc1\n" \
    "buffer_load_dwordx4 v[76:79], v20, s[60:63], %[so] offen offset:3072 sc0 sc1\n" \
    "s_mov_b32 s72, %[vc]\n" \
    "s_add_u32 %[vc], %[vc], 1\n" \
    "s_add_u32 %[xi], %[xi], 1\n" \
    "4233:\n" \
    "433:\n" \
    "s_sub_u32 %[c], %[vc], s70\n" \
    "s_waitcnt vmcnt(8)\n" \
    "s_cmp_ge_u32 %[c], 3\n" \
    "s_cbranch_scc1 443f\n" \
    "s_waitcnt vmcnt(4)\n" \
    "s_cmp_ge_u32 %[c], 2\n" \
    "s_cbranch_scc1 443f\n" \
    "s_waitcnt vmcnt(0)\n" \
    "443:\n" \
    "s_cmp_gt_u32 s76, %[thr]\n" \
    "s_cbranch_scc1 413b\n" \
    "s_branch 403b\n" \
    "9:\n" \
    "99:\n" \
    AG_STAMP_K \
    "ds_read_b32 v26, v19 offset:20\n" \
    "s_waitcnt lgkmcnt(0)\n" \
    "v_readfirstlane_b32 %[rv], v26\n" \
    "s_cmp_eq_u32 %[rv], 0\n" \
    "s_cselect_b32 s68, %[scale], 0x7fc00000\n" \
    "s_mov_b32 s69, s68\n" \
    "v_pk_mul_f32 v[160:161], v[160:161], s[68:69]\n" \
    "v_pk_mul_f32 v[162:163], v[162:163], s[68:69]\n" \
    "ds_write_b128 v0, v[160:163] offset:0\n" \
    "v_pk_mul_f32 v[164:165], v[164:165], s[68:69]\n" \
    "v_pk_mul_f32 v[166:167], v[166:167], s[68:69]\n" \
    "ds_write_b128 v1, v[164:167] offset:0\n" \
    "v_pk_mul_f32 v[168:169], v[168:169], s[68:69]\n" \
    "v_pk_mul_f32 v[170:171], v[170:171], s[68:69]\n" \
    "ds_write_b128 v0, v[168:171] offset:128\n" \
    "v_pk_mul_f32 v[172:173], v[172:173], s[68:69]\n" \
    "v_pk_mul_f32 v[174:175], v[174:175], s[68:69]\n" \
    "ds_write_b128 v1, v[172:175] offset:128\n" \
    "v_pk_mul_f32 v[176:177], v[176:177], s[68:69]\n" \
    "v_pk_mul_f32 v[178:179], v[178:179], s[68:69]\n" \
    "ds_write_b128 v0, v[176:179] offset:256\n" \
    "v_pk_mul_f32 v[180:181], v[180:181], s[68:69]\n" \
    "v_pk_mul_f32 v[182:183], v[182:183], s[68:69]\n" \
    "ds_write_b128 v1, v[180:183] offset:256\n" \
    "v_pk_mul_f32 v[184:185], v[184:185], s[68:69]\n" \
    "v_pk_mul_f32 v[186:187], v[186:187], s[68:69]\n" \
    "ds_write_b128 v0, v[184:187] offset:384\n" \
    "v_pk_mul_f32 v[188:189], v[188:189], s[68:69]\n" \
    "v_pk_mul_f32 v[190:191], v[190:191], s[68:69]\n" \
    "ds_write_b128 v1, v[188:191] offset:384\n" \
    "ds_read_b128 v[160:163], v2 offset:0\n" \
    "ds_read_b128 v[164:167], v3 offset:0\n" \
    "ds_read_b128 v[168:171], v4 offset:0\n" \
    "ds_read_b128 v[172:175], v5 offset:0\n" \
    "ds_read_b128 v[176:179], v2 offset:4096\n" \
    "ds_read_b128 v[180:183], v3 offset:4096\n" \
    "ds_read_b128 v[184:187], v4 offset:4096\n" \
    "ds_read_b128 v[188:191], v5 offset:4096\n" \
    "s_mov_b32 %[so], %[rowr]\n" \
    "s_add_u32 %[vc], %[vc], 2\n" \
    "s_waitcnt lgkmcnt(7)\n" \
    "buffer_store_dwordx4 v[160:163], v6, s[64:67], %[so] offen sc1\n" \
    "s_add_u32 %[so], %[so], 16384\n" \
    "v_pk_mul_f32 v[192:193], v[192:193], s[68:69]\n" \
    "v_pk_mul_f32 v[194:195], v[194:195], s[68:69]\n" \
    "ds_write_b128 v0, v[192:195] offset:0\n" \
    "s_waitcnt lgkmcnt(7)\n" \
    "buffer_store_dwordx4 v[164:167], v6, s[64:67], %[so] offen sc1\n" \
    "s_add_u32 %[so], %[so], 16384\n" \
    "v_pk_mul_f32 v[196:197], v[196:197], s[68:69]\n" \
    "v_pk_mul_f32 v[198:199], v[198:199], s[68:69]\n" \
    "ds_write_b128 v1, v[196:199] offset:0\n" \
    "s_waitcnt lgkmcnt(7)\n" \
    "buffer_store_dwordx4 v[168:171], v6, s[64:67], %[so] offen sc1\n" \
    "s_add_u32 %[so], %[so], 16384\n" \
    "v_pk_mul_f32 v[200:201], v[200:201], s[68:69]\n" \
    "v_pk_mul_f32 v[202:203], v[202:203], s[68:69]\n" \
    "ds_write_b128 v0, v[200:203] offset:128\n" \
    "s_waitcnt lgkmcnt(7)\n" \
    "buffer_store_dwordx4 v[172:175], v6, s[64:67], %[so] offen sc1\n" \
    "s_add_u32 %[so], %[so], 16384\n" \
    "v_pk_mul_f32 v[204:205], v[204:205], s[68:69]\n" \
    "v_pk_mul_f32 v[206:207], v[206:207], s[68:69]\n" \
    "ds_write_b128 v1, v[204:207] offset:128\n" \
    "s_waitcnt lgkmcnt(7)\n" \
    "buffer_store_dwordx4 v[176:179], v6, s[64:67], %[so] offen sc1\n" \
    "s_add_u32 %[so], %[so], 16384\n" \
    "v_pk_mul_f32 v[208:209], v[208:209], s[68:69]\n" \
    "v_pk_mul_f32 v[210:211], v[210:211], s[68:69]\n" \
    "ds_write_b128 v0, v[208:211] offset:256\n" \
    "s_waitcnt lgkmcnt(7)\n" \
    "buffer_store_dwordx4 v[180:183], v6, s[64:67], %[so] offen sc1\n" \
    "s_add_u32 %[so], %[so], 16384\n" \
    "v_pk_mul_f32 v[212:213], v[212:213], s[68:69]\n" \
    "v_pk_mul_f32 v[214:215], v[214:215], s[68:69]\n" \
    "ds_write_b128 v1, v[212:215] offset:256\n" \
    "s_waitcnt lgkmcnt(7)\n" \
    "buffer_store_dwordx4 v[184:187], v6, s[64:67], %[so] offen sc1\n" \
    "s_add_u32 %[so], %[so], 16384\n" \
    "v_pk_mul_f32 v[216:217], v[216:217], s[68:69]\n" \
    "v_pk_mul_f32 v[218:219], v[218:219], s[68:69]\n" \
    "ds_write_b128 v0, v[216:219] offset:384\n" \
    "s_waitcnt lgkmcnt(7)\n" \
    "buffer_store_dwordx4 v[188:191], v6, s[64:67], %[so] offen sc1\n" \
    "v_pk_mul_f32 v[220:221], v[220:221], s[68:69]\n" \
    "v_pk_mul_f32 v[222:223], v[222:223], s[68:69]\n" \
    "ds_write_b128 v1, v[220:223] offset:384\n" \
    "ds_read_b128 v[192:195], v2 offset:0\n" \
    "ds_read_b128 v[196:199], v3 offset:0\n" \
    "ds_read_b128 v[200:203], v4 offset:0\n" \
    "ds_read_b128 v[204:207], v5 offset:0\n" \
    "ds_read_b128 v[208:211], v2 offset:4096\n" \
    "ds_read_b128 v[212:215], v3 offset:4096\n" \
    "ds_read_b128 v[216:219], v4 offset:4096\n" \
    "ds_read_b128 v[220:223], v5 offset:4096\n" \
    "s_add_u32 %[c], %[rowr], 131072\n" \
    "s_cmp_eq_u32 %[three], 0\n" \
    "s_cbranch_scc1 96f\n" \
    "s_mov_b32 %[so], %[c]\n" \
    "s_add_u32 %[vc], %[vc], 2\n" \
    "s_waitcnt lgkmcnt(7)\n" \
    "buffer_store_dwordx4 v[192:195], v6, s[64:67], %[so] offen sc1\n" \
    "s_add_u32 %[so], %[so], 16384\n" \
    "v_pk_mul_f32 v[224:225], v[224:225], s[68:69]\n" \
    "v_pk_mul_f32 v[226:227], v[226:227], s[68:69]\n" \
    "ds_write_b128 v0, v[224:227] offset:0\n" \
    "s_waitcnt lgkmcnt(7)\n" \
    "buffer_store_dwordx4 v[196:199], v6, s[64:67], %[so] offen sc1\n" \
    "s_add_u32 %[so], %[so], 16384\n" \
    "v_pk_mul_f32 v[228:229], v[228:229], s[68:69]\n" \
    "v_pk_mul_f32 v[230:231], v[230:231], s[68:69]\n" \
    "ds_write_b128 v1, v[228:231] offset:0\n" \
    "s_waitcnt lgkmcnt(7)\n" \
    "buffer_store_dwordx4 v[200:203], v6, s[64:67], %[so] offen sc1\n" \
    "s_add_u32 %[so], %[so], 16384\n" \
    "v_pk_mul_f32 v[232:233], v[232:233], s[68:69]\n" \
    "v_pk_mul_f32 v[234:235], v[234:235], s[68:69]\n" \
    "ds_write_b128 v0, v[232:235] offset:128\n" \
    "s_waitcnt lgkmcnt(7)\n" \
    "buffer_store_dwordx4 v[204:207], v6, s[64:67], %[so] offen sc1\n" \
    "s_add_u32 %[so], %[so], 16384\n" \
    "v_pk_mul_f32 v[236:237], v[236:237], s[68:69]\n" \
    "v_pk_mul_f32 v[238:239], v[238:239], s[68:69]\n" \
    "ds_write_b128 v1, v[236:239] offset:128\n" \
    "s_waitcnt lgkmcnt(7)\n" \
    "buffer_store_dwordx4 v[208:211], v6, s[64:67], %[so] offen sc1\n" \
    "s_add_u32 %[so], %[so], 16384\n" \
    "v_pk_mul_f32 v[240:241], v[240:241], s[68:69]\n" \
    "v_pk_mul_f32 v[242:243], v[242:243], s[68:69]\n" \
    "ds_write_b128 v0, v[240:243] offset:256\n" \
    "s_waitcnt lgkmcnt(7)\n" \
    "buffer_store_dwordx4 v[212:215], v6, s[64:67], %[so] offen sc1\n" \
    "s_add_u32 %[so], %[so], 16384\n" \
    "v_pk_mul_f32 v[244:245], v[244:245], s[68:69]\n" \
    "v_pk_mul_f32 v[246:247], v[246:247], s[68:69]\n" \
    "ds_write_b128 v1, v[244:247] offset:256\n" \
    "s_waitcnt lgkmcnt(7)\n" \
    "buffer_store_dwordx4 v[216:219], v6, s[64:67], %[so] offen sc1\n" \
    "s_add_u32 %[so], %[so], 16384\n" \
    "v_pk_mul_f32 v[248:249], v[248:249], s[68:69]\n" \
    "v_pk_mul_f32 v[250:251], v[250:251], s[68:69]\n" \
    "ds_write_b128 v0, v[248:251] offset:384\n" \
    "s_waitcnt lgkmcnt(7)\n" \
    "buffer_store_dwordx4 v[220:223], v6, s[64:67], %[so] offen sc1\n" \
    "v_pk_mul_f32 v[252:253], v[252:253], s[68:69]\n" \
    "v_pk_mul_f32 v[254:255], v[254:255], s[68:69]\n" \
    "ds_write_b128 v1, v[252:255] offset:384\n" \
    "ds_read_b128 v[224:227], v2 offset:0\n" \
    "ds_read_b128 v[228:231], v3 offset:0\n" \
    "ds_read_b128 v[232:235], v4 offset:0\n" \
    "ds_read_b128 v[236:239], v5 offset:0\n" \
    "ds_read_b128 v[240:243], v2 offset:4096\n" \
    "ds_read_b128 v[244:247], v3 offset:4096\n" \
    "ds_read_b128 v[248:251], v4 offset:4096\n" \
    "ds_read_b128 v[252:255], v5 offset:4096\n" \
    "v_mov_b64 v[160:161], 0\n" \
    "v_mov_b64 v[162:163], 0\n" \
    "v_mov_b64 v[164:165], 0\n" \
    "v_mov_b64 v[166:167], 0\n" \
    "v_mov_b64 v[168:169], 0\n" \
    "v_mov_b64 v[170:171], 0\n" \
    "v_mov_b64 v[172:173], 0\n" \
    "v_mov_b64 v[174:175], 0\n" \
    "v_mov_b64 v[176:177], 0\n" \
    "v_mov_b64 v[178:179], 0\n" \
    "v_mov_b64 v[180:181], 0\n" \
    "v_mov_b64 v[182:183], 0\n" \
    "v_mov_b64 v[184:185], 0\n" \
    "v_mov_b64 v[186:187], 0\n" \
    "v_mov_b64 v[188:189], 0\n" \
    "v_mov_b64 v[190:191], 0\n" \
    "s_mov_b32 %[so], %[rowo]\n" \
    "s_add_u32 %[vc], %[vc], 2\n" \
    "s_waitcnt lgkmcnt(7)\n" \
    "buffer_store_dwordx4 v[224:227], v6, s[64:67], %[so] offen sc1\n" \
    "s_add_u32 %[so], %[so], 16384\n" \
    "s_waitcnt lgkmcnt(6)\n" \
    "buffer_store_dwordx4 v[228:231], v6, s[64:67], %[so] offen sc1\n" \
    "s_add_u32 %[so], %[so], 16384\n" \
    "s_waitcnt lgkmcnt(5)\n" \
    "buffer_store_dwordx4 v[232:235], v6, s[64:67], %[so] offen sc1\n" \
    "s_add_u32 %[so], %[so], 16384\n" \
    "s_waitcnt lgkmcnt(4)\n" \
    "buffer_store_dwordx4 v[236:239], v6, s[64:67], %[so] offen sc1\n" \
    "s_add_u32 %[so], %[so], 16384\n" \
    "s_waitcnt lgkmcnt(3)\n" \
    "buffer_store_dwordx4 v[240:243], v6, s[64:67], %[so] offen sc1\n" \
    "s_add_u32 %[so], %[so], 16384\n" \
    "s_waitcnt lgkmcnt(2)\n" \
    "buffer_store_dwordx4 v[244:247], v6, s[64:67], %[so] offen sc1\n" \
    "s_add_u32 %[so], %[so], 16384\n" \
    "s_waitcnt lgkmcnt(1)\n" \
    "buffer_store_dwordx4 v[248:251], v6, s[64:67], %[so] offen sc1\n" \
    "s_add_u32 %[so], %[so], 16384\n" \
    "s_waitcnt lgkmcnt(0)\n" \
    "buffer_store_dwordx4 v[252:255], v6, s[64:67], %[so] offen sc1\n" \
    "s_branch 97f\n" \
    "96:\n" \
    "s_mov_b32 %[so], %[c]\n" \
    "s_add_u32 %[vc], %[vc], 2\n" \
    "s_waitcnt lgkmcnt(7)\n" \
    "buffer_store_dwordx4 v[192:195], v6, s[64:67], %[so] offen sc1\n" \
    "s_add_u32 %[so], %[so], 16384\n" \
    "s_waitcnt lgkmcnt(6)\n" \
    "buffer_store_dwordx4 v[196:199], v6, s[64:67], %[so] offen sc1\n" \
    "s_add_u32 %[so], %[so], 16384\n" \
    "s_waitcnt lgkmcnt(5)\n" \
    "buffer_store_dwordx4 v[200:203], v6, s[64:67], %[so] offen sc1\n" \
    "s_add_u32 %[so], %[so], 16384\n" \
    "s_waitcnt lgkmcnt(4)\n" \
    "buffer_store_dwordx4 v[204:207], v6, s[64:67], %[so] offen sc1\n" \
    "s_add_u32 %[so], %[so], 16384\n" \
    "s_waitcnt lgkmcnt(3)\n" \
    "buffer_store_dwordx4 v[208:211], v6, s[64:67], %[so] offen sc1\n" \
    "s_add_u32 %[so], %[so], 16384\n" \
    "s_waitcnt lgkmcnt(2)\n" \
    "buffer_store_dwordx4 v[212:215], v6, s[64:67], %[so] offen sc1\n" \
    "s_add_u32 %[so], %[so], 16384\n" \
    "s_waitcnt lgkmcnt(1)\n" \
    "buffer_store_dwordx4 v[216:219], v6, s[64:67], %[so] offen sc1\n" \
    "s_add_u32 %[so], %[so], 16384\n" \
    "s_waitcnt lgkmcnt(0)\n" \
    "buffer_store_dwordx4 v[220:223], v6, s[64:67], %[so] offen sc1\n" \
    "v_mov_b64 v[160:161], 0\n" \
    "v_mov_b64 v[162:163], 0\n" \
    "v_mov_b64 v[164:165], 0\n" \
    "v_mov_b64 v[166:167], 0\n" \
    "v_mov_b64 v[168:169], 0\n" \
    "v_mov_b64 v[170:171], 0\n" \
    "v_mov_b64 v[172:173], 0\n" \
    "v_mov_b64 v[174:175], 0\n" \
    "v_mov_b64 v[176:177], 0\n" \
    "v_mov_b64 v[178:179], 0\n" \
    "v_mov_b64 v[180:181], 0\n" \
    "v_mov_b64 v[182:183], 0\n" \
    "v_mov_b64 v[184:185], 0\n" \
    "v_mov_b64 v[186:187], 0\n" \
    "v_mov_b64 v[188:189], 0\n" \
    "v_mov_b64 v[190:191], 0\n" \
    "97:\n" \
    AG_STAMP_E \
    "s_cmp_ge_u32 %[u], 64\n" \
    "s_cbranch_scc1 95f\n" \
    "v_mov_b64 v[192:193], 0\n" \
    "v_mov_b64 v[194:195], 0\n" \
    "v_mov_b64 v[196:197], 0\n" \
    "v_mov_b64 v[198:199], 0\n" \
    "v_mov_b64 v[200:201], 0\n" \
    "v_mov_b64 v[202:203], 0\n" \
    "v_mov_b64 v[204:205], 0\n" \
    "v_mov_b64 v[206:207], 0\n" \
    "v_mov_b64 v[208:209], 0\n" \
    "v_mov_b64 v[210:211], 0\n" \
    "v_mov_b64 v[212:213], 0\n" \
    "v_mov_b64 v[214:215], 0\n" \
    "v_mov_b64 v[216:217], 0\n" \
    "v_mov_b64 v[218:219], 0\n" \
    "v_mov_b64 v[220:221], 0\n" \
    "v_mov_b64 v[222:223], 0\n" \
    "s_mov_b32 %[three], 0\n" \
    "s_add_u32 %[uend], %[uend], 32\n" \
    "s_add_u32 %[rowr], %[rowr], 2097152\n" \
    AG_STAMP_M \
    "s_branch 90b\n" \
    "95:\n" \
    ""
#define AG_ASM_CLOBBERS "v26", "v27", "v28", "v29", "v30", "v31", "v32", "v33", "v34", "v35", "v36", "v37", "v38", "v39", "v40", "v41", "v42", "v43", "v44", "v45", "v46", "v47", "v48", "v49", "v50", "v51", "v52", "v53", "v54", "v55", "v56", "v57", "v58", "v59", "v60", "v61", "v62", "v63", "v64", "v65", "v66", "v67", "v68", "v69", "v70", "v71", "v72", "v73", "v74", "v75", "v76", "v77", "v78", "v79", "v80", "v81", "v82", "v83", "v84", "v85", "v86", "v87", "v88", "v89", "v90", "v91", "v92", "v93", "v94", "v95", "v96", "v97", "v98", "v99", "v100", "v101", "v102", "v103", "v104", "v105", "v106", "v107", "v108", "v109", "v110", "v111", "v112", "v113", "v114", "v115", "v116", "v117", "v118", "v119", "v120", "v121", "v122", "v123", "v124", "v125", "v126", "v127", "v128", "v129", "v130", "v131", "v132", "v133", "v134", "v135", "v136", "v137", "v138", "v139", "v140", "v141", "v142", "v143", "v144", "v145", "v146", "v147", "v148", "v149", "v150", "v151", "v152", "v153", "v154", "v155", "v156", "v157", "v158", "v159", "v160", "v161", "v162", "v163", "v164", "v165", "v166", "v167", "v168", "v169", "v170", "v171", "v172", "v173", "v174", "v175", "v176", "v177", "v178", "v179", "v180", "v181", "v182", "v183", "v184", "v185", "v186", "v187", "v188", "v189", "v190", "v191", "v192", "v193", "v194", "v195", "v196", "v197", "v198", "v199", "v200", "v201", "v202", "v203", "v204", "v205", "v206", "v207", "v208", "v209", "v210", "v211", "v212", "v213", "v214", "v215", "v216", "v217", "v218", "v219", "v220", "v221", "v222", "v223", "v224", "v225", "v226", "v227", "v228", "v229", "v230", "v231", "v232", "v233", "v234", "v235", "v236", "v237", "v238", "v239", "v240", "v241", "v242", "v243", "v244", "v245", "v246", "v247", "v248", "v249", "v250", "v251", "v252", "v253", "v254", "v255", "s68", "s69", "s70", "s71", "s72", "s73", "s74", "s75", "s76", "s77", "s78", "s79"

struct Push {
    const unsigned char* xs;
    unsigned char* xd;
    GAS unsigned* fl;
    unsigned lo, so, call, perm;
    int wid, lane;
    u32x4 buf[PWIN];
    GAS unsigned long long* stamp;
};
__device__ __forceinline__ void push_load(Push& ps, int s, u32x4& d) {
    const unsigned char* p = ps.xs + (s >> 5) * (256 * KDIM) + (s & 31) * 128;
    asm volatile("s_nop 4\n\tglobal_load_dwordx4 %0, %1, %2" : "=&v"(d) : "v"(ps.lo), "s"(p) : "memory");
}
__device__ __forceinline__ void push_store(Push& ps, int s, u32x4& d) {
    unsigned char* p = ps.xd + s * 16384;
    asm volatile("" : "+v"(d));
    u32x4 e;
#pragma unroll
    for (int i = 0; i < 4; ++i) e[i] = (unsigned)__builtin_amdgcn_ds_bpermute((int)ps.perm, (int)d[i]);
    asm volatile("s_nop 4\n\tglobal_store_dwordx4 %1, %0, %2 sc0 sc1\n\ts_nop 1" : "+v"(e) : "v"(ps.so), "s"(p) : "memory");
}
template <int N> __device__ __forceinline__ void push_done(Push& ps, int s) {
    const int d = s - PWIN;
    const unsigned back = (unsigned)(d - ps.wid) & 7u;
    if (back != 0u && back < (unsigned)PWIN && (int)back <= d) wait_vm<N + 1>();
    else wait_vm<N>();
    __builtin_amdgcn_s_barrier();
    if (back == 0u) {
        unsigned t0, t1;
        const GAS unsigned* fp = ps.fl + (d + (d & 32)) * 2;
        asm volatile("s_nop 4\n\t"
                     "v_mov_b32 %0, 0\n\t"
                     "v_mov_b32 %1, %2\n\t"
                     "s_mov_b64 exec, 1\n\t"
                     "global_store_dword %0, %1, %3 sc0 sc1\n\t"
                     "s_mov_b64 exec, -1"
                     : "=&v"(t0), "=&v"(t1) : "s"(ps.call), "s"(fp) : "memory");
    }
}
constexpr int PMAIN = PWIN + (64 - 2 * PWIN) / PWIN * PWIN;
constexpr int CLR_ASK = (AG_LL0 / PWIN - 2) * PWIN;
static_assert(CLR_ASK >= PWIN && CLR_ASK + 2 * PWIN < PMAIN && AG_LL0 > NKT, "");
constexpr int ops_after(int from, int to) { int n = 0; for (int i = from; i < to; ++i) n += 1 + (i + PWIN < 64 ? 1 : 0); return n; }
template <int S> __device__ __forceinline__ void push_head(Push& ps) {
    if constexpr (S < PWIN) {
        wait_vm<PWIN - 1 + S>();
        push_store(ps, S, ps.buf[S]);
        push_load(ps, S + PWIN, ps.buf[S]);
        push_head<S + 1>(ps);
    }
}
template <int S> __device__ __forceinline__ void push_tail(Push& ps) {
    if constexpr (S < 64) {
        push_done<ops_after(S - PWIN + 1, S)>(ps, S);
        push_store(ps, S, ps.buf[S % PWIN]);
        if constexpr (S + PWIN < 64) push_load(ps, S + PWIN, ps.buf[S % PWIN]);
        push_tail<S + 1>(ps);
    } else if constexpr (S < 64 + PWIN) {
        push_done<63 - (S - PWIN)>(ps, S);
        push_tail<S + 1>(ps);
    }
}

__global__ void __launch_bounds__(NTHREADS, 2) __attribute__((amdgpu_waves_per_eu(2, 2))) ag_gemm_kernel(Args a) {
    extern __shared__ __attribute__((aligned(16))) unsigned char smem[];
    LAS unsigned char* lds = (LAS unsigned char*)smem;
    const int wid = __builtin_amdgcn_readfirstlane(threadIdx.x >> 6);
    const int b = blockIdx.x, xcd = b & 7, bi = b >> 3;
    const int rank = a.rank;
    unsigned char* myws = a.ws;
    unsigned char* pws = a.pws[xcd];
    const unsigned char* xin = a.x;
    asm volatile("" ::"s"(myws), "s"(pws), "s"(xin), "s"(rank));
    GAS unsigned* callp = (GAS unsigned*)(myws + WS_CALL) + (xcd * 32 + bi);

    if (__builtin_expect(bi < 4, 1)) {
        const int lane = lane_id(), tid = wid * 64 + lane;
        if (pws == nullptr) {
            const unsigned call = uni(__hip_atomic_load(callp, __ATOMIC_RELAXED, __HIP_MEMORY_SCOPE_AGENT)) + 1u;
            constexpr int PER = (NSTEPS - AG_LL0) * 16;
            unsigned char* g = myws + WS_GATHER + (size_t)(call & 1u) * NR * SHARD_BYTES + (size_t)AG_LL0 * 16384 + lane * 16;
            const u32x4 ones = {~0u, ~0u, ~0u, ~0u};
            for (int q = bi * 8 + wid; q < 14 * PER; q += 32) {
                const int c = q / PER, src = (rank + 1 + (c >> 1)) & 7;
                unsigned char* p = g + (size_t)src * SHARD_BYTES + (size_t)(c & 1) * NSTEPS * 16384 + (size_t)(q % PER) * 1024;
                asm volatile("global_store_dwordx4 %0, %1, off sc0 sc1" ::"v"(p), "v"(ones) : "memory");
            }
            wait_vm<0>();
            __syncthreads();
            if (tid < NR && a.pws[tid] != nullptr)
                __hip_atomic_store((GAS unsigned*)(a.pws[tid] + WS_CLR) + rank * 32 + bi, call, __ATOMIC_RELAXED, __HIP_MEMORY_SCOPE_SYSTEM);
            if (tid == 0) __hip_atomic_store(callp, call, __ATOMIC_RELAXED, __HIP_MEMORY_SCOPE_AGENT);
            return;
        }
        if (wid == 0) STAMP(0);
        Push ps;
        ps.wid = wid; ps.lane = lane;
        ps.lo = (unsigned)(lane >> 2) * KDIM + (unsigned)(lane & 3) * 16u;
        ps.perm = (unsigned)((lane & 15) * 4 + (lane >> 4)) * 4u;
        ps.xs = xin + (size_t)(bi * 64 + (wid >> 1) * 16) * KDIM + (wid & 1) * 64;
        unsigned callv;
        asm volatile("s_nop 4\n\tglobal_load_dword %0, %1, %2 sc1" : "=&v"(callv) : "v"(0u), "s"(callp) : "memory");
#pragma unroll
        for (int i = 0; i < PWIN; ++i) push_load(ps, i, ps.buf[i]);
        asm volatile("s_waitcnt vmcnt(%1)" : "+v"(callv) : "n"(PWIN) : "memory");
        const unsigned call = uni(callv) + 1u;
        STAMP(2 + wid);
        ps.call = call;
        ps.so = (unsigned)lane * 16u + (unsigned)(wid & 3) * 1024u + (unsigned)(wid >> 2) * 8192u;
        ps.xd = pws + WS_GATHER + (size_t)(call & 1u) * NR * SHARD_BYTES + (size_t)rank * SHARD_BYTES + (bi >> 1) * NSTEPS * 16384 + (bi & 1) * 4096;
        ps.fl = (GAS unsigned*)(pws + WS_FLAGS) + rank * 256 + (bi >> 1) * 64 + (bi & 1);
        push_head<0>(ps);
        const GAS unsigned* clr = (const GAS unsigned*)(myws + WS_CLR) + ((a.peers >> (4 * xcd)) & 7u) * 32 + (lane & 3);
        unsigned cv = 0;
#pragma unroll 1
        for (int s0 = PWIN; s0 < PMAIN; s0 += PWIN) {
            if (s0 == CLR_ASK) asm volatile("global_load_dword %0, %1, off sc0 sc1" : "=&v"(cv) : "v"(clr) : "memory");
            if (s0 == CLR_ASK + 2 * PWIN) {
                asm volatile("" : "+v"(cv));
                if (__ballot(cv < call) != 0ull) {
                    GAS unsigned* tmo = (GAS unsigned*)(myws + WS_TMO);
                    const unsigned long long t0 = __builtin_amdgcn_s_memrealtime();
                    while (__ballot(__hip_atomic_load(clr, __ATOMIC_RELAXED, __HIP_MEMORY_SCOPE_SYSTEM) < call) != 0ull) {
                        if (uni(__hip_atomic_load(tmo, __ATOMIC_RELAXED, __HIP_MEMORY_SCOPE_AGENT)) == call ||
                            __builtin_amdgcn_s_memrealtime() - t0 > DEADLINE_TICKS) {
                            __hip_atomic_store(tmo, call, __ATOMIC_RELAXED, __HIP_MEMORY_SCOPE_AGENT);
                            return;
                        }
                        __builtin_amdgcn_s_sleep(2);
                    }
                }
            }
#pragma unroll
            for (int i = 0; i < PWIN; ++i) {
                push_done<2 * (PWIN - 1)>(ps, s0 + i);
                push_store(ps, s0 + i, ps.buf[i]);
                push_load(ps, s0 + i + PWIN, ps.buf[i]);
            }
        }
        push_tail<PMAIN>(ps);
        if (tid == 0) __hip_atomic_store(callp, call, __ATOMIC_RELAXED, __HIP_MEMORY_SCOPE_AGENT);
        if (wid == 0) STAMP(1);
        return;
    }

    const int lane = lane_id(), tid = wid * 64 + lane;
    const int pj = (bi - 4) >> 2, par = xcd & 1;
    const int src = (rank + 1 + pj) & 7;
    const int ct = (xcd >> 1) * 4 + (bi & 3);
    const bool ext = pj < 4;
    const int own0 = (par * 4 + pj) * 64;
    if (wid == 0) STAMP(0);
    const unsigned call = uni(__hip_atomic_load(callp, __ATOMIC_RELAXED, __HIP_MEMORY_SCOPE_AGENT)) + 1u;
    if (tid < 2 * NSLOT * 32) *(LAS u32x4*)(lds + LDS_CTRL + tid * 16) = (u32x4){0u, 0u, 0u, 0u};
    __syncthreads();
    if (tid == 0) __hip_atomic_store(callp, call, __ATOMIC_RELAXED, __HIP_MEMORY_SCOPE_AGENT);

    if (wid == 7) {
        const int lane = lane_id();
        GAS unsigned* tmo = (GAS unsigned*)(myws + WS_TMO);
        const GAS unsigned* flags = (const GAS unsigned*)(myws + WS_FLAGS) + src * 256 + par * 64 + lane;
        const unsigned long long t0 = __builtin_amdgcn_s_memrealtime();
        {
            while (__builtin_amdgcn_s_memrealtime() - t0 < AG_XWARM) __builtin_amdgcn_s_sleep(1);
            u32x4 sink[10];
#pragma unroll
            for (int i = 0; i < 10; ++i) {
                const unsigned q = (unsigned)((b - 32) * 10 + i);
                asm volatile("global_load_dwordx4 %0, %1, %2" : "=&v"(sink[i]) : "v"((q < 2048u ? q : 0u) * 1024u + (unsigned)lane * 16u), "s"(xin) : "memory");
            }
            asm volatile("s_waitcnt vmcnt(0)" : "+v"(sink[0]), "+v"(sink[1]), "+v"(sink[2]), "+v"(sink[3]), "+v"(sink[4]),
                         "+v"(sink[5]), "+v"(sink[6]), "+v"(sink[7]), "+v"(sink[8]), "+v"(sink[9]) : : "memory");
        }
        unsigned rdy[2] = {0u, 0u}, opt[2] = {0u, 0u}, seen[2] = {0u, 0u}, spins = 0;
        for (;;) {
            const unsigned lo = rdy[0] < rdy[1] ? rdy[0] : rdy[1];
            if (lo >= (unsigned)NSTEPS) break;
            const unsigned f = __hip_atomic_load(flags + (lo >> 5) * 128, __ATOMIC_RELAXED, __HIP_MEMORY_SCOPE_SYSTEM);
            const unsigned long long m = __ballot(f >= call);
            bool any = false;
#pragma unroll
            for (int h = 0; h < 2; ++h) {
                const unsigned long long mh = (m >> h) | 0xAAAAAAAAAAAAAAAAull;
                const unsigned n = (lo & 32u) + ((mh == ~0ull) ? 32u : (unsigned)(__builtin_ctzll(~mh) >> 1));
                if (n > rdy[h]) {
                    const unsigned now = (unsigned)__builtin_amdgcn_s_memrealtime();
                    if ((unsigned)lane >= rdy[h] && (unsigned)lane < n) seen[h] = now;
                    unsigned lead = (unsigned)__builtin_popcountll(__ballot((unsigned)lane < n && now - seen[h] < (unsigned)AG_LWIN)) - 1u;
                    lead = lead < (unsigned)AG_LEAD ? lead : (unsigned)AG_LEAD;
                    const unsigned o = n < (unsigned)AG_LL0 ? n : (n + lead < (unsigned)NSTEPS ? n + lead : 127u);
                    opt[h] = o > opt[h] ? o : opt[h];
                    lds_write_uni(C_READY + 256u * h, n);
                    lds_write_uni(C_OPT + 256u * h, opt[h]);
                    rdy[h] = n;
                    any = true;
                }
            }
            if (any) continue;
            if ((++spins & 63u) == 0u &&
                (uni(__hip_atomic_load(tmo, __ATOMIC_RELAXED, __HIP_MEMORY_SCOPE_AGENT)) == call ||
                 __builtin_amdgcn_s_memrealtime() - t0 > DEADLINE_TICKS)) {
                __hip_atomic_store(tmo, call, __ATOMIC_RELAXED, __HIP_MEMORY_SCOPE_AGENT);
#pragma unroll
                for (int h = 0; h < 2; ++h) {
                    lds_write_uni(C_BAD + 256u * h, 1u);
                    lds_write_uni(C_READY + 256u * h, (unsigned)NSTEPS);
                    lds_write_uni(C_OPT + 256u * h, 127u);
                }
                break;
            }
            __builtin_amdgcn_s_sleep(1);
        }
        return;
    }

    if (wid >= 4) {
        const int lane = lane_id();
        const unsigned sw0 = (unsigned)((lane & 7) ^ (lane >> 4));
        const unsigned wo0 = (unsigned)(lane >> 3) * NL + sw0 * 16u, wo1 = (unsigned)(lane >> 3) * NL + (sw0 ^ 4u) * 16u;
        const unsigned xo0 = (unsigned)(lane >> 3) * KDIM + sw0 * 16u, xo1 = (unsigned)(8 + (lane >> 3)) * KDIM + (sw0 ^ 4u) * 16u;
        auto can = [&](int u) -> bool {
            const int v = (lane & 4) ? ((ext && u < NKT) ? u - 4 : -1) : u - NSLOT;
            const unsigned addr = C_FREE + (unsigned)((v < 0 ? 0 : v) % NSLOT) * 512u + (unsigned)(lane & 3) * 128u;
            return lds_all_ge(addr, (unsigned)(v + 1 < 0 ? 0 : v + 1));
        };
        auto issue = [&](int u) {
            const int kt = u & 31;
            const unsigned char* wp = a.w + ct * 128 + (size_t)kt * 128 * NL;
            LAS unsigned char* sb = lds + L_W + (u % NSLOT) * 16384;
#pragma unroll
            for (int p = 0; p < 16; ++p)
                __builtin_amdgcn_global_load_lds((const GAS void*)(wp + (size_t)p * 8 * NL + (((p >> 1) & 1) ? wo1 : wo0)),
                                                 (LAS void*)(sb + p * 1024), 16, 0, 0);
            if (ext && u < NKT) {
                const unsigned char* lp = xin + (size_t)own0 * KDIM + kt * 128;
                LAS unsigned char* so = lds + L_OWN + (u & 3) * 2048;
#pragma unroll
                for (int p = 0; p < 8; ++p)
                    __builtin_amdgcn_global_load_lds((const GAS void*)(lp + (size_t)(p >> 1) * 16 * KDIM + ((p & 1) ? xo1 : xo0)),
                                                     (LAS void*)(so + (p >> 1) * 8192 + (p & 1) * 1024), 16, 0, 0);
            }
        };
        issue(wid - 4);
#pragma unroll 1
        for (int u = wid - 4; u < NSTEPS; u += 3) {
            const int un = u + 3;
            const bool early = un < NSTEPS && can(un);
            if (early) {
                issue(un);
                if (ext && un < NKT) wait_vm<24>(); else wait_vm<16>();
            } else {
                wait_vm<0>();
            }
            lds_write_uni(C_FULL + (unsigned)(u % NSLOT) * 512u, (unsigned)(u + 1));
            lds_write_uni(C_FULL + (unsigned)(u % NSLOT) * 512u + 256u, (unsigned)(u + 1));
            STAMPL(112 + u);
            if (!early && un < NSTEPS) {
                while (!can(un)) __builtin_amdgcn_s_sleep(2);
                issue(un);
            }
        }
        return;
    }

    {
        const int fr = lane & 15, g = lane >> 4;
        const float scale = *(const CAS float*)a.sx * *(const CAS float*)a.sw;
        const int q = fr >> 1, p2 = fr & 1;
        const unsigned F = (unsigned)(((q >> 1) & 3) | ((g & 1) << 2));
        i32x8 epi, bb, misc;
#pragma unroll
        for (int n = 0; n < 8; ++n) bb[n] = (int)(L_W + (unsigned)((16 * g + q) * 128) + (((unsigned)n ^ F) * 16u) + (unsigned)p2 * 8u);
        misc[0] = (int)(L_OWN + (unsigned)(wid * 8192 + fr * 128 + ((g ^ (fr >> 1)) * 16)));
        misc[1] = misc[0] ^ 64;
        misc[2] = (int)(C_FREE + (unsigned)wid * 128u + (unsigned)(lane & 31) * 4u);
        misc[3] = (int)(C_FULL + (unsigned)(wid >> 1) * 256u);
        misc[4] = ((wid & 1) * 2 + (wid >> 1)) * 4096 + lane * 16;
        misc[5] = misc[6] = misc[7] = 0;
        const int c2 = lane & 31, h2 = lane >> 5;
        epi[0] = (int)(L_OWN + (unsigned)(wid * 8192 + fr * 512 + ((g ^ (fr & 7)) * 16)));
        epi[1] = epi[0] ^ 64;
#pragma unroll
        for (int i = 0; i < 4; ++i) epi[2 + i] = (int)(L_OWN + (unsigned)(wid * 8192 + (2 * i + h2) * 512 + ((c2 ^ (2 * i + h2)) * 16)));
        epi[6] = h2 * NL * 4 + c2 * 16;
        epi[7] = 0;
        const unsigned long long gx = (unsigned long long)(myws + WS_GATHER + (size_t)(call & 1u) * NR * SHARD_BYTES + (size_t)src * SHARD_BYTES +
                                                           (size_t)par * NSTEPS * 16384);
        const unsigned sz = NSTEPS * 16384;
        u32x4 srd = {(unsigned)gx, (unsigned)(gx >> 32) & 0xffffu, sz, 0x00020000u};
        const unsigned long long go = (unsigned long long)(a.out + ct * 128);
        const u32x4 ord = {(unsigned)go, (unsigned)(go >> 32) & 0xffffu, (unsigned)(NR * MS * NL * 4 - ct * 512), 0x00020000u};
        const int gw = (b - 32) * 4 + wid;
        const unsigned char* wp = a.w + (size_t)(gw * 10 < 8182 ? gw * 10 : 8182) * 1024 - ((wid & 1) * 2 + (wid >> 1)) * 4096;
        { const unsigned long long t0 = __builtin_amdgcn_s_memrealtime(); while (__builtin_amdgcn_s_memrealtime() - t0 < AG_WDELAY) __builtin_amdgcn_s_sleep(1); }
#define AG_WL(o) "global_load_dwordx4 v[32:35], v20, %[wp] offset:" #o "\n"
#define AG_WWARM AG_WL(-4096) AG_WL(-3072) AG_WL(-2048) AG_WL(-1024) AG_WL(0) AG_WL(1024) AG_WL(2048) AG_WL(3072) \
                 "global_load_dwordx4 v[32:35], v20, %[wp2] offset:0\nglobal_load_dwordx4 v[32:35], v20, %[wp2] offset:1024\n"
#define AG_WOPS , [wp] "s"(wp + 4096), [wp2] "s"(wp + 8192)
        __builtin_amdgcn_s_setprio(2);
        unsigned u = (unsigned)-4, xi = 0, uend = NKT, three = uni(ext ? 1u : 0u), sa = 0, sb = 1024, dl = 0, vc = 16, thr, n1, rv, so, c;
        unsigned long long dbg;
        unsigned rowr = (unsigned)(src * MS + par * 128 + wid * 32) * (NL * 4);
#define AG_STAMP_K ""
#define AG_STAMP_E ""
#define AG_STAMP_M ""
#define AG_STAMP_END "s_endpgm\n"
#define AG_STAMP_OPS
        asm volatile(AG_ASM_CONSUMER AG_STAMP_END
                     : "=&{v[24:25]}"(dbg), "+&{v[0:7]}"(epi), "+&{v[8:15]}"(bb), "+&{v[16:23]}"(misc), [sa] "+&s"(sa), [sb] "+&s"(sb), [dl] "+&s"(dl), [vc] "+&s"(vc), "+&{s[60:63]}"(srd), [u] "+&s"(u), [xi] "+&s"(xi), [uend] "+&s"(uend), [three] "+&s"(three),
                       [rowr] "+&s"(rowr), [thr] "=&s"(thr), [n1] "=&s"(n1), [rv] "=&s"(rv), [so] "=&s"(so), [c] "=&s"(c) AG_STAMP_OPS
                     : "{s[64:67]}"(ord), [sz] "s"(sz), [scale] "s"(uni(__float_as_uint(scale))),
                       [rowo] "s"((unsigned)(rank * MS + own0 + wid * 16) * (NL * 4)), [cj] "s"(AG_CODEPF ? (unsigned)(bi - 4) * 16384u : 65536u) AG_WOPS
                     : "memory", "vcc", "scc", AG_ASM_CLOBBERS);
        __builtin_unreachable();
    }
}

}

extern "C" void kernel_launch(int rank, int nranks, ncclComm_t comm,
                              hipStream_t stream,
                              void* const* d_in, const int* in_sizes, int n_in,
                              void* d_out, int out_size,
                              const int* in_dtypes, int out_dtype,
                              void* const* d_out_all,
                              void* const* d_ws_all, size_t ws_bytes) {
    using namespace agk;
    (void)comm; (void)in_dtypes; (void)out_dtype; (void)d_out_all;
    if (nranks != NR || n_in != 4 || in_sizes[0] != MS * KDIM || in_sizes[1] != KDIM * NL || out_size != NR * MS * NL ||
        ws_bytes < WS_NEEDED || (unsigned)rank >= (unsigned)NR) {
        fprintf(stderr, "ag_gemm: unexpected problem shape\n");
        return;
    }
    static bool attr_set = false;
    if (!attr_set) {
        HIP_CHECK(hipFuncSetAttribute((const void*)ag_gemm_kernel, hipFuncAttributeMaxDynamicSharedMemorySize, LDS_BYTES));
        attr_set = true;
    }
    Args a;
    a.x = (const unsigned char*)d_in[0];
    a.w = (const unsigned char*)d_in[1];
    a.sx = (const float*)d_in[2];
    a.sw = (const float*)d_in[3];
    a.out = (float*)d_out;
    a.ws = (unsigned char*)d_ws_all[rank];
    a.peers = peer_map(rank);
    a.rank = rank;
    for (int i = 0; i < NR; ++i) {
        const unsigned p = (peer_map(rank) >> (4 * i)) & 15u;
        a.pws[i] = p < (unsigned)NR ? (unsigned char*)d_ws_all[p] : nullptr;
    }
    hipLaunchKernelGGL(ag_gemm_kernel, dim3(NWG), dim3(NTHREADS), LDS_BYTES, stream, a);
    HIP_CHECK(hipGetLastError());
}
```
